```python
import numpy as np

kernel_name = "gpu_gemm_ar_m4096_k4096_n4096_e4m3e5m2_dqsilu"


ABSMAX_RTOL = 0.05

COMPUTE_DTYPE = "float32"
EPILOGUE = "dequant_silu"
IN_QUANT = None
OUT_QUANT = None
IN_FP8 = "e4m3e5m2"
IN_DTYPES = {"x": "e4m3", "w_mat": "e5m2",
             "scale_x": "float32", "scale_w": "float32"}
WORLD_SIZE = 8

def _snap_fp8(x, mant_bits, max_abs, min_normal):
    a = np.abs(x.astype(np.float64))
    _m, _e = np.frexp(a)
    q = np.where(a >= min_normal, np.ldexp(1.0, _e - 1 - mant_bits),
                 min_normal * 2.0 ** -mant_bits)
    snapped = np.minimum(np.round(a / q) * q, max_abs)
    return (np.sign(x) * snapped).astype(np.float32)

def setup_inputs():

    x = _snap_fp8(np.random.randn(4096, 4096).astype(np.float32), 3, 448.0, 2.0 ** -6)
    w_mat = _snap_fp8(np.random.randn(4096, 4096).astype(np.float32), 2, 57344.0, 2.0 ** -14)
    scale_x = (0.02 * (1.0 + np.random.rand(1))).astype(np.float32)
    scale_w = (0.02 * (1.0 + np.random.rand(1))).astype(np.float32)
    return {"x": x, "w_mat": w_mat, "scale_x": scale_x, "scale_w": scale_w}

def reference(x, w_mat, scale_x, scale_w):
    acc = (x @ w_mat).astype(np.float32)
    y = (acc * (scale_x * scale_w)).astype(np.float32)
    return (y / (1.0 + np.exp(-np.clip(y, -60.0, 60.0)))).astype(np.float32)
```

```cpp
#include <hip/hip_runtime.h>
#include <rccl/rccl.h>
#include <cstdio>

#define HIP_CHECK(expr) do { hipError_t _e = (expr); if (_e != hipSuccess) { \
    fprintf(stderr, "HIP error %s at %s:%d\n", hipGetErrorString(_e), __FILE__, __LINE__); \
    return; } } while (0)

namespace {

constexpr int NR = 8;
constexpr int DIM = 4096;
constexpr int KSH = DIM / NR;
constexpr int NSTEP = 32;
constexpr int SLAB = DIM * 16;
constexpr int STEPB = 8 * SLAB;
constexpr int NBLK = 256, NCOMM = 32, NPREP = 64;
constexpr int CWAVES = 7;
constexpr int WPP = NCOMM;
constexpr int UPP = 512;
constexpr int OFF_TICKET = 0, OFF_TMO = 128, OFF_CLAIM = 256;
constexpr int OFF_PF = 4096;
constexpr int OFF_UF = 8192;
constexpr int OFF_DBG = 65536;
constexpr int GSTEPS = 36;
constexpr int OFF_GX = 1 << 20, OFF_GW = OFF_GX + GSTEPS * STEPB;
constexpr size_t WS_NEED = (size_t)OFF_GW + (size_t)GSTEPS * STEPB;
constexpr int LDS_BYTES = 2 * (128 * 320 + 32768) + 64;
constexpr int LDS_BC = 2 * (128 * 320 + 32768);
constexpr unsigned long long DEADLINE = 30000000ull;

typedef int v8i __attribute__((ext_vector_type(8)));
typedef int v4i __attribute__((ext_vector_type(4)));
typedef unsigned u32x4 __attribute__((ext_vector_type(4)));
typedef unsigned u32x2 __attribute__((ext_vector_type(2)));
typedef float f32x4 __attribute__((ext_vector_type(4)));
#define LAS __attribute__((address_space(3)))
#define WAIT_V(n) asm volatile("s_waitcnt vmcnt(%0)" ::"i"(n) : "memory")
#define WAIT_L(n) asm volatile("s_waitcnt lgkmcnt(%0)" ::"i"(n) : "memory")
#define BAR __builtin_amdgcn_s_barrier()
#define SCHED __builtin_amdgcn_sched_barrier(0)

struct Params {
  const unsigned char* x;
  const unsigned char* w;
  const float* sx;
  const float* sw;
  float* out;
  char* const* ws_all;
  char* ws;
  int rank;
  int pad;
};

#define STAMP(ws, idx) do {} while (0)

__device__ __forceinline__ void st16_sys(void* p, u32x4 v) { asm volatile("global_store_dwordx4 %0, %1, off sc0 sc1\n\ts_nop 1" ::"v"(p), "v"(v) : "memory"); }
__device__ __forceinline__ void st4_sys(void* p, unsigned v) { asm volatile("global_store_dword %0, %1, off sc0 sc1" ::"v"(p), "v"(v) : "memory"); }
__device__ __forceinline__ unsigned ld4_sys(const void* p) {
  unsigned v; asm volatile("global_load_dword %0, %1, off sc0 sc1\n\ts_waitcnt vmcnt(0)" : "=&v"(v) : "v"(p) : "memory"); return v;
}
__device__ __forceinline__ void ticket_issue(unsigned& t, void* p) {
  unsigned long long keep;
  asm volatile("s_mov_b64 %1, exec\n\ts_mov_b64 exec, 1\n\tglobal_atomic_add %0, %2, %3, off sc0\n\ts_mov_b64 exec, %1"
               : "+v"(t), "=&s"(keep) : "v"(p), "v"(1u) : "memory");
}
__device__ __forceinline__ bool spin_fail(char* ws, unsigned call, unsigned long long t0) {
  unsigned* tmo = (unsigned*)(ws + OFF_TMO);
  if (__builtin_amdgcn_readfirstlane(ld4_sys(tmo)) == call) return true;
  if (__builtin_amdgcn_s_memrealtime() - t0 > DEADLINE) { st4_sys(tmo, call); return true; }
  return false;
}
__device__ __forceinline__ int gstep_off(int step, unsigned par) { return (step < 4 ? (int)par * 4 + step : step + 4) * STEPB; }

__device__ __forceinline__ void prep_load(const Params& P, char* shm, int b, int wid, int lane, u32x4 (&r)[8]) {
  const int a = b >> 4, tensor = (b >> 3) & 1, g = b & 7;
  if (tensor == 0) {
    auto rs = __builtin_amdgcn_make_buffer_rsrc((void*)P.x, 0, DIM * KSH, 0x00020000);
#pragma unroll
    for (int i = 0; i < 8; ++i) {
      const int R = wid * 64 + i * 8 + (lane >> 3);
      const int voff = (g * 512 + R) * KSH + a * 128 + (((lane & 7) ^ ((R >> 1) & 7)) << 4);
      __builtin_amdgcn_raw_ptr_buffer_load_lds(rs, (LAS void*)(shm + wid * 8192 + i * 1024), 16, voff, 0, 0, 0);
    }
    WAIT_V(0); BAR;
#pragma unroll
    for (int i = 0; i < 8; ++i) {
      const int R = i * 64 + lane;
      r[i] = *(const LAS u32x4*)((LAS char*)shm + R * 128 + ((wid ^ ((R >> 1) & 7)) << 4));
    }
  } else {
    const unsigned char* src = P.w + (size_t)(a * 128 + wid * 16) * DIM + g * 512 + lane * 4;
    unsigned d[2][16];
#pragma unroll
    for (int h = 0; h < 2; ++h)
#pragma unroll
      for (int k = 0; k < 16; ++k) d[h][k] = *(const unsigned*)(src + (size_t)k * DIM + h * 256);
#pragma unroll
    for (int h = 0; h < 2; ++h)
#pragma unroll
      for (int q = 0; q < 4; ++q) {
        const unsigned a0 = d[h][4 * q], a1 = d[h][4 * q + 1], a2 = d[h][4 * q + 2], a3 = d[h][4 * q + 3];
        const unsigned l01 = __builtin_amdgcn_perm(a1, a0, 0x05010400), h01 = __builtin_amdgcn_perm(a1, a0, 0x07030602);
        const unsigned l23 = __builtin_amdgcn_perm(a3, a2, 0x05010400), h23 = __builtin_amdgcn_perm(a3, a2, 0x07030602);
        r[h * 4 + 0][q] = __builtin_amdgcn_perm(l23, l01, 0x05040100);
        r[h * 4 + 1][q] = __builtin_amdgcn_perm(l23, l01, 0x07060302);
        r[h * 4 + 2][q] = __builtin_amdgcn_perm(h23, h01, 0x05040100);
        r[h * 4 + 3][q] = __builtin_amdgcn_perm(h23, h01, 0x07060302);
      }
  }
}
__device__ __forceinline__ void prep_store(const Params& P, int b, int wid, int lane, unsigned call, const u32x4 (&r)[8]) {
  const int a = b >> 4, tensor = (b >> 3) & 1, g = b & 7;
  char* dst = P.ws + (tensor ? OFF_GW : OFF_GX) + gstep_off(a, call & 1) + wid * SLAB + g * 8192 + lane * 16;
#pragma unroll
  for (int i = 0; i < 8; ++i) st16_sys(dst + i * 1024, r[i]);
  WAIT_V(0);
  if (lane == 0) st4_sys(P.ws + OFF_UF + (((wid * 2 + tensor) * 8 + g) * 32 + a) * 4, call);
  BAR;
  if (wid == 0 && lane < NR - 1) {
    const unsigned p = (unsigned)(P.rank + 1 + lane) & (NR - 1);
    st4_sys(P.ws_all[p] + OFF_PF + P.rank * 256 + (tensor * 32 + a * 8 + g) * 4, call);
  }
}

struct Unit { const char* src; char* dst; char* flag; int pf; };
__device__ __forceinline__ Unit unit_of(const Params& P, const char* peer, int q, int k, unsigned call, int lane) {
  const int c = k >> 4, tensor = k & 1, g = (k >> 1) & 7;
  const int ci = c * 7 + q, step = 4 + (ci >> 3), slot = ci & 7;
  const int base = tensor ? OFF_GW : OFF_GX;
  Unit u;
  u.src = peer + base + gstep_off(c >> 3, call & 1) + (c & 7) * SLAB + g * 8192 + lane * 16;
  u.dst = P.ws + base + gstep_off(step, 0) + slot * SLAB + g * 8192 + lane * 16;
  u.flag = P.ws + OFF_UF + (((slot * 2 + tensor) * 8 + g) * 32 + step) * 4;
  u.pf = tensor * 32 + (c >> 3) * 8 + g;
  return u;
}
#define LDU(r, s, i) asm volatile("global_load_dwordx4 %0, %1, off offset:%2 sc0 sc1" : "=v"(r[i]) : "v"((s) + ((i) >> 2) * 4096), "i"(((i) & 3) * 1024) : "memory")
constexpr int WIN = 2;

__device__ __forceinline__ void comm_main(const Params& P, int b, int q, int lane, unsigned call) {
  const unsigned p = (unsigned)(P.rank + 1 + q) & (NR - 1);
  const char* peer = P.ws_all[p];
  const char* pfp = P.ws + OFF_PF + p * 256 + lane * 4;
  unsigned* claim = (unsigned*)(P.ws + OFF_CLAIM + q * 128);
  const unsigned cbase = (call - 1) * (UPP + WPP);
  unsigned long long ready = 0;
  bool dead = false;
  auto need = [&](int pf) {
    if (dead || ((ready >> pf) & 1)) return;
    const unsigned long long t0 = __builtin_amdgcn_s_memrealtime();
    while (true) {
      ready = __ballot(ld4_sys(pfp) >= call);
      if ((ready >> pf) & 1) break;
      if (spin_fail(P.ws, call, t0)) { dead = true; break; }
      __builtin_amdgcn_s_sleep(1);
    }
  };
  u32x4 r0[8], r1[8];
  unsigned tk = 0;
  ticket_issue(tk, claim); WAIT_V(0); asm volatile("" : "+v"(tk));
  int k = (int)(__builtin_amdgcn_readfirstlane(tk) - cbase);
  if (k >= UPP) return;
  ticket_issue(tk, claim); WAIT_V(0); asm volatile("" : "+v"(tk));
  int kn = (int)(__builtin_amdgcn_readfirstlane(tk) - cbase);
  Unit u = unit_of(P, peer, q, k, call, lane);
  need(u.pf);
#pragma unroll
  for (int i = 0; i < WIN; ++i) LDU(r0, u.src, i);
  char* pflag = nullptr;
  auto body = [&](u32x4 (&rc)[8], u32x4 (&rn)[8]) -> bool {
    const bool more = kn < UPP;
    if (more) ticket_issue(tk, claim);
#pragma unroll
    for (int i = WIN; i < 8; ++i) {
      WAIT_V(WIN - 1);
      if (i == WIN + 1 && pflag) { if (lane == 0) st4_sys(pflag, call); }
      LDU(rc, u.src, i);
    }
    Unit un = u;
    int k2 = UPP;
    if (more) {
      asm volatile("" : "+v"(tk));
      k2 = (int)(__builtin_amdgcn_readfirstlane(tk) - cbase);
      un = unit_of(P, peer, q, kn, call, lane);
      need(un.pf);
#pragma unroll
      for (int i = 0; i < WIN; ++i) { WAIT_V(WIN - 1); LDU(rn, un.src, i); }
      WAIT_V(WIN);
    } else {
      WAIT_V(0);
    }
#pragma unroll
    for (int i = 0; i < 8; ++i) asm volatile("" : "+v"(rc[i]));
#pragma unroll
    for (int i = 0; i < 8; ++i) st16_sys(u.dst + i * 1024, rc[i]);
    pflag = u.flag;
    u = un; kn = k2;
    return more;
  };
  while (true) {
    if (!body(r0, r1)) break;
    if (!body(r1, r0)) break;
  }
  WAIT_V(0);
  if (lane == 0) st4_sys(pflag, call);
}

__device__ __forceinline__ float silu1(float a, float s) {
  const float y = a * s;
  const float c = __builtin_amdgcn_fmed3f(y, -60.f, 60.f);
  const float e = __builtin_amdgcn_exp2f(c * -1.44269504088896341f);
  return y * __builtin_amdgcn_rcpf(1.f + e);
}

template <int MA0, int MA1>
__device__ __forceinline__ void gemm_main(const Params& P, char* shm, int brow, int bcol, int wid, int lane, unsigned call) {
  constexpr int RA0 = 32 * MA0, RA1 = 32 * MA1, BM = RA0 + RA1;
  constexpr int SLA0 = RA0 * 16, SLA1 = RA1 * 16, SLB = 128 * 16;
  constexpr int A0B = 8 * SLA0, A1B = 8 * SLA1, BB = 8 * SLB, BUFB = A0B + A1B + 2 * BB;
  constexpr int NA0 = (RA0 + 63) / 64, NA1 = (RA1 + 63) / 64;
  static_assert(2 * BUFB <= LDS_BC, "LDS");
  const int wr = wid >> 2, wc = wid & 3, fr = lane & 15, fq = lane >> 4;
  const unsigned par = call & 1;
  auto rX = __builtin_amdgcn_make_buffer_rsrc((void*)(P.ws + OFF_GX), 0, GSTEPS * STEPB, 0x00020000);
  auto rW = __builtin_amdgcn_make_buffer_rsrc((void*)(P.ws + OFF_GW), 0, GSTEPS * STEPB, 0x00020000);
  const int voff = lane * 16;
  const int sA = wid * SLAB + brow * 16, sB = wid * SLAB + bcol * 16;
  const char* pollp;
  { const int which = fq == 3 ? 0 : fq;
    const int tg = which == 0 ? (brow >> 9) : which == 1 ? ((brow + BM - 1) >> 9) : 8 + (bcol >> 9);
    pollp = P.ws + OFF_UF + ((wid * 16 + tg) * 32 + fr * 2) * 4; }
  u32x2 pv;
  unsigned rdy0 = 0, rdy1 = 0;
  bool dead = false;
#define POLL_ISSUE() asm volatile("global_load_dwordx2 %0, %1, off sc0 sc1" : "=v"(pv) : "v"(pollp) : "memory")
#define POLL_TAKE() do { asm volatile("" : "+v"(pv)); const unsigned long long m0_ = __ballot(pv.x >= call), m1_ = __ballot(pv.y >= call); \
    rdy0 = (unsigned)(m0_ & (m0_ >> 16) & (m0_ >> 32)) & 0xffffu; rdy1 = (unsigned)(m1_ & (m1_ >> 16) & (m1_ >> 32)) & 0xffffu; } while (0)
#define HAVE(s) (((((s) & 1) ? rdy1 : rdy0) >> ((s) >> 1)) & 1u)
#define NEED(s) do { if (!HAVE(s) && !dead) { const unsigned long long t0_ = __builtin_amdgcn_s_memrealtime(); \
    while (true) { POLL_ISSUE(); WAIT_V(0); POLL_TAKE(); if (HAVE(s)) break; if (spin_fail(P.ws, call, t0_)) { dead = true; break; } __builtin_amdgcn_s_sleep(2); } \
    WAIT_L(0); } } while (0)
#define OFF_A0(b) ((b) * BUFB)
#define OFF_A1(b) ((b) * BUFB + A0B)
#define OFF_B(b, h) ((b) * BUFB + A0B + A1B + (h) * BB)
#define GL(rs, ldsoff, soff, imm) __builtin_amdgcn_raw_ptr_buffer_load_lds(rs, (LAS void*)(shm + (ldsoff)), 16, voff, soff, imm, 17)
#define STAGE_A0(b, go) do { const int so_ = sA + (go), lo_ = OFF_A0(b) + wid * SLA0; GL(rX, lo_, so_, 0); GL(rX, lo_, so_, 1024); \
    if (NA0 == 3) { if (lane < 32) GL(rX, lo_, so_, 2048); } } while (0)
#define STAGE_A1(b, go) do { const int so_ = sA + RA0 * 16 + (go), lo_ = OFF_A1(b) + wid * SLA1; GL(rX, lo_, so_, 0); GL(rX, lo_, so_, 1024); \
    if (NA1 == 3) { if (lane < 32) GL(rX, lo_, so_, 2048); } } while (0)
#define STAGE_B(b, h, go) do { const int so_ = sB + (h) * 2048 + (go), lo_ = OFF_B(b, h) + wid * SLB; GL(rW, lo_, so_, 0); GL(rW, lo_, so_, 1024); } while (0)
  const LAS char* lsm = (const LAS char*)shm;
  const LAS char *pa0[2], *pa1[2], *pb[2];
#pragma unroll
  for (int b = 0; b < 2; ++b) {
    pa0[b] = lsm + OFF_A0(b) + fq * SLA0 + (wr * 16 * MA0 + fr) * 16;
    pa1[b] = lsm + OFF_A1(b) + fq * SLA1 + (wr * 16 * MA1 + fr) * 16;
    pb[b] = lsm + OFF_B(b, 0) + fq * SLB + (wc * 16 + fr) * 16;
    asm volatile("" : "+v"(pa0[b])); asm volatile("" : "+v"(pa1[b])); asm volatile("" : "+v"(pb[b]));
  }
#define LD8(p, sl) __builtin_shufflevector(*(const LAS v4i*)(p), *(const LAS v4i*)((p) + 4 * (sl)), 0, 1, 2, 3, 4, 5, 6, 7)
#define LDA0(b) _Pragma("unroll") for (int m = 0; m < MA0; ++m) At[m] = LD8(pa0[b] + m * 256, SLA0)
#define LDA1(b) _Pragma("unroll") for (int m = 0; m < MA1; ++m) At[m] = LD8(pa1[b] + m * 256, SLA1)
#define LDB(dst, b, h) _Pragma("unroll") for (int n = 0; n < 2; ++n) dst[n] = LD8(pb[b] + (h) * BB + n * 1024, SLB)
#define MFMA(c, a, b) asm volatile("v_mfma_f32_16x16x128_f8f6f4 %0, %1, %2, %0 blgp:1" : "+v"(c) : "v"(a), "v"(b))
#define MMA0(bj, Bf) do { __builtin_amdgcn_s_setprio(1); _Pragma("unroll") for (int m = 0; m < MA0; ++m) _Pragma("unroll") for (int n = 0; n < 2; ++n) \
    MFMA(acc0[m][(bj) * 2 + n], At[m], Bf[n]); __builtin_amdgcn_s_setprio(0); } while (0)
#define MMA1(bj, Bf) do { __builtin_amdgcn_s_setprio(1); _Pragma("unroll") for (int m = 0; m < MA1; ++m) _Pragma("unroll") for (int n = 0; n < 2; ++n) \
    MFMA(acc1[m][(bj) * 2 + n], At[m], Bf[n]); __builtin_amdgcn_s_setprio(0); } while (0)
#define TILE(b, nb, t) do { POLL_TAKE(); NEED((t) + 1); const int go_ = gstep_off((t) + 1, par); \
    LDB(B0, b, 0); LDA0(b); STAGE_B(nb, 0, go_); STAGE_A0(nb, go_); POLL_ISSUE(); \
    WAIT_V(NA1 + 2 + NA0 + 1); BAR; WAIT_L(0); MMA0(0, B0); BAR; SCHED; \
    LDB(B1, b, 1); STAGE_B(nb, 1, go_); STAGE_A1(nb, go_); \
    WAIT_V(2 + NA0 + 1 + 2 + NA1); BAR; WAIT_L(0); MMA0(1, B1); BAR; SCHED; \
    LDA1(b); BAR; WAIT_L(0); MMA1(0, B0); BAR; SCHED; \
    WAIT_V(2 + NA1); BAR; MMA1(1, B1); BAR; SCHED; } while (0)
  f32x4 acc0[MA0][4] = {}, acc1[MA1][4] = {};
  v8i At[MA0 > MA1 ? MA0 : MA1], B0[2], B1[2];
  NEED(0);
  STAMP(P.ws, 2);
  { const int go = gstep_off(0, par);
    STAGE_B(0, 0, go); STAGE_A0(0, go); POLL_ISSUE(); STAGE_B(0, 1, go); STAGE_A1(0, go); }
  if (wr == 1) BAR;
  WAIT_V(2 + NA1); BAR; BAR; SCHED;
  for (int t = 0; t < NSTEP - 2; t += 2) {
    TILE(0, 1, t);
    TILE(1, 0, t + 1);
  }
  TILE(0, 1, NSTEP - 2);
  { LDB(B0, 1, 0); LDA0(1);
    WAIT_V(NA1); BAR; WAIT_L(0); MMA0(0, B0); BAR; SCHED;
    LDB(B1, 1, 1);
    WAIT_V(0); BAR; WAIT_L(0); MMA0(1, B1); BAR; SCHED;
    LDA1(1); BAR; WAIT_L(0); MMA1(0, B0); MMA1(1, B1); }
  asm volatile("s_nop 15\n\ts_nop 15"); SCHED;
  if (wr == 0) BAR;
  asm volatile("" : "+v"(pv));
  STAMP(P.ws, 3);
  const bool bad = __builtin_amdgcn_readfirstlane(ld4_sys(P.ws + OFF_TMO)) == call;
  const float scale = bad ? __builtin_nanf("") : P.sx[0] * P.sw[0];
  int tz = threadIdx.x; asm volatile("" : "+v"(tz));
  const int cv = (((tz & 63) >> 4) * 4 * DIM + (tz & 15) * 4) * 4;
  auto rC = __builtin_amdgcn_make_buffer_rsrc((void*)P.out, 0, DIM * DIM * 4, 0x00020000);
#define ST_TILE(acc, row0) _Pragma("unroll") for (int j = 0; j < 4; ++j) { \
    const f32x4 o_ = {silu1(acc[0][j], scale), silu1(acc[1][j], scale), silu1(acc[2][j], scale), silu1(acc[3][j], scale)}; \
    const int so_ = (((row0) + j) * DIM + bcol + wc * 64) * 4; \
    asm volatile("buffer_store_dwordx4 %0, %1, %2, %3 offen\n\ts_nop 1" ::"v"(o_), "v"(cv), "s"(rC), "s"(so_) : "memory"); }
#pragma unroll
  for (int m = 0; m < MA0; ++m) { ST_TILE(acc0[m], brow + wr * 16 * MA0 + m * 16); }
#pragma unroll
  for (int m = 0; m < MA1; ++m) { ST_TILE(acc1[m], brow + RA0 + wr * 16 * MA1 + m * 16); }
}

__global__ __launch_bounds__(512, 2) void fused_kernel(Params P) {
  extern __shared__ __attribute__((aligned(16))) char shm[];
  const int b = blockIdx.x;
  const int wid = __builtin_amdgcn_readfirstlane(threadIdx.x >> 6), lane = threadIdx.x & 63;
  STAMP(P.ws, 0);
  unsigned tk = 0;
  if (wid == 7) ticket_issue(tk, P.ws + OFF_TICKET);
  u32x4 r[8];
  if (b < NPREP) prep_load(P, shm, b, wid, lane, r);
  if (wid == 7) {
    WAIT_V(0); asm volatile("" : "+v"(tk));
    *(LAS unsigned*)((LAS char*)shm + LDS_BC) = __builtin_amdgcn_readfirstlane(tk) / NBLK + 1;
  }
  WAIT_L(0); BAR; asm volatile("" ::: "memory");
  const unsigned call = __builtin_amdgcn_readfirstlane(*(volatile LAS unsigned*)((LAS char*)shm + LDS_BC));
  if (b < NPREP) prep_store(P, b, wid, lane, call, r);
  STAMP(P.ws, 1);
  if (b < NCOMM) {
    if (wid < CWAVES) comm_main(P, b, (wid + b) % CWAVES, lane, call);
    STAMP(P.ws, 5);
    return;
  }
  WAIT_V(0);
  const int xcd = b & 7, idx = (b - NCOMM) >> 3;
  const int rb = idx % 7, bcol = ((xcd >> 1) * 4 + idx / 7) * 256;
  const int brow = (xcd & 1) * 2048 + (rb == 0 ? 0 : 32 + rb * 288);
  if (rb == 0) gemm_main<5, 5>(P, shm, brow, bcol, wid, lane, call);
  else gemm_main<5, 4>(P, shm, brow, bcol, wid, lane, call);
}

}

extern "C" void kernel_launch(int rank, int nranks, ncclComm_t comm, hipStream_t stream, void* const* d_in, const int* in_sizes, int n_in,
                              void* d_out, int out_size, const int* in_dtypes, int out_dtype, void* const* d_out_all,
                              void* const* d_ws_all, size_t ws_bytes) {
  if (nranks != NR || n_in != 4 || in_sizes[0] != DIM * KSH || in_sizes[1] != KSH * DIM || out_size != DIM * DIM || ws_bytes < WS_NEED) {
    fprintf(stderr, "unexpected problem shape\n");
    return;
  }
  constexpr int lds = LDS_BYTES;
  HIP_CHECK(hipFuncSetAttribute((const void*)fused_kernel, hipFuncAttributeMaxDynamicSharedMemorySize, lds));
  Params P;
  P.x = (const unsigned char*)d_in[0];
  P.w = (const unsigned char*)d_in[1];
  P.sx = (const float*)d_in[2];
  P.sw = (const float*)d_in[3];
  P.out = (float*)d_out;
  P.ws_all = (char* const*)d_ws_all;
  P.ws = (char*)d_ws_all[rank];
  P.rank = rank;
  P.pad = 0;
  hipLaunchKernelGGL(fused_kernel, dim3(NBLK), dim3(512), lds, stream, P);
  HIP_CHECK(hipGetLastError());
}
```

```cpp
#include <hip/hip_runtime.h>
#include <rccl/rccl.h>
#include <cstdio>

#define HIP_CHECK(expr) do { hipError_t _e = (expr); if (_e != hipSuccess) { \
    fprintf(stderr, "HIP error %s at %s:%d\n", hipGetErrorString(_e), __FILE__, __LINE__); \
    return; } } while (0)

namespace {

constexpr int NR = 8;
constexpr int DIM = 4096;
constexpr int KSH = DIM / NR;
constexpr int NSTEP = 32;
constexpr int SLAB = DIM * 16;
constexpr int STEPB = 8 * SLAB;
constexpr int NBLK = 256, NCOMM = 32;
constexpr int UPP = 512;
constexpr int RAWC = 2;
constexpr int NPULL = UPP - 16 * RAWC;
#define LPP 9
#define WIN 8
constexpr int ECAP = 126;
constexpr int OFF_CNT = 0, OFF_TMO = 128, OFF_CLAIM = 256;
constexpr int OFF_PCNT = 2048;
constexpr int OFF_PF = 4096;
constexpr int OFF_UF = 8192;
constexpr int OFF_DBG = 65536;
constexpr int PSTEPS = 6;
constexpr int GSTEPS = NSTEP + PSTEPS;
constexpr int OFF_GX = 1 << 20, OFF_GW = OFF_GX + GSTEPS * STEPB;
constexpr int OFF_PK = OFF_GW + GSTEPS * STEPB;
constexpr int PK_EXC = 7168;
constexpr int WS_NEED = OFF_PK + 2 * UPP * 8192;
constexpr int LDS_BYTES = 2 * (128 * 320 + 32768);
constexpr unsigned long long DEADLINE = 30000000ull;

typedef int v8i __attribute__((ext_vector_type(8)));
typedef int v4i __attribute__((ext_vector_type(4)));
typedef unsigned u32x4 __attribute__((ext_vector_type(4)));
typedef unsigned u32x2 __attribute__((ext_vector_type(2)));
typedef float f32x4 __attribute__((ext_vector_type(4)));
#define LAS __attribute__((address_space(3)))
#define WAIT_V(n) asm volatile("s_waitcnt vmcnt(%0)" ::"i"(n) : "memory")
#define WAIT_L(n) asm volatile("s_waitcnt lgkmcnt(%0)" ::"i"(n) : "memory")
#define BAR __builtin_amdgcn_s_barrier()
#define SCHED __builtin_amdgcn_sched_barrier(0)

struct Params {
  const unsigned char* x;
  const unsigned char* w;
  const float* sx;
  const float* sw;
  float* out;
  char* const* ws_all;
  char* ws;
  int rank;
  int pad;
};

#define STAMP(ws, idx) do {} while (0)
#define DBG(x)

__device__ __forceinline__ void st16_sys(void* p, u32x4 v) { asm volatile("global_store_dwordx4 %0, %1, off sc0 sc1\n\ts_nop 1" ::"v"(p), "v"(v) : "memory"); }
__device__ __forceinline__ void st4_sys(void* p, unsigned v) { asm volatile("global_store_dword %0, %1, off sc0 sc1" ::"v"(p), "v"(v) : "memory"); }
__device__ __forceinline__ unsigned ld4_sys(const void* p) {
  unsigned v; asm volatile("global_load_dword %0, %1, off sc0 sc1\n\ts_waitcnt vmcnt(0)" : "=&v"(v) : "v"(p) : "memory"); return v;
}
__device__ __forceinline__ void ticket_issue(unsigned& t, void* p, unsigned long long lanes) {
  unsigned long long keep;
  asm volatile("s_mov_b64 %1, exec\n\ts_mov_b64 exec, %4\n\tglobal_atomic_add %0, %2, %3, off sc0\n\ts_mov_b64 exec, %1"
               : "+v"(t), "=&s"(keep) : "v"(p), "v"(1u), "s"(lanes) : "memory");
}
__device__ __forceinline__ bool spin_fail(char* ws, unsigned call, unsigned long long t0) {
  unsigned* tmo = (unsigned*)(ws + OFF_TMO);
  if (__builtin_amdgcn_readfirstlane(ld4_sys(tmo)) == call) return true;
  if (__builtin_amdgcn_s_memrealtime() - t0 > DEADLINE) { st4_sys(tmo, call); return true; }
  return false;
}
__device__ __forceinline__ int gstep_off(int step, unsigned par) { return (step < PSTEPS ? (int)par * PSTEPS + step : step + PSTEPS) * STEPB; }

typedef __amdgpu_buffer_rsrc_t rsrc_t;
#define BST16(v, rs, voff, soff, imm) asm volatile("buffer_store_dwordx4 %0, %1, %2, %3 offen offset:%4 sc0 sc1\n\ts_nop 1" ::"v"(v), "v"(voff), "s"(rs), "s"(soff), "i"(imm) : "memory")
#define BLD16(v, rs, voff, soff, imm) asm volatile("buffer_load_dwordx4 %0, %1, %2, %3 offen offset:%4 sc0 sc1" : "=v"(v) : "v"(voff), "s"(rs), "s"(soff), "i"(imm) : "memory")
__device__ __forceinline__ void store_unit(const u32x4 (&r)[8], int n, rsrc_t rs, int voff, int soff) {
#pragma unroll
  for (int i = 0; i < 8; ++i)
    if (i < n) BST16(r[i], rs, voff, soff + (i >> 2) * 4096, (i & 3) * 1024);
}
__device__ __forceinline__ int slab_off(int tensor, int gstep_bytes, int slot, int g) { return (tensor ? OFF_GW : OFF_GX) + gstep_bytes + slot * SLAB + g * 8192; }
__device__ __forceinline__ int uflag_off(int tensor, int step, int slot, int g) { return OFF_UF + (((slot * 2 + tensor) * 8 + g) * 32 + step) * 4; }

__device__ __forceinline__ void prep_load(const Params& P, int c, int tensor, int g, int lane, u32x4 (&r)[8]) {
  if (tensor == 0) {
    auto rs = __builtin_amdgcn_make_buffer_rsrc((void*)P.x, 0, DIM * KSH, 0x00020000);
#pragma unroll
    for (int i = 0; i < 8; ++i) r[i] = __builtin_amdgcn_raw_buffer_load_b128(rs, lane * KSH, (g * 512 + i * 64) * KSH + c * 16, 0);
  } else {
    auto rs = __builtin_amdgcn_make_buffer_rsrc((void*)P.w, 0, DIM * KSH, 0x00020000);
    unsigned d[2][16];
#pragma unroll
    for (int k = 0; k < 16; ++k)
#pragma unroll
      for (int h = 0; h < 2; ++h) d[h][k] = __builtin_amdgcn_raw_buffer_load_b32(rs, lane * 4 + h * 256, (c * 16 + k) * DIM + g * 512, 0);
#pragma unroll
    for (int h = 0; h < 2; ++h)
#pragma unroll
      for (int q = 0; q < 4; ++q) {
        const unsigned a0 = d[h][4 * q], a1 = d[h][4 * q + 1], a2 = d[h][4 * q + 2], a3 = d[h][4 * q + 3];
        const unsigned l01 = __builtin_amdgcn_perm(a1, a0, 0x05010400), h01 = __builtin_amdgcn_perm(a1, a0, 0x07030602);
        const unsigned l23 = __builtin_amdgcn_perm(a3, a2, 0x05010400), h23 = __builtin_amdgcn_perm(a3, a2, 0x07030602);
        r[h * 4 + 0][q] = __builtin_amdgcn_perm(l23, l01, 0x05040100);
        r[h * 4 + 1][q] = __builtin_amdgcn_perm(l23, l01, 0x07060302);
        r[h * 4 + 2][q] = __builtin_amdgcn_perm(h23, h01, 0x05040100);
        r[h * 4 + 3][q] = __builtin_amdgcn_perm(h23, h01, 0x07060302);
      }
  }
}

template <int TENSOR>
__device__ __forceinline__ unsigned pack_unit(u32x4 (&r)[8], LAS char* lds  , int lane) {
  constexpr unsigned S = 0x80808080u, CM = TENSOR ? 0x1f1f1f1fu : 0x3f3f3f3fu;
  unsigned sum = 0;
#pragma unroll
  for (int i = 0; i < 8; ++i) {
    *(LAS u32x4*)(lds + i * 1024 + lane * 16) = r[i];
#pragma unroll
    for (int d = 0; d < 4; ++d) sum = __builtin_amdgcn_sad_u8(r[i][d] & ~S, 0u, sum);
  }
  sum += (unsigned)__builtin_amdgcn_update_dpp(0, (int)sum, 0xb1, 0xf, 0xf, true);
  sum += (unsigned)__builtin_amdgcn_update_dpp(0, (int)sum, 0x4e, 0xf, 0xf, true);
  sum += (unsigned)__builtin_amdgcn_update_dpp(0, (int)sum, 0x141, 0xf, 0xf, true);
  sum += (unsigned)__builtin_amdgcn_update_dpp(0, (int)sum, 0x140, 0xf, 0xf, true);
  const int total = __builtin_amdgcn_readlane((int)sum, 0) + __builtin_amdgcn_readlane((int)sum, 16) + __builtin_amdgcn_readlane((int)sum, 32) + __builtin_amdgcn_readlane((int)sum, 48);
  int c0 = (total - (TENSOR ? 166707 : 343245)) >> 13;
  c0 = c0 < 0 ? 0 : c0 > (TENSOR ? 96 : 64) ? (TENSOR ? 96 : 64) : c0;
  const unsigned C0 = (unsigned)c0 * 0x01010101u;
  unsigned M[4] = {0, 0, 0, 0};
#pragma unroll
  for (int i = 0; i < 8; ++i)
#pragma unroll
    for (int d = 0; d < 4; ++d) {
      const unsigned v = r[i][d], t = (v | S) - C0;
      const unsigned e = (TENSOR ? (~t | (t << 1) | (t << 2)) : (~t | (t << 1))) & S;
      M[d] |= e >> i;
      r[i][d] = (t & CM) | (v & S);
    }
#pragma unroll
  for (int d = 0; d < 4; ++d) {
    if (TENSOR == 0) {
      const unsigned q = r[7][d];
#pragma unroll
      for (int i = 0; i < 6; ++i) r[i][d] |= (q << (6 - i)) & 0x40404040u;
      r[6][d] |= (q >> 1) & 0x40404040u;
    } else {
      const unsigned q3 = r[3][d], q7 = r[7][d];
      r[0][d] |= (q3 << 5) & 0x60606060u; r[1][d] |= (q3 << 3) & 0x60606060u; r[2][d] |= ((q3 << 1) & 0x20202020u) | ((q3 >> 1) & 0x40404040u);
      r[3][d] = r[4][d] | ((q7 << 5) & 0x60606060u); r[4][d] = r[5][d] | ((q7 << 3) & 0x60606060u);
      r[5][d] = r[6][d] | ((q7 << 1) & 0x20202020u) | ((q7 >> 1) & 0x40404040u);
    }
  }
  LAS unsigned short* list = (LAS unsigned short*)(lds + 8192);
  int n = 0;
#pragma unroll
  for (int d = 0; d < 4; ++d) {
    unsigned m = M[d];
    while (n <= ECAP) {
      const unsigned long long any = __ballot(m != 0);
      if (!any) break;
      if (m != 0) {
        const int b = __builtin_ctz(m);
        m &= m - 1;
        const int idx = n + (int)__builtin_amdgcn_mbcnt_hi((unsigned)(any >> 32), __builtin_amdgcn_mbcnt_lo((unsigned)any, 0u));
        if (idx < ECAP) list[2 + idx] = (unsigned short)((7 - (b & 7)) * 1024 + lane * 16 + d * 4 + (b >> 3));
      }
      n += __builtin_popcountll(any);
    }
  }
  unsigned h = 0;
#pragma unroll
  for (int k = 0; k < 2; ++k) {
    unsigned e = 4u << 13;
    if (lane > 0 && 2 * lane - 2 + k < n) {
      const unsigned pos = list[2 * lane + k] & 0x1fffu;
      const unsigned t = (*(LAS unsigned char*)(lds + pos) | 0x80u) - (unsigned)c0;
      e = pos | (((t >> 5) & (TENSOR ? 7u : 6u)) << 13);
    }
    h |= e << (16 * k);
  }
  if (lane == 0) h = (n > ECAP ? 0xffffu : (unsigned)n) | ((unsigned)c0 << 16);
  return h;
}

template <int TENSOR>
__device__ __forceinline__ void unpack_unit(u32x4 (&r)[8], unsigned h, LAS char* lds  , int lane) {
  const unsigned C0 = ((unsigned)__builtin_amdgcn_readfirstlane((int)h) >> 16) * 0x01010101u;
#pragma unroll
  for (int d = 0; d < 4; ++d) {
    if (TENSOR == 0) {
      unsigned q = (r[6][d] << 1) & 0x80808080u;
#pragma unroll
      for (int i = 0; i < 6; ++i) q |= (r[i][d] >> (6 - i)) & (0x01010101u << i);
#pragma unroll
      for (int i = 0; i < 7; ++i) r[i][d] = (r[i][d] & 0xbfbfbfbfu) + C0;
      r[7][d] = q + C0;
    } else {
      const unsigned p0 = r[0][d], p1 = r[1][d], p2 = r[2][d], p3 = r[3][d], p4 = r[4][d], p5 = r[5][d];
      r[0][d] = (p0 & 0x9f9f9f9fu) + C0; r[1][d] = (p1 & 0x9f9f9f9fu) + C0; r[2][d] = (p2 & 0x9f9f9f9fu) + C0;
      r[3][d] = (((p0 >> 5) & 0x03030303u) | ((p1 >> 3) & 0x0c0c0c0cu) | ((p2 >> 1) & 0x10101010u) | ((p2 << 1) & 0x80808080u)) + C0;
      r[4][d] = (p3 & 0x9f9f9f9fu) + C0; r[5][d] = (p4 & 0x9f9f9f9fu) + C0; r[6][d] = (p5 & 0x9f9f9f9fu) + C0;
      r[7][d] = (((p3 >> 5) & 0x03030303u) | ((p4 >> 3) & 0x0c0c0c0cu) | ((p5 >> 1) & 0x10101010u) | ((p5 << 1) & 0x80808080u)) + C0;
    }
  }
#pragma unroll
  for (int i = 0; i < 8; ++i) *(LAS u32x4*)(lds + i * 1024 + lane * 16) = r[i];
#pragma unroll
  for (int k = 0; k < 2; ++k) {
    const unsigned e = (h >> (16 * k)) & 0xffffu, f = e >> 13;
    if (lane > 0 && f != 4u)
      __hip_atomic_fetch_add((LAS unsigned*)(lds + (e & 0x1ffcu)), (f - 4u) << (5 + 8 * (e & 3u)), __ATOMIC_RELAXED, __HIP_MEMORY_SCOPE_WORKGROUP);
  }
  asm volatile("" ::: "memory");
#pragma unroll
  for (int i = 0; i < 8; ++i) r[i] = *(const LAS u32x4*)(lds + i * 1024 + lane * 16);
}

__device__ __forceinline__ void post_peers(const Params& P, int word, int lane, unsigned call, unsigned bits) {
  if (lane < NR - 1) {
    const unsigned p = (unsigned)(P.rank + 1 + lane) & (NR - 1);
    const u32x2 v = {call, bits};
    asm volatile("global_store_dwordx2 %0, %1, off sc0 sc1" ::"v"(P.ws_all[p] + OFF_PF + P.rank * 512 + word * 8), "v"(v) : "memory");
  }
}
__device__ __forceinline__ rsrc_t peer_rsrc(const Params& P, unsigned p) {
  const unsigned long long pa = (unsigned long long)P.ws_all[p & (NR - 1)];
  const unsigned long long pu = ((unsigned long long)(unsigned)__builtin_amdgcn_readfirstlane((int)(pa >> 32)) << 32) | (unsigned)__builtin_amdgcn_readfirstlane((int)pa);
  return __builtin_amdgcn_make_buffer_rsrc((void*)pu, 0, WS_NEED, 0x00020000);
}
__device__ __forceinline__ void prep_push(const Params& P, int c, int tensor, int g, int lane, unsigned call, const u32x4 (&r)[8]) {
  rsrc_t rp[NR - 1];
#pragma unroll
  for (int d = 0; d < NR - 1; ++d) rp[d] = peer_rsrc(P, (unsigned)(P.rank + 1 + d));
  asm volatile("s_nop 4");
#pragma unroll
  for (int i = 0; i < 8; ++i)
#pragma unroll
    for (int d = 0; d < NR - 1; ++d) {
      const int ci = c * 7 + 6 - d;
      BST16(r[i], rp[d], lane * 16, slab_off(tensor, gstep_off(4 + (ci >> 3), call & 1), ci & 7, g) + (i >> 2) * 4096, (i & 3) * 1024);
    }
  const rsrc_t rs = __builtin_amdgcn_make_buffer_rsrc((void*)P.ws, 0, WS_NEED, 0x00020000);
  WAIT_V(48);
  store_unit(r, 8, rs, lane * 16, slab_off(tensor, gstep_off(0, call & 1), c, g));
  WAIT_V(0);
  const int ci = c * 7 + 6 - lane;
  if (lane < NR - 1) st4_sys(P.ws_all[(unsigned)(P.rank + 1 + lane) & (NR - 1)] + uflag_off(tensor, 4 + (ci >> 3), ci & 7, g), call);
  if (lane == NR - 1) st4_sys(P.ws + uflag_off(tensor, 0, c, g), call);
}
__device__ __forceinline__ void prep_store(const Params& P, LAS char* lds, int c, int tensor, int g, int lane, unsigned call, u32x4 (&r)[8]) {
  const rsrc_t rs = __builtin_amdgcn_make_buffer_rsrc((void*)P.ws, 0, WS_NEED, 0x00020000);
  int pk = OFF_PK + (int)(call & 1) * (UPP * 8192) + (c * 16 + tensor * 8 + g) * 8192;
  int raw = slab_off(tensor, gstep_off(c >> 3, call & 1), c & 7, g);
  asm volatile("" : "+s"(pk), "+s"(raw));
  const unsigned h = tensor ? pack_unit<1>(r, lds, lane) : pack_unit<0>(r, lds, lane);
  STAMP(P.ws, 7);
  const unsigned ovf = ((unsigned)__builtin_amdgcn_readfirstlane((int)h) & 0xffffu) == 0xffffu;
#define PDELAY 24
  if (c >= 4) __builtin_amdgcn_s_sleep(PDELAY);
  store_unit(r, tensor ? 6 : 7, rs, lane * 16, pk);
  asm volatile("buffer_store_dword %0, %1, %2, %3 offen sc0 sc1" ::"v"(h), "v"(lane * 4), "s"(rs), "s"(pk + PK_EXC) : "memory");
#pragma unroll
  for (int i = 0; i < 8; ++i) r[i] = *(const LAS u32x4*)(lds + i * 1024 + lane * 16);
  if (c < 4 && !ovf) { WAIT_V(0); post_peers(P, (c - 2) * 16 + tensor * 8 + g, lane, call, 0u); }
  store_unit(r, 8, rs, lane * 16, raw);
  unsigned* pc = (unsigned*)(P.ws + OFF_PCNT + (c >> 1) * 128);
  if (ovf && c >= 4 && lane == 0) __hip_atomic_fetch_or(pc + 1 + (call & 1), 1u << ((c & 1) * 16 + tensor * 8 + g), __ATOMIC_RELAXED, __HIP_MEMORY_SCOPE_AGENT);
  WAIT_V(0);
  if (lane == 0) st4_sys(P.ws + uflag_off(tensor, c >> 3, c & 7, g), call);
  if (c < 4) { if (ovf) post_peers(P, (c - 2) * 16 + tensor * 8 + g, lane, call, ~0u); return; }
  unsigned tk = 0;
  ticket_issue(tk, pc, 1);
  WAIT_V(0); asm volatile("" : "+v"(tk));
  if ((unsigned)__builtin_amdgcn_readfirstlane((int)tk) == call * 32 - 1) {
    post_peers(P, 30 + (c >> 1), lane, call, ld4_sys(pc + 1 + (call & 1)));
    if (lane == 0) st4_sys(pc + 1 + (~call & 1), 0u);
  }
}

#define RAMP 12
#define LEADX 0, 0, 0, 0, 0, 0, 0, 0
#define LEADW 0, 0, 0, 0, 0, 0, 0, 0
struct Order { int e[NPULL]; };
constexpr Order make_order() {
  constexpr int lead[16] = {LEADX, LEADW};
  Order o{};
  int next[16] = {};
  for (int t = 0; t < 16; ++t) next[t] = RAWC;
  for (int k = 0; k < NPULL; ++k) {
    int best = -1, key = 0;
    for (int v = 0; v < 16; ++v) {
      const int t = (v >> 1) + 8 * (v & 1);
      const int kt = 4 * RAMP * next[t] - lead[t] * (next[t] - RAWC < RAMP ? next[t] - RAWC : RAMP);
      if (next[t] < 32 && (best < 0 || kt < key)) { best = t; key = kt; }
    }
    o.e[k] = next[best] * 16 + best;
    ++next[best];
  }
  return o;
}
__constant__ const Order ORDER = make_order();

constexpr int NDEC = 3;
constexpr int NSLOT = 6, SLOTB = 8192 + 256;
constexpr int RING_CTL = 2 * NSLOT * SLOTB;
static_assert(WIN >= 3 && WIN <= 8 && NSLOT % NDEC == 0 && RING_CTL + 256 <= LDS_BYTES && LPP * 7 <= 2 * NCOMM, "ring");
__device__ __forceinline__ unsigned lds_ld(unsigned a) {
  unsigned v; asm volatile("ds_read_b32 %0, %1\n\ts_waitcnt lgkmcnt(0)" : "=v"(v) : "v"(a) : "memory");
  return (unsigned)__builtin_amdgcn_readfirstlane((int)v);
}
__device__ __forceinline__ void lds_st(unsigned a, unsigned v) { asm volatile("ds_write_b32 %0, %1" ::"v"(a), "v"(v) : "memory"); }

__device__ __forceinline__ void loader_main(const Params& P, char* shm, int l, int lid, int lane, unsigned call) {
  const unsigned ctl = (unsigned)(size_t)(LAS char*)shm + RING_CTL + l * 128;
  if (lid >= LPP * 7) { lds_st(ctl + 4, 0); return; }
  const int q = lid % 7, li = lid / 7;
  const unsigned p = (unsigned)(P.rank + 1 + q) & (NR - 1);
  const rsrc_t rp = peer_rsrc(P, p);
  const char* pfp = P.ws + OFF_PF + p * 512 + lane * 8;
  unsigned* claim = (unsigned*)(P.ws + OFF_CLAIM + q * 128);
  const unsigned cbase = (call - 1) * NPULL - 2 * LPP;
  const int pkb = OFF_PK + (int)(call & 1) * (UPP * 8192);
  char* ring = shm + l * NSLOT * SLOTB;
  unsigned long long ready = 0;
  u32x2 pf = {0u, 0u};
  bool dead = false;
  int n = 0, slot = 0, src, nl;
  auto setup = [&](int e, int un, int sl, int landed) {
    const int w = e < 64 ? e - 32 : 30 + (e >> 5);
    if (!dead && !((ready >> w) & 1)) {
      WAIT_V(0); lds_st(ctl, (unsigned)landed);
      const unsigned long long t0 = __builtin_amdgcn_s_memrealtime();
      while (true) {
        asm volatile("global_load_dwordx2 %0, %1, off sc0 sc1\n\ts_waitcnt vmcnt(0)" : "=&v"(pf) : "v"(pfp) : "memory");
        ready = __ballot(pf.x >= call);
        if ((ready >> w) & 1) break;
        if (spin_fail(P.ws, call, t0)) { dead = true; break; }
        __builtin_amdgcn_s_sleep(1);
      }
    }
    if (un >= NSLOT && (int)lds_ld(ctl + 8 + (un % NDEC) * 4) <= un - NSLOT) {
      const unsigned long long t0 = __builtin_amdgcn_s_memrealtime();
      while ((int)lds_ld(ctl + 8 + (un % NDEC) * 4) <= un - NSLOT && __builtin_amdgcn_s_memrealtime() - t0 < 2 * DEADLINE) {}
    }
    const int c = e >> 4, tensor = (e >> 3) & 1, g = e & 7, ci = c * 7 + q, step = 4 + (ci >> 3);
    const bool raw = ((unsigned)__builtin_amdgcn_readlane((int)pf.y, w) >> (e & 31)) & 1u;
    nl = raw ? 8 : 7 - tensor;
    src = __builtin_amdgcn_readfirstlane(raw ? slab_off(tensor, gstep_off(c >> 3, call & 1), c & 7, g) : pkb + e * 8192);
    const u32x4 dsc = {(unsigned)nl, (unsigned)slab_off(tensor, gstep_off(step, call & 1), ci & 7, g), (unsigned)uflag_off(tensor, step, ci & 7, g), 0u};
    if (lane == 0) asm volatile("ds_write_b128 %0, %1" ::"v"(ctl + 32 + sl * 16), "v"(dsc) : "memory");
  };
  unsigned tk = 0;
  int e = ORDER.e[li];
  setup(e, 0, 0, 0);
  STAMP(P.ws, 2);
  while (true) {
    const int s0 = src, n0 = nl;
    char* lp = ring + slot * SLOTB;
#define PLANE(i) do { WAIT_V(WIN - 1); if ((i) == WIN - 3) lds_st(ctl, (unsigned)n);   \
      __builtin_amdgcn_raw_ptr_buffer_load_lds(rp, (LAS void*)(lp + ((i) >> 2) * 4096), 16, lane * 16, s0 + ((i) >> 2) * 4096, ((i) & 3) * 1024, 17); } while (0)
    ticket_issue(tk, claim, 1ull << (n & 1));
    __builtin_amdgcn_raw_ptr_buffer_load_lds(rp, (LAS void*)(lp + 8192), 4, lane * 4, pkb + e * 8192 + PK_EXC, 0, 17);
    PLANE(0); PLANE(1);
    asm volatile("" : "+v"(tk));
    const int kn = n == 0 ? li + LPP : (int)((unsigned)__builtin_amdgcn_readlane((int)tk, ~n & 1) - cbase);
    const bool more = kn < NPULL;
    e = ORDER.e[more ? kn : 0];
    PLANE(2); PLANE(3); PLANE(4); PLANE(5);
    if (n0 > 6) PLANE(6);
    if (n0 > 7) PLANE(7);
    ++n; slot = slot == NSLOT - 1 ? 0 : slot + 1;
    if (!more) break;
    setup(e, n, slot, n);
  }
  WAIT_V(0);
  asm volatile("" : "+v"(tk));
  lds_st(ctl, (unsigned)n);
  lds_st(ctl + 4, (unsigned)n);
  DBG(STAMPV(P.ws, 10, (unsigned long long)n);)
}

__device__ __forceinline__ void decoder_main(const Params& P, char* shm, int l, int j, int lane, unsigned call) {
  const unsigned ctl = (unsigned)(size_t)(LAS char*)shm + RING_CTL + l * 128;
  const rsrc_t rs = __builtin_amdgcn_make_buffer_rsrc((void*)P.ws, 0, WS_NEED, 0x00020000);
  const unsigned long long t0 = __builtin_amdgcn_s_memrealtime();
  DBG(unsigned long long tdec = 0; unsigned long long tst = 0; unsigned long long cntu = 0;)
  for (int n = j;; n += NDEC) {
    while ((int)lds_ld(ctl) <= n) {
      if (lds_ld(ctl + 4) <= (unsigned)n || __builtin_amdgcn_s_memrealtime() - t0 > 4 * DEADLINE) return;
      __builtin_amdgcn_s_sleep(1);
    }
    DBG(unsigned long long ta = NOW();)
    const int slot = n % NSLOT;
    LAS char* sp = (LAS char*)shm + (l * NSLOT + slot) * SLOTB;
    const u32x4 dsc = *(const LAS u32x4*)((LAS char*)shm + RING_CTL + l * 128 + 32 + slot * 16);
    const unsigned h = *(const LAS unsigned*)(sp + 8192 + lane * 4);
    u32x4 r[8];
#pragma unroll
    for (int i = 0; i < 8; ++i) r[i] = *(const LAS u32x4*)(sp + i * 1024 + lane * 16);
    const int nl = __builtin_amdgcn_readfirstlane((int)dsc.x), dst = __builtin_amdgcn_readfirstlane((int)dsc.y), flag = __builtin_amdgcn_readfirstlane((int)dsc.z);
    if (nl == 7) unpack_unit<0>(r, h, sp, lane);
    else if (nl == 6) unpack_unit<1>(r, h, sp, lane);
    WAIT_L(0);
#pragma unroll
    for (int i = 0; i < 8; ++i) asm volatile("" : "+v"(r[i]));
    lds_st(ctl + 8 + j * 4, (unsigned)(n + 1));
    DBG(unsigned long long tb = NOW();)
    asm volatile("s_nop 4");
    store_unit(r, 8, rs, lane * 16, dst);
    WAIT_V(0);
    if (lane == 0) st4_sys(P.ws + flag, call);
    DBG(tdec += tb - ta; tst += NOW() - tb; ++cntu; STAMPV(P.ws, 8, tdec); STAMPV(P.ws, 9, tst); STAMPV(P.ws, 10, cntu);)
  }
}

typedef float f32x2 __attribute__((ext_vector_type(2)));
__device__ __forceinline__ f32x2 gate2(f32x2 a, float ke, float ys) {
  constexpr float LIM = 60.f * 1.44269504088896341f, EPS = 0x1p-24f;
  const f32x2 t = a * ke;
  const f32x2 e = {__builtin_amdgcn_exp2f(__builtin_amdgcn_fmed3f(t.x, -LIM, LIM)), __builtin_amdgcn_exp2f(__builtin_amdgcn_fmed3f(t.y, -LIM, LIM))};
  const f32x2 d = e * EPS + EPS;
  const float r = __builtin_amdgcn_rcpf(d.x * d.y) * ys;
  const f32x2 sw = {d.y, d.x};
  return sw * r;
}

template <int MA0, int MA1>
__device__ __forceinline__ void gemm_main(const Params& P, char* shm, int brow, int bcol, int wid, int lane, unsigned call) {
  constexpr int RA0 = 32 * MA0, RA1 = 32 * MA1, BM = RA0 + RA1;
  constexpr int SLA0 = RA0 * 16, SLA1 = RA1 * 16, SLB = 128 * 16;
  constexpr int A0B = 8 * SLA0, A1B = 8 * SLA1, BB = 8 * SLB, BUFB = A0B + A1B + 2 * BB;
  constexpr int NA0 = (RA0 + 63) / 64, NA1 = (RA1 + 63) / 64;
  static_assert(2 * BUFB <= LDS_BYTES, "LDS");
  const int wr = wid >> 2, wc = wid & 3, fr = lane & 15, fq = lane >> 4;
  const unsigned par = call & 1;
  auto rX = __builtin_amdgcn_make_buffer_rsrc((void*)(P.ws + OFF_GX), 0, GSTEPS * STEPB, 0x00020000);
  auto rW = __builtin_amdgcn_make_buffer_rsrc((void*)(P.ws + OFF_GW), 0, GSTEPS * STEPB, 0x00020000);
  const int voff = lane * 16;
  const int sA = wid * SLAB + brow * 16, sB = wid * SLAB + bcol * 16;
  const char* pollp;
  { const int which = fq == 3 ? 0 : fq;
    const int tg = which == 0 ? (brow >> 9) : which == 1 ? ((brow + BM - 1) >> 9) : 8 + (bcol >> 9);
    pollp = P.ws + OFF_UF + ((wid * 16 + tg) * 32 + fr * 2) * 4; }
  u32x2 pv;
  unsigned rdy0 = 0, rdy1 = 0;
  bool dead = false;
#define POLL_ISSUE() asm volatile("global_load_dwordx2 %0, %1, off sc0 sc1" : "=v"(pv) : "v"(pollp) : "memory")
#define POLL_TAKE() do { asm volatile("" : "+v"(pv)); const unsigned long long m0_ = __ballot(pv.x >= call), m1_ = __ballot(pv.y >= call); \
    rdy0 = (unsigned)(m0_ & (m0_ >> 16) & (m0_ >> 32)) & 0xffffu; rdy1 = (unsigned)(m1_ & (m1_ >> 16) & (m1_ >> 32)) & 0xffffu; } while (0)
#define HAVE(s) (((((s) & 1) ? rdy1 : rdy0) >> ((s) >> 1)) & 1u)
#define NEED(s) do { if (!HAVE(s) && !dead) { const unsigned long long t0_ = __builtin_amdgcn_s_memrealtime(); \
    while (true) { POLL_ISSUE(); WAIT_V(0); POLL_TAKE(); if (HAVE(s)) break; if (spin_fail(P.ws, call, t0_)) { dead = true; break; } __builtin_amdgcn_s_sleep(2); } \
    WAIT_L(0); } } while (0)
#define OFF_A0(b) ((b) * BUFB)
#define OFF_A1(b) ((b) * BUFB + A0B)
#define OFF_B(b, h) ((b) * BUFB + A0B + A1B + (h) * BB)
#define GL(rs, ldsoff, soff, imm) __builtin_amdgcn_raw_ptr_buffer_load_lds(rs, (LAS void*)(shm + (ldsoff)), 16, voff, soff, imm, 17)
#define STAGE_A0(b, go) do { const int so_ = sA + (go), lo_ = OFF_A0(b) + wid * SLA0; GL(rX, lo_, so_, 0); GL(rX, lo_, so_, 1024); \
    if (NA0 == 3) { if (lane < 32) GL(rX, lo_, so_, 2048); } } while (0)
#define STAGE_A1(b, go) do { const int so_ = sA + RA0 * 16 + (go), lo_ = OFF_A1(b) + wid * SLA1; GL(rX, lo_, so_, 0); GL(rX, lo_, so_, 1024); \
    if (NA1 == 3) { if (lane < 32) GL(rX, lo_, so_, 2048); } } while (0)
#define STAGE_B(b, h, go) do { const int so_ = sB + (h) * 2048 + (go), lo_ = OFF_B(b, h) + wid * SLB; GL(rW, lo_, so_, 0); GL(rW, lo_, so_, 1024); } while (0)
  const LAS char* lsm = (const LAS char*)shm;
  const LAS char *pa0[2], *pa1[2], *pb[2];
#pragma unroll
  for (int b = 0; b < 2; ++b) {
    pa0[b] = lsm + OFF_A0(b) + fq * SLA0 + (wr * 16 * MA0 + fr) * 16;
    pa1[b] = lsm + OFF_A1(b) + fq * SLA1 + (wr * 16 * MA1 + fr) * 16;
    pb[b] = lsm + OFF_B(b, 0) + fq * SLB + (wc * 16 + fr) * 16;
    asm volatile("" : "+v"(pa0[b])); asm volatile("" : "+v"(pa1[b])); asm volatile("" : "+v"(pb[b]));
  }
#define LD8(p, sl) __builtin_shufflevector(*(const LAS v4i*)(p), *(const LAS v4i*)((p) + 4 * (sl)), 0, 1, 2, 3, 4, 5, 6, 7)
#define LDA0(b) _Pragma("unroll") for (int m = 0; m < MA0; ++m) At[m] = LD8(pa0[b] + m * 256, SLA0)
#define LDA1(b) _Pragma("unroll") for (int m = 0; m < MA1; ++m) At[m] = LD8(pa1[b] + m * 256, SLA1)
#define LDB(dst, b, h) _Pragma("unroll") for (int n = 0; n < 2; ++n) dst[n] = LD8(pb[b] + (h) * BB + n * 1024, SLB)
#define MFMA(c, a, b) asm volatile("v_mfma_f32_16x16x128_f8f6f4 %0, %1, %2, %0 blgp:1" : "+v"(c) : "v"(a), "v"(b))
#define MMA0(bj, Bf) do { __builtin_amdgcn_s_setprio(1); _Pragma("unroll") for (int m = 0; m < MA0; ++m) _Pragma("unroll") for (int n = 0; n < 2; ++n) \
    MFMA(acc0[m][(bj) * 2 + n], At[m], Bf[n]); __builtin_amdgcn_s_setprio(0); } while (0)
#define MMA1(bj, Bf) do { __builtin_amdgcn_s_setprio(1); _Pragma("unroll") for (int m = 0; m < MA1; ++m) _Pragma("unroll") for (int n = 0; n < 2; ++n) \
    MFMA(acc1[m][(bj) * 2 + n], At[m], Bf[n]); __builtin_amdgcn_s_setprio(0); } while (0)
#define TILE(b, nb, t) do { POLL_TAKE(); NEED((t) + 1); const int go_ = gstep_off((t) + 1, par); \
    LDB(B0, b, 0); LDA0(b); STAGE_B(nb, 0, go_); STAGE_A0(nb, go_); POLL_ISSUE(); \
    WAIT_V(NA1 + 2 + NA0 + 1); BAR; WAIT_L(0); MMA0(0, B0); BAR; SCHED; \
    LDB(B1, b, 1); STAGE_B(nb, 1, go_); STAGE_A1(nb, go_); \
    WAIT_V(2 + NA0 + 1 + 2 + NA1); BAR; WAIT_L(0); MMA0(1, B1); BAR; SCHED; \
    LDA1(b); BAR; WAIT_L(0); MMA1(0, B0); BAR; SCHED; \
    WAIT_V(2 + NA1); BAR; MMA1(1, B1); BAR; SCHED; } while (0)
  f32x4 acc0[MA0][4] = {}, acc1[MA1][4] = {};
  v8i At[MA0 > MA1 ? MA0 : MA1], B0[2], B1[2];
  NEED(0);
  STAMP(P.ws, 2);
  { const int go = gstep_off(0, par);
    STAGE_B(0, 0, go); STAGE_A0(0, go); POLL_ISSUE(); STAGE_B(0, 1, go); STAGE_A1(0, go); }
  if (wr == 1) BAR;
  WAIT_V(2 + NA1); BAR; BAR; SCHED;
  for (int t = 0; t < NSTEP - 2; t += 2) {
    TILE(0, 1, t);
    TILE(1, 0, t + 1);
  }
  TILE(0, 1, NSTEP - 2);
  { LDB(B0, 1, 0); LDA0(1);
    WAIT_V(NA1); BAR; WAIT_L(0); MMA0(0, B0); BAR; SCHED;
    LDB(B1, 1, 1);
    WAIT_V(0); BAR; WAIT_L(0); MMA0(1, B1); BAR; SCHED;
    LDA1(1); BAR; WAIT_L(0); MMA1(0, B0); MMA1(1, B1); }
  asm volatile("s_nop 15\n\ts_nop 15"); SCHED;
  if (wr == 0) BAR;
  asm volatile("" : "+v"(pv));
  STAMP(P.ws, 3);
  const bool bad = __builtin_amdgcn_readfirstlane(ld4_sys(P.ws + OFF_TMO)) == call;
  const float scale = bad ? __builtin_nanf("") : P.sx[0] * P.sw[0];
  const float ke = scale * -1.44269504088896341f, ys = scale * 0x1p-24f;
  int tz = threadIdx.x; asm volatile("" : "+v"(tz));
  const int cv = (((tz & 63) >> 4) * 4 * DIM + (tz & 15) * 4) * 4;
  auto rC = __builtin_amdgcn_make_buffer_rsrc((void*)P.out, 0, DIM * DIM * 4, 0x00020000);
#define OUTPOL "sc1"
#define ST_ROW(o, row) do { const int so_ = ((row) * DIM + bcol + wc * 64) * 4; \
    asm volatile("buffer_store_dwordx4 %0, %1, %2, %3 offen " OUTPOL "\n\ts_nop 1" ::"v"(o), "v"(cv), "s"(rC), "s"(so_) : "memory"); } while (0)
#define ST_TILE(acc, row0) _Pragma("unroll") for (int j = 0; j < 4; j += 2) { f32x4 o0_, o1_; \
    _Pragma("unroll") for (int u = 0; u < 4; ++u) { const f32x2 a_ = {acc[u][j], acc[u][j + 1]}, g_ = gate2(a_, ke, ys); o0_[u] = a_.x * g_.x; o1_[u] = a_.y * g_.y; } \
    ST_ROW(o0_, (row0) + j); ST_ROW(o1_, (row0) + j + 1); }
#pragma unroll
  for (int m = 0; m < MA0; ++m) { ST_TILE(acc0[m], brow + wr * 16 * MA0 + m * 16); }
#pragma unroll
  for (int m = 0; m < MA1; ++m) { ST_TILE(acc1[m], brow + RA0 + wr * 16 * MA1 + m * 16); }
}

__global__ __launch_bounds__(512, 2) void fused_kernel(Params P) {
  extern __shared__ __attribute__((aligned(16))) char shm[];
  const int b = blockIdx.x;
  const int wid = __builtin_amdgcn_readfirstlane(threadIdx.x >> 6), lane = threadIdx.x & 63;
  STAMP(P.ws, 0);
  unsigned cnt;
  asm volatile("global_load_dword %0, %1, off sc0 sc1" : "=v"(cnt) : "v"(P.ws + OFF_CNT) : "memory");
  int pc = -1;
  const int j = b - NCOMM;
  if (b < NCOMM) { if (wid == 7) pc = b >> 4; }
  else if (wid < 2) pc = RAWC + 2 * (j >> 4) + wid;
  else if (wid == 2 && j < 32) pc = 30 + (j >> 4);
  const int ptensor = (b >> 3) & 1, pg = b & 7;
  u32x4 r[8];
  if (pc >= 0) prep_load(P, pc, ptensor, pg, lane, r);
  WAIT_V(0); asm volatile("" : "+v"(cnt));
  const unsigned call = (unsigned)__builtin_amdgcn_readfirstlane((int)cnt) / NBLK + 1;
  STAMP(P.ws, 6);
  if (b < NCOMM) {
    if (wid == 0) *(LAS unsigned*)((LAS char*)shm + RING_CTL + lane * 4) = (lane & 31) == 1 ? 0xffffffffu : 0u;
    WAIT_L(0); BAR;
  }
  if (pc >= RAWC) prep_store(P, (LAS char*)shm + wid * 8448, pc, ptensor, pg, lane, call, r);
  else if (pc >= 0) prep_push(P, pc, ptensor, pg, lane, call, r);
  STAMP(P.ws, 1);
  if (b < NCOMM) {
    if (wid < 2) loader_main(P, shm, wid, b * 2 + wid, lane, call);
    else decoder_main(P, shm, wid & 1, (wid - 2) >> 1, lane, call);
    if (threadIdx.x == 0) __hip_atomic_fetch_add((unsigned*)(P.ws + OFF_CNT), 1u, __ATOMIC_RELAXED, __HIP_MEMORY_SCOPE_AGENT);
    STAMP(P.ws, 5);
    return;
  }
  WAIT_V(0); WAIT_L(0); BAR;
  const int xcd = b & 7, idx = j >> 3;
  const int rb = idx % 7, bcol = ((xcd >> 1) * 4 + idx / 7) * 256;
  const int brow = (xcd & 1) * 2048 + (rb == 0 ? 0 : 32 + rb * 288);
  if (rb == 0) gemm_main<5, 5>(P, shm, brow, bcol, wid, lane, call);
  else gemm_main<5, 4>(P, shm, brow, bcol, wid, lane, call);
  if (threadIdx.x == 0) __hip_atomic_fetch_add((unsigned*)(P.ws + OFF_CNT), 1u, __ATOMIC_RELAXED, __HIP_MEMORY_SCOPE_AGENT);
}

}

extern "C" void kernel_launch(int rank, int nranks, ncclComm_t comm, hipStream_t stream, void* const* d_in, const int* in_sizes, int n_in,
                              void* d_out, int out_size, const int* in_dtypes, int out_dtype, void* const* d_out_all,
                              void* const* d_ws_all, size_t ws_bytes) {
  if (nranks != NR || n_in != 4 || in_sizes[0] != DIM * KSH || in_sizes[1] != KSH * DIM || out_size != DIM * DIM || ws_bytes < (size_t)WS_NEED) {
    fprintf(stderr, "unexpected problem shape\n");
    return;
  }
  constexpr int lds = LDS_BYTES;
  HIP_CHECK(hipFuncSetAttribute((const void*)fused_kernel, hipFuncAttributeMaxDynamicSharedMemorySize, lds));
  Params P;
  P.x = (const unsigned char*)d_in[0];
  P.w = (const unsigned char*)d_in[1];
  P.sx = (const float*)d_in[2];
  P.sw = (const float*)d_in[3];
  P.out = (float*)d_out;
  P.ws_all = (char* const*)d_ws_all;
  P.ws = (char*)d_ws_all[rank];
  P.rank = rank;
  P.pad = 0;
  hipLaunchKernelGGL(fused_kernel, dim3(NBLK), dim3(512), lds, stream, P);
  HIP_CHECK(hipGetLastError());
}
```

```cpp
#include <hip/hip_runtime.h>
#include <rccl/rccl.h>
#include <cstdio>
#include <type_traits>

#define HIP_CHECK(expr) do { hipError_t _e = (expr); if (_e != hipSuccess) { \
    fprintf(stderr, "HIP error %s at %s:%d\n", hipGetErrorString(_e), __FILE__, __LINE__); \
    return; } } while (0)

namespace {

constexpr int NR = 8;
constexpr int DIM = 4096;
constexpr int KSH = DIM / NR;
constexpr int NSTEP = 32;
constexpr int SLAB = DIM * 16;
constexpr int STEPB = 8 * SLAB;
constexpr int NBLK = 256, NCOMM = 32;
constexpr int UPP = 512;
constexpr int RAWC = 2;
constexpr int NPULL = UPP - 16 * RAWC;
#define LPP 9
#define WIN 8
#define TAILS 4
constexpr int JSTEP = NSTEP - TAILS;
constexpr int ECAP = 126;
constexpr int OFF_CNT = 0, OFF_TMO = 128, OFF_CLAIM = 256;
constexpr int OFF_PCNT = 2048;
constexpr int OFF_PF = 4096;
constexpr int OFF_UF = 8192;
constexpr int OFF_DBG = 65536;
constexpr int PSTEPS = 6;
constexpr int GSTEPS = NSTEP + PSTEPS;
constexpr int OFF_GX = 1 << 20, OFF_GW = OFF_GX + GSTEPS * STEPB;
constexpr int OFF_PK = OFF_GW + GSTEPS * STEPB;
constexpr int PK_EXC = 7168;
constexpr int WS_NEED = OFF_PK + 2 * UPP * 8192;
constexpr int LDS_BYTES = 2 * (128 * 320 + 32768);
constexpr unsigned long long DEADLINE = 30000000ull;

typedef int v8i __attribute__((ext_vector_type(8)));
typedef int v4i __attribute__((ext_vector_type(4)));
typedef unsigned u32x4 __attribute__((ext_vector_type(4)));
typedef unsigned u32x2 __attribute__((ext_vector_type(2)));
typedef float f32x4 __attribute__((ext_vector_type(4)));
#define LAS __attribute__((address_space(3)))
#define WAIT_V(n) asm volatile("s_waitcnt vmcnt(%0)" ::"i"(n) : "memory")
#define WAIT_L(n) asm volatile("s_waitcnt lgkmcnt(%0)" ::"i"(n) : "memory")
#define BAR __builtin_amdgcn_s_barrier()
#define SCHED __builtin_amdgcn_sched_barrier(0)

struct Params {
  const unsigned char* x;
  const unsigned char* w;
  const float* sx;
  const float* sw;
  float* out;
  char* const* ws_all;
  char* ws;
  int rank;
  int pad;
};

#define STAMP(ws, idx) do {} while (0)
#define DBG(x)

__device__ __forceinline__ void st16_sys(void* p, u32x4 v) { asm volatile("global_store_dwordx4 %0, %1, off sc0 sc1\n\ts_nop 1" ::"v"(p), "v"(v) : "memory"); }
__device__ __forceinline__ void st4_sys(void* p, unsigned v) { asm volatile("global_store_dword %0, %1, off sc0 sc1" ::"v"(p), "v"(v) : "memory"); }
__device__ __forceinline__ unsigned ld4_sys(const void* p) {
  unsigned v; asm volatile("global_load_dword %0, %1, off sc0 sc1\n\ts_waitcnt vmcnt(0)" : "=&v"(v) : "v"(p) : "memory"); return v;
}
__device__ __forceinline__ void ticket_issue(unsigned& t, void* p, unsigned long long lanes) {
  unsigned long long keep;
  asm volatile("s_mov_b64 %1, exec\n\ts_mov_b64 exec, %4\n\tglobal_atomic_add %0, %2, %3, off sc0\n\ts_mov_b64 exec, %1"
               : "+v"(t), "=&s"(keep) : "v"(p), "v"(1u), "s"(lanes) : "memory");
}
__device__ __forceinline__ void give_up(char* ws, unsigned call) { st4_sys(ws + OFF_TMO + (threadIdx.x & 31) * 4, call); WAIT_V(0); }
__device__ __forceinline__ bool spin_fail(char* ws, unsigned call, unsigned long long t0) {
  if ((unsigned)__builtin_amdgcn_readfirstlane((int)ld4_sys(ws + OFF_TMO)) == call) return true;
  if (__builtin_amdgcn_s_memrealtime() - t0 > DEADLINE) { give_up(ws, call); return true; }
  return false;
}
__device__ __forceinline__ int gstep_off(int step, unsigned par) { return (step < PSTEPS ? (int)par * PSTEPS + step : step + PSTEPS) * STEPB; }

typedef __amdgpu_buffer_rsrc_t rsrc_t;
#define BST16(v, rs, voff, soff, imm) asm volatile("buffer_store_dwordx4 %0, %1, %2, %3 offen offset:%4 sc0 sc1\n\ts_nop 1" ::"v"(v), "v"(voff), "s"(rs), "s"(soff), "i"(imm) : "memory")
#define BLD16(v, rs, voff, soff, imm) asm volatile("buffer_load_dwordx4 %0, %1, %2, %3 offen offset:%4 sc0 sc1" : "=v"(v) : "v"(voff), "s"(rs), "s"(soff), "i"(imm) : "memory")
__device__ __forceinline__ void store_unit(const u32x4 (&r)[8], int n, rsrc_t rs, int voff, int soff) {
#pragma unroll
  for (int i = 0; i < 8; ++i)
    if (i < n) BST16(r[i], rs, voff, soff + (i >> 2) * 4096, (i & 3) * 1024);
}
__device__ __forceinline__ int slab_off(int tensor, int gstep_bytes, int slot, int g) { return (tensor ? OFF_GW : OFF_GX) + gstep_bytes + slot * SLAB + g * 8192; }
__device__ __forceinline__ int uflag_off(int tensor, int step, int slot, int g) { return OFF_UF + (((slot * 2 + tensor) * 8 + g) * 32 + step) * 4; }

__device__ __forceinline__ void prep_load(const Params& P, int c, int tensor, int g, int lane, u32x4 (&r)[8]) {
  if (tensor == 0) {
    auto rs = __builtin_amdgcn_make_buffer_rsrc((void*)P.x, 0, DIM * KSH, 0x00020000);
#pragma unroll
    for (int i = 0; i < 8; ++i) r[i] = __builtin_amdgcn_raw_buffer_load_b128(rs, lane * KSH, (g * 512 + i * 64) * KSH + c * 16, 0);
  } else {
    auto rs = __builtin_amdgcn_make_buffer_rsrc((void*)P.w, 0, DIM * KSH, 0x00020000);
    unsigned d[2][16];
#pragma unroll
    for (int k = 0; k < 16; ++k)
#pragma unroll
      for (int h = 0; h < 2; ++h) d[h][k] = __builtin_amdgcn_raw_buffer_load_b32(rs, lane * 4 + h * 256, (c * 16 + k) * DIM + g * 512, 0);
#pragma unroll
    for (int h = 0; h < 2; ++h)
#pragma unroll
      for (int q = 0; q < 4; ++q) {
        const unsigned a0 = d[h][4 * q], a1 = d[h][4 * q + 1], a2 = d[h][4 * q + 2], a3 = d[h][4 * q + 3];
        const unsigned l01 = __builtin_amdgcn_perm(a1, a0, 0x05010400), h01 = __builtin_amdgcn_perm(a1, a0, 0x07030602);
        const unsigned l23 = __builtin_amdgcn_perm(a3, a2, 0x05010400), h23 = __builtin_amdgcn_perm(a3, a2, 0x07030602);
        r[h * 4 + 0][q] = __builtin_amdgcn_perm(l23, l01, 0x05040100);
        r[h * 4 + 1][q] = __builtin_amdgcn_perm(l23, l01, 0x07060302);
        r[h * 4 + 2][q] = __builtin_amdgcn_perm(h23, h01, 0x05040100);
        r[h * 4 + 3][q] = __builtin_amdgcn_perm(h23, h01, 0x07060302);
      }
  }
}

template <int TENSOR>
__device__ __forceinline__ unsigned pack_unit(u32x4 (&r)[8], LAS char* lds  , int lane) {
  constexpr unsigned S = 0x80808080u, CM = TENSOR ? 0x1f1f1f1fu : 0x3f3f3f3fu;
  unsigned sum = 0;
#pragma unroll
  for (int i = 0; i < 8; ++i) {
    *(LAS u32x4*)(lds + i * 1024 + lane * 16) = r[i];
#pragma unroll
    for (int d = 0; d < 4; ++d) sum = __builtin_amdgcn_sad_u8(r[i][d] & ~S, 0u, sum);
  }
  sum += (unsigned)__builtin_amdgcn_update_dpp(0, (int)sum, 0xb1, 0xf, 0xf, true);
  sum += (unsigned)__builtin_amdgcn_update_dpp(0, (int)sum, 0x4e, 0xf, 0xf, true);
  sum += (unsigned)__builtin_amdgcn_update_dpp(0, (int)sum, 0x141, 0xf, 0xf, true);
  sum += (unsigned)__builtin_amdgcn_update_dpp(0, (int)sum, 0x140, 0xf, 0xf, true);
  const int total = __builtin_amdgcn_readlane((int)sum, 0) + __builtin_amdgcn_readlane((int)sum, 16) + __builtin_amdgcn_readlane((int)sum, 32) + __builtin_amdgcn_readlane((int)sum, 48);
  int c0 = (total - (TENSOR ? 166707 : 343245)) >> 13;
  c0 = c0 < 0 ? 0 : c0 > (TENSOR ? 96 : 64) ? (TENSOR ? 96 : 64) : c0;
  const unsigned C0 = (unsigned)c0 * 0x01010101u;
  unsigned M[4] = {0, 0, 0, 0};
#pragma unroll
  for (int i = 0; i < 8; ++i)
#pragma unroll
    for (int d = 0; d < 4; ++d) {
      const unsigned v = r[i][d], t = (v | S) - C0;
      const unsigned e = (TENSOR ? (~t | (t << 1) | (t << 2)) : (~t | (t << 1))) & S;
      M[d] |= e >> i;
      r[i][d] = (t & CM) | (v & S);
    }
#pragma unroll
  for (int d = 0; d < 4; ++d) {
    if (TENSOR == 0) {
      const unsigned q = r[7][d];
#pragma unroll
      for (int i = 0; i < 6; ++i) r[i][d] |= (q << (6 - i)) & 0x40404040u;
      r[6][d] |= (q >> 1) & 0x40404040u;
    } else {
      const unsigned q3 = r[3][d], q7 = r[7][d];
      r[0][d] |= (q3 << 5) & 0x60606060u; r[1][d] |= (q3 << 3) & 0x60606060u; r[2][d] |= ((q3 << 1) & 0x20202020u) | ((q3 >> 1) & 0x40404040u);
      r[3][d] = r[4][d] | ((q7 << 5) & 0x60606060u); r[4][d] = r[5][d] | ((q7 << 3) & 0x60606060u);
      r[5][d] = r[6][d] | ((q7 << 1) & 0x20202020u) | ((q7 >> 1) & 0x40404040u);
    }
  }
  LAS unsigned short* list = (LAS unsigned short*)(lds + 8192);
  int n = 0;
#pragma unroll
  for (int d = 0; d < 4; ++d) {
    unsigned m = M[d];
    while (n <= ECAP) {
      const unsigned long long any = __ballot(m != 0);
      if (!any) break;
      if (m != 0) {
        const int b = __builtin_ctz(m);
        m &= m - 1;
        const int idx = n + (int)__builtin_amdgcn_mbcnt_hi((unsigned)(any >> 32), __builtin_amdgcn_mbcnt_lo((unsigned)any, 0u));
        if (idx < ECAP) list[2 + idx] = (unsigned short)((7 - (b & 7)) * 1024 + lane * 16 + d * 4 + (b >> 3));
      }
      n += __builtin_popcountll(any);
    }
  }
  unsigned h = 0;
#pragma unroll
  for (int k = 0; k < 2; ++k) {
    unsigned e = 4u << 13;
    if (lane > 0 && 2 * lane - 2 + k < n) {
      const unsigned pos = list[2 * lane + k] & 0x1fffu;
      const unsigned t = (*(LAS unsigned char*)(lds + pos) | 0x80u) - (unsigned)c0;
      e = pos | (((t >> 5) & (TENSOR ? 7u : 6u)) << 13);
    }
    h |= e << (16 * k);
  }
  if (lane == 0) h = (n > ECAP ? 0xffffu : (unsigned)n) | ((unsigned)c0 << 16);
  return h;
}

template <int TENSOR>
__device__ __forceinline__ void unpack_unit(u32x4 (&r)[8], unsigned h, LAS char* lds  , int lane) {
  const unsigned C0 = ((unsigned)__builtin_amdgcn_readfirstlane((int)h) >> 16) * 0x01010101u;
#pragma unroll
  for (int d = 0; d < 4; ++d) {
    if (TENSOR == 0) {
      unsigned q = (r[6][d] << 1) & 0x80808080u;
#pragma unroll
      for (int i = 0; i < 6; ++i) q |= (r[i][d] >> (6 - i)) & (0x01010101u << i);
#pragma unroll
      for (int i = 0; i < 7; ++i) r[i][d] = (r[i][d] & 0xbfbfbfbfu) + C0;
      r[7][d] = q + C0;
    } else {
      const unsigned p0 = r[0][d], p1 = r[1][d], p2 = r[2][d], p3 = r[3][d], p4 = r[4][d], p5 = r[5][d];
      r[0][d] = (p0 & 0x9f9f9f9fu) + C0; r[1][d] = (p1 & 0x9f9f9f9fu) + C0; r[2][d] = (p2 & 0x9f9f9f9fu) + C0;
      r[3][d] = (((p0 >> 5) & 0x03030303u) | ((p1 >> 3) & 0x0c0c0c0cu) | ((p2 >> 1) & 0x10101010u) | ((p2 << 1) & 0x80808080u)) + C0;
      r[4][d] = (p3 & 0x9f9f9f9fu) + C0; r[5][d] = (p4 & 0x9f9f9f9fu) + C0; r[6][d] = (p5 & 0x9f9f9f9fu) + C0;
      r[7][d] = (((p3 >> 5) & 0x03030303u) | ((p4 >> 3) & 0x0c0c0c0cu) | ((p5 >> 1) & 0x10101010u) | ((p5 << 1) & 0x80808080u)) + C0;
    }
  }
#pragma unroll
  for (int i = 0; i < 8; ++i) *(LAS u32x4*)(lds + i * 1024 + lane * 16) = r[i];
#pragma unroll
  for (int k = 0; k < 2; ++k) {
    const unsigned e = (h >> (16 * k)) & 0xffffu, f = e >> 13;
    if (lane > 0 && f != 4u)
      __hip_atomic_fetch_add((LAS unsigned*)(lds + (e & 0x1ffcu)), (f - 4u) << (5 + 8 * (e & 3u)), __ATOMIC_RELAXED, __HIP_MEMORY_SCOPE_WORKGROUP);
  }
  asm volatile("" ::: "memory");
#pragma unroll
  for (int i = 0; i < 8; ++i) r[i] = *(const LAS u32x4*)(lds + i * 1024 + lane * 16);
}

__device__ __forceinline__ void post_peers(const Params& P, int word, int lane, unsigned call, unsigned bits) {
  if (lane < NR - 1) {
    const unsigned p = (unsigned)(P.rank + 1 + lane) & (NR - 1);
    const u32x2 v = {call, bits};
    asm volatile("global_store_dwordx2 %0, %1, off sc0 sc1" ::"v"(P.ws_all[p] + OFF_PF + P.rank * 512 + word * 8), "v"(v) : "memory");
  }
}
__device__ __forceinline__ rsrc_t peer_rsrc(const Params& P, unsigned p) {
  const unsigned long long pa = (unsigned long long)P.ws_all[p & (NR - 1)];
  const unsigned long long pu = ((unsigned long long)(unsigned)__builtin_amdgcn_readfirstlane((int)(pa >> 32)) << 32) | (unsigned)__builtin_amdgcn_readfirstlane((int)pa);
  return __builtin_amdgcn_make_buffer_rsrc((void*)pu, 0, WS_NEED, 0x00020000);
}
__device__ __forceinline__ void prep_push(const Params& P, int c, int tensor, int g, int lane, unsigned call, const u32x4 (&r)[8]) {
  rsrc_t rp[NR - 1];
#pragma unroll
  for (int d = 0; d < NR - 1; ++d) rp[d] = peer_rsrc(P, (unsigned)(P.rank + 1 + d));
  asm volatile("s_nop 4");
#pragma unroll
  for (int i = 0; i < 8; ++i)
#pragma unroll
    for (int d = 0; d < NR - 1; ++d) {
      const int ci = c * 7 + 6 - d;
      BST16(r[i], rp[d], lane * 16, slab_off(tensor, gstep_off(4 + (ci >> 3), call & 1), ci & 7, g) + (i >> 2) * 4096, (i & 3) * 1024);
    }
  const rsrc_t rs = __builtin_amdgcn_make_buffer_rsrc((void*)P.ws, 0, WS_NEED, 0x00020000);
  WAIT_V(48);
  store_unit(r, 8, rs, lane * 16, slab_off(tensor, gstep_off(0, call & 1), c, g));
  WAIT_V(0);
  const int ci = c * 7 + 6 - lane;
  if (lane < NR - 1) st4_sys(P.ws_all[(unsigned)(P.rank + 1 + lane) & (NR - 1)] + uflag_off(tensor, 4 + (ci >> 3), ci & 7, g), call);
  if (lane == NR - 1) st4_sys(P.ws + uflag_off(tensor, 0, c, g), call);
}
__device__ __forceinline__ void prep_store(const Params& P, LAS char* lds, int c, int tensor, int g, int lane, unsigned call, u32x4 (&r)[8]) {
  const rsrc_t rs = __builtin_amdgcn_make_buffer_rsrc((void*)P.ws, 0, WS_NEED, 0x00020000);
  int pk = OFF_PK + (int)(call & 1) * (UPP * 8192) + (c * 16 + tensor * 8 + g) * 8192;
  int raw = slab_off(tensor, gstep_off(c >> 3, call & 1), c & 7, g);
  asm volatile("" : "+s"(pk), "+s"(raw));
  const unsigned h = tensor ? pack_unit<1>(r, lds, lane) : pack_unit<0>(r, lds, lane);
  STAMP(P.ws, 30);
  const unsigned ovf = ((unsigned)__builtin_amdgcn_readfirstlane((int)h) & 0xffffu) == 0xffffu;
#define PDELAY 24
  if (c >= 4) __builtin_amdgcn_s_sleep(PDELAY);
  store_unit(r, tensor ? 6 : 7, rs, lane * 16, pk);
  asm volatile("buffer_store_dword %0, %1, %2, %3 offen sc0 sc1" ::"v"(h), "v"(lane * 4), "s"(rs), "s"(pk + PK_EXC) : "memory");
#pragma unroll
  for (int i = 0; i < 8; ++i) r[i] = *(const LAS u32x4*)(lds + i * 1024 + lane * 16);
  if (c < 4 && !ovf) { WAIT_V(0); post_peers(P, (c - 2) * 16 + tensor * 8 + g, lane, call, 0u); }
  store_unit(r, 8, rs, lane * 16, raw);
  unsigned* pc = (unsigned*)(P.ws + OFF_PCNT + (c >> 1) * 128);
  if (ovf && c >= 4 && lane == 0) __hip_atomic_fetch_or(pc + 1 + (call & 1), 1u << ((c & 1) * 16 + tensor * 8 + g), __ATOMIC_RELAXED, __HIP_MEMORY_SCOPE_AGENT);
  WAIT_V(0);
  if (lane == 0) st4_sys(P.ws + uflag_off(tensor, c >> 3, c & 7, g), call);
  if (c < 4) { if (ovf) post_peers(P, (c - 2) * 16 + tensor * 8 + g, lane, call, ~0u); return; }
  unsigned tk = 0;
  ticket_issue(tk, pc, 1);
  WAIT_V(0); asm volatile("" : "+v"(tk));
  if ((unsigned)__builtin_amdgcn_readfirstlane((int)tk) == call * 32 - 1) {
    post_peers(P, 30 + (c >> 1), lane, call, ld4_sys(pc + 1 + (call & 1)));
    if (lane == 0) st4_sys(pc + 1 + (~call & 1), 0u);
  }
}

#define RAMP 12
#define LEADX 0, 0, 0, 0, 0, 0, 0, 0
#define LEADW 0, 0, 0, 0, 0, 0, 0, 0
struct Order { int e[NPULL]; };
constexpr Order make_order() {
  constexpr int lead[16] = {LEADX, LEADW};
  Order o{};
  int next[16] = {};
  for (int t = 0; t < 16; ++t) next[t] = RAWC;
  for (int k = 0; k < NPULL; ++k) {
    int best = -1, key = 0;
    for (int v = 0; v < 16; ++v) {
      const int t = (v >> 1) + 8 * (v & 1);
      const int kt = 4 * RAMP * next[t] - lead[t] * (next[t] - RAWC < RAMP ? next[t] - RAWC : RAMP) +
                     (t >= 4 && t < 8 && 7 * next[t] >= 8 * (JSTEP - 4) ? 1 << 20 : 0);
      if (next[t] < 32 && (best < 0 || kt < key)) { best = t; key = kt; }
    }
    o.e[k] = next[best] * 16 + best;
    ++next[best];
  }
  return o;
}
__constant__ const Order ORDER = make_order();

constexpr int NDEC = 3;
constexpr int NSLOT = 6, SLOTB = 8192 + 256;
constexpr int RING_CTL = 2 * NSLOT * SLOTB;
static_assert(WIN >= 3 && WIN <= 8 && NSLOT % NDEC == 0 && RING_CTL + 256 <= LDS_BYTES && LPP * 7 <= 2 * NCOMM, "ring");
__device__ __forceinline__ unsigned lds_ld(unsigned a) {
  unsigned v; asm volatile("ds_read_b32 %0, %1\n\ts_waitcnt lgkmcnt(0)" : "=v"(v) : "v"(a) : "memory");
  return (unsigned)__builtin_amdgcn_readfirstlane((int)v);
}
__device__ __forceinline__ void lds_st(unsigned a, unsigned v) { asm volatile("ds_write_b32 %0, %1" ::"v"(a), "v"(v) : "memory"); }

__device__ __forceinline__ void loader_main(const Params& P, char* shm, int l, int lid, int lane, unsigned call) {
  const unsigned ctl = (unsigned)(size_t)(LAS char*)shm + RING_CTL + l * 128;
  if (lid >= LPP * 7) { lds_st(ctl + 4, 0); return; }
  const int q = lid % 7, li = lid / 7;
  const unsigned p = (unsigned)(P.rank + 1 + q) & (NR - 1);
  const rsrc_t rp = peer_rsrc(P, p);
  const char* pfp = P.ws + OFF_PF + p * 512 + lane * 8;
  unsigned* claim = (unsigned*)(P.ws + OFF_CLAIM + q * 128);
  const unsigned cbase = (call - 1) * NPULL - 2 * LPP;
  const int pkb = OFF_PK + (int)(call & 1) * (UPP * 8192);
  char* ring = shm + l * NSLOT * SLOTB;
  unsigned long long ready = 0;
  u32x2 pf = {0u, 0u};
  bool dead = false;
  int n = 0, slot = 0, src, nl;
  auto setup = [&](int e, int un, int sl, int landed) {
    const int w = e < 64 ? e - 32 : 30 + (e >> 5);
    if (!dead && !((ready >> w) & 1)) {
      WAIT_V(0); lds_st(ctl, (unsigned)landed);
      const unsigned long long t0 = __builtin_amdgcn_s_memrealtime();
      while (true) {
        asm volatile("global_load_dwordx2 %0, %1, off sc0 sc1\n\ts_waitcnt vmcnt(0)" : "=&v"(pf) : "v"(pfp) : "memory");
        ready = __ballot(pf.x >= call);
        if ((ready >> w) & 1) break;
        if (spin_fail(P.ws, call, t0)) { dead = true; break; }
        __builtin_amdgcn_s_sleep(1);
      }
    }
    if (un >= NSLOT && (int)lds_ld(ctl + 8 + (un % NDEC) * 4) <= un - NSLOT) {
      const unsigned long long t0 = __builtin_amdgcn_s_memrealtime();
      while ((int)lds_ld(ctl + 8 + (un % NDEC) * 4) <= un - NSLOT && __builtin_amdgcn_s_memrealtime() - t0 < 2 * DEADLINE) {}
    }
    const int c = e >> 4, tensor = (e >> 3) & 1, g = e & 7, ci = c * 7 + q, step = 4 + (ci >> 3);
    const bool raw = ((unsigned)__builtin_amdgcn_readlane((int)pf.y, w) >> (e & 31)) & 1u;
    nl = raw ? 8 : 7 - tensor;
    src = __builtin_amdgcn_readfirstlane(raw ? slab_off(tensor, gstep_off(c >> 3, call & 1), c & 7, g) : pkb + e * 8192);
    const u32x4 dsc = {(unsigned)nl, (unsigned)slab_off(tensor, gstep_off(step, call & 1), ci & 7, g), (unsigned)uflag_off(tensor, step, ci & 7, g), 0u};
    if (lane == 0) asm volatile("ds_write_b128 %0, %1" ::"v"(ctl + 32 + sl * 16), "v"(dsc) : "memory");
  };
  unsigned tk = 0;
  int e = ORDER.e[li];
  setup(e, 0, 0, 0);
  STAMP(P.ws, 2);
  while (true) {
    const int s0 = src, n0 = nl;
    char* lp = ring + slot * SLOTB;
#define PLANE(i) do { WAIT_V(WIN - 1); if ((i) == WIN - 3) lds_st(ctl, (unsigned)n);   \
      __builtin_amdgcn_raw_ptr_buffer_load_lds(rp, (LAS void*)(lp + ((i) >> 2) * 4096), 16, lane * 16, s0 + ((i) >> 2) * 4096, ((i) & 3) * 1024, 17); } while (0)
    ticket_issue(tk, claim, 1ull << (n & 1));
    __builtin_amdgcn_raw_ptr_buffer_load_lds(rp, (LAS void*)(lp + 8192), 4, lane * 4, pkb + e * 8192 + PK_EXC, 0, 17);
    PLANE(0); PLANE(1);
    asm volatile("" : "+v"(tk));
    const int kn = n == 0 ? li + LPP : (int)((unsigned)__builtin_amdgcn_readlane((int)tk, ~n & 1) - cbase);
    const bool more = kn < NPULL;
    e = ORDER.e[more ? kn : 0];
    PLANE(2); PLANE(3); PLANE(4); PLANE(5);
    if (n0 > 6) PLANE(6);
    if (n0 > 7) PLANE(7);
    ++n; slot = slot == NSLOT - 1 ? 0 : slot + 1;
    if (!more) break;
    setup(e, n, slot, n);
  }
  WAIT_V(0);
  asm volatile("" : "+v"(tk));
  lds_st(ctl, (unsigned)n);
  lds_st(ctl + 4, (unsigned)n);
  DBG(STAMPV(P.ws, 10, (unsigned long long)n);)
}

__device__ __forceinline__ void decoder_main(const Params& P, char* shm, int l, int j, int lane, unsigned call) {
  const unsigned ctl = (unsigned)(size_t)(LAS char*)shm + RING_CTL + l * 128;
  const rsrc_t rs = __builtin_amdgcn_make_buffer_rsrc((void*)P.ws, 0, WS_NEED, 0x00020000);
  const unsigned long long t0 = __builtin_amdgcn_s_memrealtime();
  DBG(unsigned long long tdec = 0; unsigned long long tst = 0; unsigned long long cntu = 0;)
  for (int n = j;; n += NDEC) {
    while ((int)lds_ld(ctl) <= n) {
      if (lds_ld(ctl + 4) <= (unsigned)n || __builtin_amdgcn_s_memrealtime() - t0 > 4 * DEADLINE) return;
      __builtin_amdgcn_s_sleep(1);
    }
    DBG(unsigned long long ta = NOW();)
    const int slot = n % NSLOT;
    LAS char* sp = (LAS char*)shm + (l * NSLOT + slot) * SLOTB;
    const u32x4 dsc = *(const LAS u32x4*)((LAS char*)shm + RING_CTL + l * 128 + 32 + slot * 16);
    const unsigned h = *(const LAS unsigned*)(sp + 8192 + lane * 4);
    u32x4 r[8];
#pragma unroll
    for (int i = 0; i < 8; ++i) r[i] = *(const LAS u32x4*)(sp + i * 1024 + lane * 16);
    const int nl = __builtin_amdgcn_readfirstlane((int)dsc.x), dst = __builtin_amdgcn_readfirstlane((int)dsc.y), flag = __builtin_amdgcn_readfirstlane((int)dsc.z);
    if (nl == 7) unpack_unit<0>(r, h, sp, lane);
    else if (nl == 6) unpack_unit<1>(r, h, sp, lane);
    WAIT_L(0);
#pragma unroll
    for (int i = 0; i < 8; ++i) asm volatile("" : "+v"(r[i]));
    lds_st(ctl + 8 + j * 4, (unsigned)(n + 1));
    DBG(unsigned long long tb = NOW();)
    asm volatile("s_nop 4");
    store_unit(r, 8, rs, lane * 16, dst);
    WAIT_V(0);
    if (lane == 0) st4_sys(P.ws + flag, call);
    DBG(tdec += tb - ta; tst += NOW() - tb; ++cntu; STAMPV(P.ws, 8, tdec); STAMPV(P.ws, 9, tst); STAMPV(P.ws, 10, cntu);)
  }
}

typedef float f32x2 __attribute__((ext_vector_type(2)));
__device__ __forceinline__ f32x2 gate2(f32x2 a, float ke, float ys) {
  constexpr float LIM = 60.f * 1.44269504088896341f, EPS = 0x1p-24f;
  const f32x2 t = a * ke;
  const f32x2 e = {__builtin_amdgcn_exp2f(__builtin_amdgcn_fmed3f(t.x, -LIM, LIM)), __builtin_amdgcn_exp2f(__builtin_amdgcn_fmed3f(t.y, -LIM, LIM))};
  const f32x2 d = e * EPS + EPS;
  const float r = __builtin_amdgcn_rcpf(d.x * d.y) * ys;
  const f32x2 sw = {d.y, d.x};
  return sw * r;
}

template <int I, int E, typename F> __device__ __forceinline__ void static_for(F&& f) {
  if constexpr (I < E) { f(std::integral_constant<int, I>{}); static_for<I + 1, E>(f); }
}

template <int MA0, int MA1>
__device__ __forceinline__ void gemm_main(const Params& P, char* shm, int browA, int browB, int bcol, int wid, int lane, unsigned call) {
  constexpr int RA0 = 32 * MA0, RA1 = 32 * MA1;
  constexpr int SLA0 = RA0 * 16, SLA1 = RA1 * 16, SLB = 128 * 16;
  constexpr int A0B = 8 * SLA0, A1B = 8 * SLA1, BB = 8 * SLB, BUFB = A0B + A1B + 2 * BB;
  constexpr int NA0 = (RA0 + 63) / 64, NA1 = (RA1 + 63) / 64;
  constexpr int S = TAILS, J = JSTEP, END = NSTEP + S;
  static_assert(2 * BUFB <= LDS_BYTES && S % 2 == 0 && S >= 4 && J >= 8, "shape");
  const int wr = wid >> 2, wc = wid & 3, fr = lane & 15, fq = lane >> 4;
  const unsigned par = call & 1;
  auto rX = __builtin_amdgcn_make_buffer_rsrc((void*)(P.ws + OFF_GX), 0, GSTEPS * STEPB, 0x00020000);
  auto rW = __builtin_amdgcn_make_buffer_rsrc((void*)(P.ws + OFF_GW), 0, GSTEPS * STEPB, 0x00020000);
  auto rU = __builtin_amdgcn_make_buffer_rsrc((void*)P.ws, 0, OFF_DBG, 0x00020000);
  auto rC = __builtin_amdgcn_make_buffer_rsrc((void*)P.out, 0, DIM * DIM * 4, 0x00020000);
  const int voff = lane * 16;
  const int sA0 = wid * SLAB + browA * 16, sA1 = wid * SLAB + browB * 16, sB = wid * SLAB + bcol * 16;
  constexpr unsigned M0 = 0x13, M1 = 0x0c, MALL = 0x1f, MBAD = 0x20;
  int pollv;
  { const int tg = lane == 0 ? browA >> 9 : lane == 1 ? (browA + RA0 - 1) >> 9 : lane == 2 ? browB >> 9 : lane == 3 ? (browB + RA1 - 1) >> 9 : 8 + (bcol >> 9);
    pollv = lane == 5 ? OFF_TMO : OFF_UF + (wid * 16 + tg) * 128; }
  unsigned pv = 0, got = 0;
  bool dead = false;
  const float scale = P.sx[0] * P.sw[0];
  int kei, ysi, isi;
  asm volatile("s_nop 1\n\tv_readfirstlane_b32 %0, %1" : "=s"(isi) : "v"(1.0f / scale));
  asm volatile("s_nop 1\n\tv_readfirstlane_b32 %0, %2\n\tv_readfirstlane_b32 %1, %3" : "=s"(kei), "=s"(ysi) : "v"(scale * -1.44269504088896341f), "v"(scale * 0x1p-24f));
  int t4 = 0;
#define POLL(step) asm volatile("buffer_load_dword %0, %1, %2, %3 offen offset:%4 sc0 sc1" : "=v"(pv) : "v"(pollv), "s"(rU), "s"(t4), "i"((step) * 4) : "memory")
#define TAKE() do { asm volatile("" : "+v"(pv)); got = (unsigned)__ballot(pv >= call); } while (0)
#define NEED(step, m) do { if ((got & (m)) != (m) && !dead) { const unsigned long long t0_ = __builtin_amdgcn_s_memrealtime(); \
    while (true) { unsigned sv_;     \
      asm volatile("buffer_load_dword %0, %1, %2, %3 offen offset:%4 sc0 sc1\n\ts_waitcnt vmcnt(0)" : "=&v"(sv_) : "v"(pollv), "s"(rU), "s"(t4), "i"((step) * 4) : "memory"); \
      got = (unsigned)__ballot(sv_ >= call); if ((got & (m)) == (m)) break; \
      if (got & MBAD) { dead = true; break; } \
      if (__builtin_amdgcn_s_memrealtime() - t0_ > DEADLINE) { give_up(P.ws, call); dead = true; break; } \
      __builtin_amdgcn_s_sleep(1); } \
    WAIT_L(0); } } while (0)
#define OFF_A0(b) ((b) * BUFB)
#define OFF_A1(b) ((b) * BUFB + A0B)
#define OFF_B(b, h) ((b) * BUFB + A0B + A1B + (h) * BB)
#define GL(rs, ldsoff, soff, imm) __builtin_amdgcn_raw_ptr_buffer_load_lds(rs, (LAS void*)(shm + (ldsoff)), 16, voff, soff, imm, 17)
#define STAGE_A0(b, go) do { const int so_ = sA0 + (go), lo_ = OFF_A0(b) + wid * SLA0; GL(rX, lo_, so_, 0); GL(rX, lo_, so_, 1024); \
    if (NA0 == 3) { if (lane < 32) GL(rX, lo_, so_, 2048); } } while (0)
#define STAGE_A1(b, go) do { const int so_ = sA1 + (go), lo_ = OFF_A1(b) + wid * SLA1; GL(rX, lo_, so_, 0); GL(rX, lo_, so_, 1024); \
    if (NA1 == 3) { if (lane < 32) GL(rX, lo_, so_, 2048); } } while (0)
#define STAGE_B(b, h, go) do { const int so_ = sB + (h) * 2048 + (go), lo_ = OFF_B(b, h) + wid * SLB; GL(rW, lo_, so_, 0); GL(rW, lo_, so_, 1024); } while (0)
  const LAS char* lsm = (const LAS char*)shm;
  const LAS char *pa0[2], *pa1[2], *pb[2];
#pragma unroll
  for (int b = 0; b < 2; ++b) {
    pa0[b] = lsm + OFF_A0(b) + fq * SLA0 + (wr * 16 * MA0 + fr) * 16;
    pa1[b] = lsm + OFF_A1(b) + fq * SLA1 + (wr * 16 * MA1 + fr) * 16;
    pb[b] = lsm + OFF_B(b, 0) + fq * SLB + (wc * 16 + fr) * 16;
    asm volatile("" : "+v"(pa0[b])); asm volatile("" : "+v"(pa1[b])); asm volatile("" : "+v"(pb[b]));
  }
#define LD8(p, sl) __builtin_shufflevector(*(const LAS v4i*)(p), *(const LAS v4i*)((p) + 4 * (sl)), 0, 1, 2, 3, 4, 5, 6, 7)
#define LDA0(b) _Pragma("unroll") for (int m = 0; m < MA0; ++m) At[m] = LD8(pa0[b] + m * 256, SLA0)
#define LDA1(b) _Pragma("unroll") for (int m = 0; m < MA1; ++m) At[m] = LD8(pa1[b] + m * 256, SLA1)
#define LDB(dst, b, h) _Pragma("unroll") for (int n = 0; n < 2; ++n) dst[n] = LD8(pb[b] + (h) * BB + n * 1024, SLB)
#define MFMA(c, a, b) asm volatile("v_mfma_f32_16x16x128_f8f6f4 %0, %1, %2, %0 blgp:1" : "+v"(c) : "v"(a), "v"(b))
#define MMA0(bj, Bf) do { __builtin_amdgcn_s_setprio(1); _Pragma("unroll") for (int m = 0; m < MA0; ++m) _Pragma("unroll") for (int n = 0; n < 2; ++n) \
    MFMA(acc0[m][(bj) * 2 + n], At[m], Bf[n]); __builtin_amdgcn_s_setprio(0); } while (0)
#define MMA1(bj, Bf) do { __builtin_amdgcn_s_setprio(1); _Pragma("unroll") for (int m = 0; m < MA1; ++m) _Pragma("unroll") for (int n = 0; n < 2; ++n) \
    MFMA(acc1[m][(bj) * 2 + n], At[m], Bf[n]); __builtin_amdgcn_s_setprio(0); } while (0)
#define TILE(b, nb, t, r) do { TAKE(); NEED((r) + 1, MALL); const int go_ = gstep_off((t) + 1, par); \
    LDB(B0, b, 0); LDA0(b); STAGE_B(nb, 0, go_); STAGE_A0(nb, go_); POLL((r) + 2); \
    WAIT_V(NA1 + 2 + NA0 + 1); BAR; WAIT_L(0); MMA0(0, B0); BAR; SCHED; \
    LDB(B1, b, 1); STAGE_B(nb, 1, go_); STAGE_A1(nb, go_); \
    WAIT_V(2 + NA0 + 1 + 2 + NA1); BAR; WAIT_L(0); MMA0(1, B1); BAR; SCHED; \
    LDA1(b); BAR; WAIT_L(0); MMA1(0, B0); BAR; SCHED; \
    WAIT_V(2 + NA1); BAR; MMA1(1, B1); BAR; SCHED; } while (0)
#define H0STEP(b, nb, t, r) do { \
    LDB(B0, b, 0); LDA0(b); STAGE_B(nb, 1, ((t) + 1 + PSTEPS) * STEPB); WAIT_V(NA0 + 4); TAKE(); POLL((r) + 3); \
    WAIT_L(0); BAR; MMA0(0, B0); BAR; SCHED; \
    LDB(B1, b, 1); NEED((r) + 2, M0); STAGE_B(b, 0, ((t) + 2 + PSTEPS) * STEPB); STAGE_A0(b, ((t) + 2 + PSTEPS) * STEPB); WAIT_V(NA0 + 5); \
    WAIT_L(0); BAR; MMA0(1, B1); BAR; SCHED; } while (0)
#define OUTPOL "sc1"
#define ST_ROW(o, row) do { const int so_ = ((row) * DIM + bcol + wc * 64) * 4; \
    asm volatile("buffer_store_dwordx4 %0, %1, %2, %3 offen " OUTPOL "\n\ts_nop 1" ::"v"(o), "v"(cv), "s"(rC), "s"(so_) : "memory"); } while (0)
#define EARLYPOL OUTPOL
#define ST_ROWP(o, row, POL, vo) do { const int so_ = ((row) * DIM + bcol + wc * 64) * 4; \
    asm volatile("buffer_store_dwordx4 %0, %1, %2, %3 offen " POL "\n\ts_nop 1" ::"v"(o), "v"(vo), "s"(rC), "s"(so_) : "memory"); } while (0)
#define ST_PAIR(acc, j, row0) do { f32x4 o0_, o1_; \
    _Pragma("unroll") for (int u = 0; u < 4; ++u) { const f32x2 a_ = {acc[u][j], acc[u][(j) + 1]}, g_ = gate2(a_, __builtin_bit_cast(float, kei), __builtin_bit_cast(float, ysi)); \
      float p_, q_; asm("v_mul_f32 %0, %1, %2" : "=v"(p_) : "v"(a_.x), "v"(g_.x)); asm("v_mul_f32 %0, %1, %2" : "=v"(q_) : "v"(a_.y), "v"(g_.y)); \
      o0_[u] = p_; o1_[u] = q_; } \
    ST_ROW(o0_, (row0) + (j)); ST_ROW(o1_, (row0) + (j) + 1); } while (0)
  f32x4 acc0[MA0][4] = {}, acc1[MA1][4] = {};
  v8i At[MA0 > MA1 ? MA0 : MA1], B0[2], B1[2];
  NEED(0, MALL);
  STAMP(P.ws, 2);
  { const int go = gstep_off(0, par);
    STAGE_B(0, 0, go); STAGE_A0(0, go); POLL(1); STAGE_B(0, 1, go); STAGE_A1(0, go); }
  if (wr == 1) BAR;
  WAIT_V(2 + NA1); BAR; BAR; SCHED;
  for (int t = 0; t < J - 2; t += 2) {
    asm volatile("s_lshl_b32 %0, %1, 2" : "=s"(t4) : "s"(t) : "scc");
    TILE(0, 1, t, 0);
    TILE(1, 0, t + 1, 1);
  }
  t4 = 0;
  TILE(0, 1, J - 2, J - 2);
  {
    TAKE(); NEED(J, M0);
    LDB(B0, 1, 0); LDA0(1); STAGE_B(0, 0, (J + PSTEPS) * STEPB); STAGE_A0(0, (J + PSTEPS) * STEPB); POLL(J + 1);
    WAIT_V(NA1 + 2 + NA0 + 1); BAR; WAIT_L(0); MMA0(0, B0); BAR; SCHED;
    LDB(B1, 1, 1); STAGE_B(0, 1, (J + PSTEPS) * STEPB);
    WAIT_V(2 + NA0 + 1 + 2); BAR; WAIT_L(0); MMA0(1, B1); BAR; SCHED;
    LDA1(1); BAR; WAIT_L(0); MMA1(0, B0); BAR; SCHED;
    WAIT_V(2); TAKE(); NEED(J + 1, M0); POLL(J + 2); STAGE_B(1, 0, (J + 1 + PSTEPS) * STEPB); STAGE_A0(1, (J + 1 + PSTEPS) * STEPB);
    BAR; MMA1(1, B1); BAR; SCHED; }
  for (int t = J; t < NSTEP - 4; t += 2) {
    asm volatile("s_lshl_b32 %0, %1, 2" : "=s"(t4) : "s"(t) : "scc");
    H0STEP(0, 1, t, 0);
    H0STEP(1, 0, t + 1, 1);
  }
  t4 = 0;
  int tz = threadIdx.x; asm volatile("" : "+v"(tz));
  const int cv = (((tz & 63) >> 4) * 4 * DIM + (tz & 15) * 4) * 4;
  float lim = 60.f * 1.44269504088896341f; asm volatile("" : "+v"(lim));
  f32x4 o0, o1;
  auto cluster = [&](auto phc, auto bjc, v8i (&Bf)[2]) {
    constexpr int ph = decltype(phc)::value, bj = decltype(bjc)::value, NU = 8 * MA0, G = 2 * MA1;
    constexpr int u0 = (ph * NU + 2 * S - 1) / (2 * S), u1 = ((ph + 1) * NU + 2 * S - 1) / (2 * S), V = 2 * (u1 - u0);
    float tv[V > 0 ? V : 1];
    __builtin_amdgcn_s_setprio(1);
    static_for<0, G>([&](auto gc) {
      constexpr int g = decltype(gc)::value;
      MFMA(acc1[g >> 1][bj * 2 + (g & 1)], At[g >> 1], Bf[g & 1]);
      static_for<0, 6 * V>([&](auto kc) {
        constexpr int k = decltype(kc)::value, op = k / V, v = k % V, u = u0 + (v >> 1);
        constexpr int cost = 4 * k + 4 * (op > 2 ? V : op == 2 ? v : 0) + 4 * (op > 4 ? V : op == 4 ? v : 0);
        if constexpr (cost * G / (32 * V) == g) {
          const float a = acc0[u >> 3][u & 3][(u >> 2 & 1) * 2 + (v & 1)];
          if constexpr (op == 0) asm volatile("v_mul_f32 %0, %1, %2" : "=v"(tv[v]) : "s"(kei), "v"(a));
          if constexpr (op == 1) asm volatile("v_med3_f32 %0, %0, %1, %2" : "+v"(tv[v]) : "s"(0xc2ad1f97), "v"(lim));
          if constexpr (op == 2) asm volatile("v_exp_f32 %0, %0" : "+v"(tv[v]));
          if constexpr (op == 3) asm volatile("v_fma_f32 %0, %0, %1, %1" : "+v"(tv[v]) : "s"(isi));
          if constexpr (op == 4) asm volatile("v_rcp_f32 %0, %0" : "+v"(tv[v]));
          if constexpr (op == 5) {
            float r; asm volatile("v_mul_f32 %0, %1, %2" : "=v"(r) : "v"(a), "v"(tv[v]));
            if constexpr (v & 1) o1[u & 3] = r; else o0[u & 3] = r;
            if constexpr ((u & 3) == 3 && (v & 1)) {
              const int row = browA + wr * 16 * MA0 + (u >> 3) * 16 + (u >> 2 & 1) * 2;
              ST_ROWP(o0, row, EARLYPOL, cv); ST_ROWP(o1, row + 1, EARLYPOL, cv);
            }
          }
        }
      });
    });
    __builtin_amdgcn_s_setprio(0);
  };
  static_for<NSTEP - 4, END>([&](auto ic) {
    constexpr int i = decltype(ic)::value, b = i & 1, nb = b ^ 1;
    constexpr bool h = i >= NSTEP;
    constexpr auto step = [](int k) { return k < NSTEP ? k : k - S; };
    constexpr auto nst = [](int k, int q) {
      const int ph = 2 * (k - NSTEP) + q;
      return ph < 0 ? 0 : 2 * (((ph + 1) * 8 * MA0 + 2 * S - 1) / (2 * S) / 4 - (ph * 8 * MA0 + 2 * S - 1) / (2 * S) / 4); };
    constexpr auto nq1 = [](int k) { return k + 1 < END ? 2 : 0; };
    constexpr auto nq2 = [](int k) { return k + 2 < END ? 2 + (k + 2 < NSTEP ? NA0 : NA1) : 0; };
    LDB(B0, b, 0);
    if constexpr (h) { LDA1(b); } else { LDA0(b); }
    if constexpr (i + 1 < END) STAGE_B(nb, 1, (step(i + 1) + PSTEPS) * STEPB);
    WAIT_V(nst(i - 1, 0) + nq2(i - 1) + nst(i - 1, 1) + nq1(i));
    TAKE();
    if constexpr (i == NSTEP) {
      STAMP(P.ws, 7);
      if (dead || (got & MBAD)) isi = 0x7fc00000;
      asm volatile("s_nop 15\n\ts_nop 15"); SCHED;
    }
    if constexpr (i + 3 < END) POLL(step(i + 3));
    WAIT_L(0); BAR;
    if constexpr (h) cluster(std::integral_constant<int, 2 * (i - NSTEP)>{}, std::integral_constant<int, 0>{}, B0); else MMA0(0, B0);
    BAR; SCHED;
    LDB(B1, b, 1);
    if constexpr (i + 2 < END) {
      NEED(step(i + 2), (i + 2 < NSTEP ? M0 : M1));
      STAGE_B(b, 0, (step(i + 2) + PSTEPS) * STEPB);
      if constexpr (i + 2 < NSTEP) STAGE_A0(b, (step(i + 2) + PSTEPS) * STEPB); else STAGE_A1(b, (step(i + 2) + PSTEPS) * STEPB);
    }
    if constexpr (i + 1 < END) WAIT_V(nst(i - 1, 1) + nq1(i) + (i + 3 < END ? 1 : 0) + nst(i, 0) + nq2(i));
    WAIT_L(0); BAR;
    if constexpr (h) cluster(std::integral_constant<int, 2 * (i - NSTEP) + 1>{}, std::integral_constant<int, 1>{}, B1); else MMA0(1, B1);
    if constexpr (i + 1 < END) { BAR; SCHED; }
  });
  asm volatile("s_nop 15\n\ts_nop 15"); SCHED;
  if (wr == 0) BAR;
  STAMP(P.ws, 3);
  POLL(0); WAIT_V(0); TAKE();
  if (dead || (got & MBAD)) kei = ysi = 0x7fc00000;
#pragma unroll
  for (int m = 0; m < MA1; ++m) { ST_PAIR(acc1[m], 0, browB + wr * 16 * MA1 + m * 16); ST_PAIR(acc1[m], 2, browB + wr * 16 * MA1 + m * 16); }
}

__global__ __launch_bounds__(512, 2) void fused_kernel(Params P) {
  extern __shared__ __attribute__((aligned(16))) char shm[];
  const int b = blockIdx.x;
  const int wid = __builtin_amdgcn_readfirstlane(threadIdx.x >> 6), lane = threadIdx.x & 63;
  STAMP(P.ws, 0);
  unsigned cnt;
  asm volatile("global_load_dword %0, %1, off sc0 sc1" : "=v"(cnt) : "v"(P.ws + OFF_CNT) : "memory");
  int pc = -1;
  const int j = b - NCOMM;
  if (b < NCOMM) { if (wid == 7) pc = b >> 4; }
  else if (wid < 2) pc = RAWC + 2 * (j >> 4) + wid;
  else if (wid == 2 && j < 32) pc = 30 + (j >> 4);
  const int ptensor = (b >> 3) & 1, pg = b & 7;
  u32x4 r[8];
#define LDELAY 0
  if (LDELAY > 0 && pc >= 4) __builtin_amdgcn_s_sleep(LDELAY);
  if (pc >= 0) prep_load(P, pc, ptensor, pg, lane, r);
  WAIT_V(0); asm volatile("" : "+v"(cnt));
  const unsigned call = (unsigned)__builtin_amdgcn_readfirstlane((int)cnt) / NBLK + 1;
  STAMP(P.ws, 6);
  if (b < NCOMM) {
    if (wid == 0) *(LAS unsigned*)((LAS char*)shm + RING_CTL + lane * 4) = (lane & 31) == 1 ? 0xffffffffu : 0u;
    WAIT_L(0); BAR;
  }
  if (pc >= RAWC) prep_store(P, (LAS char*)shm + wid * 8448, pc, ptensor, pg, lane, call, r);
  else if (pc >= 0) prep_push(P, pc, ptensor, pg, lane, call, r);
  STAMP(P.ws, 1);
  if (b < NCOMM) {
    if (wid < 2) loader_main(P, shm, wid, b * 2 + wid, lane, call);
    else decoder_main(P, shm, wid & 1, (wid - 2) >> 1, lane, call);
    if (threadIdx.x == 0) __hip_atomic_fetch_add((unsigned*)(P.ws + OFF_CNT), 1u, __ATOMIC_RELAXED, __HIP_MEMORY_SCOPE_AGENT);
    STAMP(P.ws, 5);
    return;
  }
  WAIT_V(0); WAIT_L(0); BAR;
  const int xcd = b & 7, idx = j >> 3;
  const int rb = idx % 7, bcol = ((xcd >> 1) * 4 + idx / 7) * 256;
  const int browA = (xcd & 1) * 1024 + (rb <= 4 ? rb * 160 : 128 + rb * 128);
  const int browB = 2048 + (xcd & 1) * 1024 + (rb == 0 ? 0 : rb <= 4 ? 32 + rb * 128 : rb * 160 - 96);
  if (rb == 0) gemm_main<5, 5>(P, shm, browA, browB, bcol, wid, lane, call);
  else if (rb < 4) gemm_main<5, 4>(P, shm, browA, browB, bcol, wid, lane, call);
  else gemm_main<4, 5>(P, shm, browA, browB, bcol, wid, lane, call);
  if (threadIdx.x == 0) __hip_atomic_fetch_add((unsigned*)(P.ws + OFF_CNT), 1u, __ATOMIC_RELAXED, __HIP_MEMORY_SCOPE_AGENT);
}

}

extern "C" void kernel_launch(int rank, int nranks, ncclComm_t comm, hipStream_t stream, void* const* d_in, const int* in_sizes, int n_in,
                              void* d_out, int out_size, const int* in_dtypes, int out_dtype, void* const* d_out_all,
                              void* const* d_ws_all, size_t ws_bytes) {
  if (nranks != NR || n_in != 4 || in_sizes[0] != DIM * KSH || in_sizes[1] != KSH * DIM || out_size != DIM * DIM || ws_bytes < (size_t)WS_NEED) {
    fprintf(stderr, "unexpected problem shape\n");
    return;
  }
  constexpr int lds = LDS_BYTES;
  HIP_CHECK(hipFuncSetAttribute((const void*)fused_kernel, hipFuncAttributeMaxDynamicSharedMemorySize, lds));
  Params P;
  P.x = (const unsigned char*)d_in[0];
  P.w = (const unsigned char*)d_in[1];
  P.sx = (const float*)d_in[2];
  P.sw = (const float*)d_in[3];
  P.out = (float*)d_out;
  P.ws_all = (char* const*)d_ws_all;
  P.ws = (char*)d_ws_all[rank];
  P.rank = rank;
  P.pad = 0;
  hipLaunchKernelGGL(fused_kernel, dim3(NBLK), dim3(512), lds, stream, P);
  HIP_CHECK(hipGetLastError());
}
```

```cpp
#include <hip/hip_runtime.h>
#include <rccl/rccl.h>
#include <cstdio>
#include <type_traits>

#define HIP_CHECK(expr) do { hipError_t _e = (expr); if (_e != hipSuccess) { \
    fprintf(stderr, "HIP error %s at %s:%d\n", hipGetErrorString(_e), __FILE__, __LINE__); \
    return; } } while (0)

namespace {

constexpr int NR = 8;
constexpr int DIM = 4096;
constexpr int KSH = DIM / NR;
constexpr int NSTEP = 32;
constexpr int SLAB = DIM * 16;
constexpr int STEPB = 8 * SLAB;
constexpr int NBLK = 256, NCOMM = 32;
constexpr int UPP = 512;
constexpr int RAWC = 2;
constexpr int NPULL = UPP - 16 * RAWC;
#define LPP 9
#define WIN 8
#define TAILS 4
constexpr int JSTEP = NSTEP - TAILS;
constexpr int ECAP = 126;
constexpr int OFF_CNT = 0, OFF_TMO = 128, OFF_CLAIM = 256;
constexpr int OFF_PCNT = 2048;
constexpr int OFF_PF = 4096;
constexpr int OFF_UF = 8192;
constexpr int OFF_DBG = 65536;
constexpr int PSTEPS = 6;
constexpr int GSTEPS = NSTEP + PSTEPS;
constexpr int OFF_GX = 1 << 20, OFF_GW = OFF_GX + GSTEPS * STEPB;
constexpr int OFF_PK = OFF_GW + GSTEPS * STEPB;
constexpr int PK_EXC = 7168;
constexpr int WS_NEED = OFF_PK + 2 * UPP * 8192;
constexpr int LDS_BYTES = 2 * (128 * 320 + 32768);
constexpr unsigned long long DEADLINE = 30000000ull;

typedef int v8i __attribute__((ext_vector_type(8)));
typedef int v4i __attribute__((ext_vector_type(4)));
typedef unsigned u32x4 __attribute__((ext_vector_type(4)));
typedef unsigned u32x2 __attribute__((ext_vector_type(2)));
typedef float f32x4 __attribute__((ext_vector_type(4)));
#define LAS __attribute__((address_space(3)))
#define WAIT_V(n) asm volatile("s_waitcnt vmcnt(%0)" ::"i"(n) : "memory")
#define WAIT_L(n) asm volatile("s_waitcnt lgkmcnt(%0)" ::"i"(n) : "memory")
#define BAR __builtin_amdgcn_s_barrier()
#define SCHED __builtin_amdgcn_sched_barrier(0)

struct Params {
  const unsigned char* x;
  const unsigned char* w;
  const float* sx;
  const float* sw;
  float* out;
  char* const* ws_all;
  char* ws;
  int rank;
  int pad;
};

#define STAMP(ws, idx) do {} while (0)
#define DBG(x)

__device__ __forceinline__ void st16_sys(void* p, u32x4 v) { asm volatile("global_store_dwordx4 %0, %1, off sc0 sc1\n\ts_nop 1" ::"v"(p), "v"(v) : "memory"); }
__device__ __forceinline__ void st4_sys(void* p, unsigned v) { asm volatile("global_store_dword %0, %1, off sc0 sc1" ::"v"(p), "v"(v) : "memory"); }
__device__ __forceinline__ unsigned ld4_sys(const void* p) {
  unsigned v; asm volatile("global_load_dword %0, %1, off sc0 sc1\n\ts_waitcnt vmcnt(0)" : "=&v"(v) : "v"(p) : "memory"); return v;
}
__device__ __forceinline__ void ticket_issue(unsigned& t, void* p, unsigned long long lanes) {
  unsigned long long keep;
  asm volatile("s_mov_b64 %1, exec\n\ts_mov_b64 exec, %4\n\tglobal_atomic_add %0, %2, %3, off sc0\n\ts_mov_b64 exec, %1"
               : "+v"(t), "=&s"(keep) : "v"(p), "v"(1u), "s"(lanes) : "memory");
}
__device__ __forceinline__ void give_up(char* ws, unsigned call) { st4_sys(ws + OFF_TMO + (threadIdx.x & 31) * 4, call); WAIT_V(0); }
__device__ __forceinline__ bool spin_fail(char* ws, unsigned call, unsigned long long t0) {
  if ((unsigned)__builtin_amdgcn_readfirstlane((int)ld4_sys(ws + OFF_TMO)) == call) return true;
  if (__builtin_amdgcn_s_memrealtime() - t0 > DEADLINE) { give_up(ws, call); return true; }
  return false;
}
__device__ __forceinline__ int gstep_off(int step, unsigned par) { return (step < PSTEPS ? (int)par * PSTEPS + step : step + PSTEPS) * STEPB; }

typedef __amdgpu_buffer_rsrc_t rsrc_t;
#define BST16(v, rs, voff, soff, imm) asm volatile("buffer_store_dwordx4 %0, %1, %2, %3 offen offset:%4 sc0 sc1\n\ts_nop 1" ::"v"(v), "v"(voff), "s"(rs), "s"(soff), "i"(imm) : "memory")
#define BLD16(v, rs, voff, soff, imm) asm volatile("buffer_load_dwordx4 %0, %1, %2, %3 offen offset:%4 sc0 sc1" : "=v"(v) : "v"(voff), "s"(rs), "s"(soff), "i"(imm) : "memory")
__device__ __forceinline__ void store_unit(const u32x4 (&r)[8], int n, rsrc_t rs, int voff, int soff) {
#pragma unroll
  for (int i = 0; i < 8; ++i)
    if (i < n) BST16(r[i], rs, voff, soff + (i >> 2) * 4096, (i & 3) * 1024);
}
__device__ __forceinline__ int slab_off(int tensor, int gstep_bytes, int slot, int g) { return (tensor ? OFF_GW : OFF_GX) + gstep_bytes + slot * SLAB + g * 8192; }
__device__ __forceinline__ int uflag_off(int tensor, int step, int slot, int g) { return OFF_UF + (((slot * 2 + tensor) * 8 + g) * 32 + step) * 4; }

__device__ __forceinline__ void prep_load(const Params& P, int c, int tensor, int g, int lane, u32x4 (&r)[8]) {
  if (tensor == 0) {
    auto rs = __builtin_amdgcn_make_buffer_rsrc((void*)P.x, 0, DIM * KSH, 0x00020000);
#pragma unroll
    for (int i = 0; i < 8; ++i) r[i] = __builtin_amdgcn_raw_buffer_load_b128(rs, lane * KSH, (g * 512 + i * 64) * KSH + c * 16, 0);
  } else {
    auto rs = __builtin_amdgcn_make_buffer_rsrc((void*)P.w, 0, DIM * KSH, 0x00020000);
    unsigned d[2][16];
#pragma unroll
    for (int k = 0; k < 16; ++k)
#pragma unroll
      for (int h = 0; h < 2; ++h) d[h][k] = __builtin_amdgcn_raw_buffer_load_b32(rs, lane * 4 + h * 256, (c * 16 + k) * DIM + g * 512, 0);
#pragma unroll
    for (int h = 0; h < 2; ++h)
#pragma unroll
      for (int q = 0; q < 4; ++q) {
        const unsigned a0 = d[h][4 * q], a1 = d[h][4 * q + 1], a2 = d[h][4 * q + 2], a3 = d[h][4 * q + 3];
        const unsigned l01 = __builtin_amdgcn_perm(a1, a0, 0x05010400), h01 = __builtin_amdgcn_perm(a1, a0, 0x07030602);
        const unsigned l23 = __builtin_amdgcn_perm(a3, a2, 0x05010400), h23 = __builtin_amdgcn_perm(a3, a2, 0x07030602);
        r[h * 4 + 0][q] = __builtin_amdgcn_perm(l23, l01, 0x05040100);
        r[h * 4 + 1][q] = __builtin_amdgcn_perm(l23, l01, 0x07060302);
        r[h * 4 + 2][q] = __builtin_amdgcn_perm(h23, h01, 0x05040100);
        r[h * 4 + 3][q] = __builtin_amdgcn_perm(h23, h01, 0x07060302);
      }
  }
}

template <int TENSOR>
__device__ __forceinline__ unsigned pack_unit(u32x4 (&r)[8], LAS char* lds  , int lane) {
  constexpr unsigned S = 0x80808080u, CM = TENSOR ? 0x1f1f1f1fu : 0x3f3f3f3fu;
  unsigned sum = 0;
#pragma unroll
  for (int i = 0; i < 8; ++i) {
    *(LAS u32x4*)(lds + i * 1024 + lane * 16) = r[i];
#pragma unroll
    for (int d = 0; d < 4; ++d) sum = __builtin_amdgcn_sad_u8(r[i][d] & ~S, 0u, sum);
  }
  sum += (unsigned)__builtin_amdgcn_update_dpp(0, (int)sum, 0xb1, 0xf, 0xf, true);
  sum += (unsigned)__builtin_amdgcn_update_dpp(0, (int)sum, 0x4e, 0xf, 0xf, true);
  sum += (unsigned)__builtin_amdgcn_update_dpp(0, (int)sum, 0x141, 0xf, 0xf, true);
  sum += (unsigned)__builtin_amdgcn_update_dpp(0, (int)sum, 0x140, 0xf, 0xf, true);
  const int total = __builtin_amdgcn_readlane((int)sum, 0) + __builtin_amdgcn_readlane((int)sum, 16) + __builtin_amdgcn_readlane((int)sum, 32) + __builtin_amdgcn_readlane((int)sum, 48);
  int c0 = (total - (TENSOR ? 166707 : 343245)) >> 13;
  c0 = c0 < 0 ? 0 : c0 > (TENSOR ? 96 : 64) ? (TENSOR ? 96 : 64) : c0;
  const unsigned C0 = (unsigned)c0 * 0x01010101u;
  unsigned M[4] = {0, 0, 0, 0};
#pragma unroll
  for (int i = 0; i < 8; ++i)
#pragma unroll
    for (int d = 0; d < 4; ++d) {
      const unsigned v = r[i][d], t = (v | S) - C0;
      const unsigned e = (TENSOR ? (~t | (t << 1) | (t << 2)) : (~t | (t << 1))) & S;
      M[d] |= e >> i;
      r[i][d] = (t & CM) | (v & S);
    }
#pragma unroll
  for (int d = 0; d < 4; ++d) {
    if (TENSOR == 0) {
      const unsigned q = r[7][d];
#pragma unroll
      for (int i = 0; i < 6; ++i) r[i][d] |= (q << (6 - i)) & 0x40404040u;
      r[6][d] |= (q >> 1) & 0x40404040u;
    } else {
      const unsigned q3 = r[3][d], q7 = r[7][d];
      r[0][d] |= (q3 << 5) & 0x60606060u; r[1][d] |= (q3 << 3) & 0x60606060u; r[2][d] |= ((q3 << 1) & 0x20202020u) | ((q3 >> 1) & 0x40404040u);
      r[3][d] = r[4][d] | ((q7 << 5) & 0x60606060u); r[4][d] = r[5][d] | ((q7 << 3) & 0x60606060u);
      r[5][d] = r[6][d] | ((q7 << 1) & 0x20202020u) | ((q7 >> 1) & 0x40404040u);
    }
  }
  LAS unsigned short* list = (LAS unsigned short*)(lds + 8192);
  int n = 0;
#pragma unroll
  for (int d = 0; d < 4; ++d) {
    unsigned m = M[d];
    while (n <= ECAP) {
      const unsigned long long any = __ballot(m != 0);
      if (!any) break;
      if (m != 0) {
        const int b = __builtin_ctz(m);
        m &= m - 1;
        const int idx = n + (int)__builtin_amdgcn_mbcnt_hi((unsigned)(any >> 32), __builtin_amdgcn_mbcnt_lo((unsigned)any, 0u));
        if (idx < ECAP) list[2 + idx] = (unsigned short)((7 - (b & 7)) * 1024 + lane * 16 + d * 4 + (b >> 3));
      }
      n += __builtin_popcountll(any);
    }
  }
  unsigned h = 0;
#pragma unroll
  for (int k = 0; k < 2; ++k) {
    unsigned e = 4u << 13;
    if (lane > 0 && 2 * lane - 2 + k < n) {
      const unsigned pos = list[2 * lane + k] & 0x1fffu;
      const unsigned t = (*(LAS unsigned char*)(lds + pos) | 0x80u) - (unsigned)c0;
      e = pos | (((t >> 5) & (TENSOR ? 7u : 6u)) << 13);
    }
    h |= e << (16 * k);
  }
  if (lane == 0) h = (n > ECAP ? 0xffffu : (unsigned)n) | ((unsigned)c0 << 16);
  return h;
}

template <int TENSOR>
__device__ __forceinline__ void unpack_unit(u32x4 (&r)[8], unsigned h, LAS char* lds  , int lane) {
  const unsigned C0 = ((unsigned)__builtin_amdgcn_readfirstlane((int)h) >> 16) * 0x01010101u;
#pragma unroll
  for (int d = 0; d < 4; ++d) {
    if (TENSOR == 0) {
      unsigned q = (r[6][d] << 1) & 0x80808080u;
#pragma unroll
      for (int i = 0; i < 6; ++i) q |= (r[i][d] >> (6 - i)) & (0x01010101u << i);
#pragma unroll
      for (int i = 0; i < 7; ++i) r[i][d] = (r[i][d] & 0xbfbfbfbfu) + C0;
      r[7][d] = q + C0;
    } else {
      const unsigned p0 = r[0][d], p1 = r[1][d], p2 = r[2][d], p3 = r[3][d], p4 = r[4][d], p5 = r[5][d];
      r[0][d] = (p0 & 0x9f9f9f9fu) + C0; r[1][d] = (p1 & 0x9f9f9f9fu) + C0; r[2][d] = (p2 & 0x9f9f9f9fu) + C0;
      r[3][d] = (((p0 >> 5) & 0x03030303u) | ((p1 >> 3) & 0x0c0c0c0cu) | ((p2 >> 1) & 0x10101010u) | ((p2 << 1) & 0x80808080u)) + C0;
      r[4][d] = (p3 & 0x9f9f9f9fu) + C0; r[5][d] = (p4 & 0x9f9f9f9fu) + C0; r[6][d] = (p5 & 0x9f9f9f9fu) + C0;
      r[7][d] = (((p3 >> 5) & 0x03030303u) | ((p4 >> 3) & 0x0c0c0c0cu) | ((p5 >> 1) & 0x10101010u) | ((p5 << 1) & 0x80808080u)) + C0;
    }
  }
#pragma unroll
  for (int i = 0; i < 8; ++i) *(LAS u32x4*)(lds + i * 1024 + lane * 16) = r[i];
#pragma unroll
  for (int k = 0; k < 2; ++k) {
    const unsigned e = (h >> (16 * k)) & 0xffffu, f = e >> 13;
    if (lane > 0 && f != 4u)
      __hip_atomic_fetch_add((LAS unsigned*)(lds + (e & 0x1ffcu)), (f - 4u) << (5 + 8 * (e & 3u)), __ATOMIC_RELAXED, __HIP_MEMORY_SCOPE_WORKGROUP);
  }
  asm volatile("" ::: "memory");
#pragma unroll
  for (int i = 0; i < 8; ++i) r[i] = *(const LAS u32x4*)(lds + i * 1024 + lane * 16);
}

__device__ __forceinline__ void post_peers(const Params& P, int word, int lane, unsigned call, unsigned bits) {
  if (lane < NR - 1) {
    const unsigned p = (unsigned)(P.rank + 1 + lane) & (NR - 1);
    const u32x2 v = {call, bits};
    asm volatile("global_store_dwordx2 %0, %1, off sc0 sc1" ::"v"(P.ws_all[p] + OFF_PF + P.rank * 512 + word * 8), "v"(v) : "memory");
  }
}
__device__ __forceinline__ rsrc_t peer_rsrc(const Params& P, unsigned p) {
  const unsigned long long pa = (unsigned long long)P.ws_all[p & (NR - 1)];
  const unsigned long long pu = ((unsigned long long)(unsigned)__builtin_amdgcn_readfirstlane((int)(pa >> 32)) << 32) | (unsigned)__builtin_amdgcn_readfirstlane((int)pa);
  return __builtin_amdgcn_make_buffer_rsrc((void*)pu, 0, WS_NEED, 0x00020000);
}
#define PUSHWIN 13
__device__ __forceinline__ void prep_push(const Params& P, int c, int tensor, int g, int lane, unsigned call, const u32x4 (&r)[8]) {
  rsrc_t rp[NR - 1];
#pragma unroll
  for (int d = 0; d < NR - 1; ++d) rp[d] = peer_rsrc(P, (unsigned)(P.rank + 1 + d));
  asm volatile("s_nop 4");
#pragma unroll
  for (int i = 0; i < 8; ++i)
#pragma unroll
    for (int d = 0; d < NR - 1; ++d) {
      const int ci = c * 7 + 6 - d;
      BST16(r[i], rp[d], lane * 16, slab_off(tensor, gstep_off(4 + (ci >> 3), call & 1), ci & 7, g) + (i >> 2) * 4096, (i & 3) * 1024);
      WAIT_V(PUSHWIN);
    }
  const rsrc_t rs = __builtin_amdgcn_make_buffer_rsrc((void*)P.ws, 0, WS_NEED, 0x00020000);
  store_unit(r, 8, rs, lane * 16, slab_off(tensor, gstep_off(0, call & 1), c, g));
  WAIT_V(0);
  const int ci = c * 7 + 6 - lane;
  if (lane < NR - 1) st4_sys(P.ws_all[(unsigned)(P.rank + 1 + lane) & (NR - 1)] + uflag_off(tensor, 4 + (ci >> 3), ci & 7, g), call);
  if (lane == NR - 1) st4_sys(P.ws + uflag_off(tensor, 0, c, g), call);
}
__device__ __forceinline__ void prep_store(const Params& P, LAS char* lds, int c, int tensor, int g, int lane, unsigned call, u32x4 (&r)[8]) {
  const rsrc_t rs = __builtin_amdgcn_make_buffer_rsrc((void*)P.ws, 0, WS_NEED, 0x00020000);
  int pk = OFF_PK + (int)(call & 1) * (UPP * 8192) + (c * 16 + tensor * 8 + g) * 8192;
  int raw = slab_off(tensor, gstep_off(c >> 3, call & 1), c & 7, g);
  asm volatile("" : "+s"(pk), "+s"(raw));
  const unsigned h = tensor ? pack_unit<1>(r, lds, lane) : pack_unit<0>(r, lds, lane);
  STAMP(P.ws, 30);
  const unsigned ovf = ((unsigned)__builtin_amdgcn_readfirstlane((int)h) & 0xffffu) == 0xffffu;
#define PDELAY 24
  if (c >= 4) __builtin_amdgcn_s_sleep(PDELAY);
  store_unit(r, tensor ? 6 : 7, rs, lane * 16, pk);
  asm volatile("buffer_store_dword %0, %1, %2, %3 offen sc0 sc1" ::"v"(h), "v"(lane * 4), "s"(rs), "s"(pk + PK_EXC) : "memory");
#pragma unroll
  for (int i = 0; i < 8; ++i) r[i] = *(const LAS u32x4*)(lds + i * 1024 + lane * 16);
  if (c < 4 && !ovf) { WAIT_V(0); post_peers(P, (c - 2) * 16 + tensor * 8 + g, lane, call, 0u); }
  store_unit(r, 8, rs, lane * 16, raw);
  unsigned* pc = (unsigned*)(P.ws + OFF_PCNT + (c >> 1) * 128);
  if (ovf && c >= 4 && lane == 0) __hip_atomic_fetch_or(pc + 1 + (call & 1), 1u << ((c & 1) * 16 + tensor * 8 + g), __ATOMIC_RELAXED, __HIP_MEMORY_SCOPE_AGENT);
  WAIT_V(0);
  if (lane == 0) st4_sys(P.ws + uflag_off(tensor, c >> 3, c & 7, g), call);
  if (c < 4) { if (ovf) post_peers(P, (c - 2) * 16 + tensor * 8 + g, lane, call, ~0u); return; }
  unsigned tk = 0;
  ticket_issue(tk, pc, 1);
  WAIT_V(0); asm volatile("" : "+v"(tk));
  if ((unsigned)__builtin_amdgcn_readfirstlane((int)tk) == call * 32 - 1) {
    post_peers(P, 30 + (c >> 1), lane, call, ld4_sys(pc + 1 + (call & 1)));
    if (lane == 0) st4_sys(pc + 1 + (~call & 1), 0u);
  }
}

#define RAMP 12
#define LEADX 0, 0, 0, 0, 0, 0, 0, 0
#define LEADW 0, 0, 0, 0, 0, 0, 0, 0
struct Order { int e[NPULL]; };
constexpr Order make_order() {
  constexpr int lead[16] = {LEADX, LEADW};
  Order o{};
  int next[16] = {};
  for (int t = 0; t < 16; ++t) next[t] = RAWC;
  for (int k = 0; k < NPULL; ++k) {
    int best = -1, key = 0;
    for (int v = 0; v < 16; ++v) {
      const int t = (v >> 1) + 8 * (v & 1);
      const int kt = 4 * RAMP * next[t] - lead[t] * (next[t] - RAWC < RAMP ? next[t] - RAWC : RAMP) +
                     (t >= 4 && t < 8 && 7 * next[t] >= 8 * (JSTEP - 4) ? 1 << 20 : 0);
      if (next[t] < 32 && (best < 0 || kt < key)) { best = t; key = kt; }
    }
    o.e[k] = next[best] * 16 + best;
    ++next[best];
  }
  return o;
}
__constant__ const Order ORDER = make_order();

constexpr int NDEC = 3;
constexpr int NSLOT = 6, SLOTB = 8192 + 256;
constexpr int RING_CTL = 2 * NSLOT * SLOTB;
static_assert(WIN >= 3 && WIN <= 8 && NSLOT % NDEC == 0 && RING_CTL + 256 <= LDS_BYTES && LPP * 7 <= 2 * NCOMM, "ring");
__device__ __forceinline__ unsigned lds_ld(unsigned a) {
  unsigned v; asm volatile("ds_read_b32 %0, %1\n\ts_waitcnt lgkmcnt(0)" : "=v"(v) : "v"(a) : "memory");
  return (unsigned)__builtin_amdgcn_readfirstlane((int)v);
}
__device__ __forceinline__ void lds_st(unsigned a, unsigned v) { asm volatile("ds_write_b32 %0, %1" ::"v"(a), "v"(v) : "memory"); }

__device__ __forceinline__ void loader_main(const Params& P, char* shm, int l, int lid, int lane, unsigned call) {
  const unsigned ctl = (unsigned)(size_t)(LAS char*)shm + RING_CTL + l * 128;
  if (lid >= LPP * 7) { lds_st(ctl + 4, 0); return; }
  const int q = lid % 7, li = lid / 7;
  const unsigned p = (unsigned)(P.rank + 1 + q) & (NR - 1);
  const rsrc_t rp = peer_rsrc(P, p);
  const char* pfp = P.ws + OFF_PF + p * 512 + lane * 8;
  unsigned* claim = (unsigned*)(P.ws + OFF_CLAIM + q * 128);
  const unsigned cbase = (call - 1) * NPULL - 2 * LPP;
  const int pkb = OFF_PK + (int)(call & 1) * (UPP * 8192);
  char* ring = shm + l * NSLOT * SLOTB;
  unsigned long long ready = 0;
  u32x2 pf = {0u, 0u};
  bool dead = false;
  int n = 0, slot = 0, src, nl;
  auto setup = [&](int e, int un, int sl, int landed) {
    const int w = e < 64 ? e - 32 : 30 + (e >> 5);
    if (!dead && !((ready >> w) & 1)) {
      WAIT_V(0); lds_st(ctl, (unsigned)landed);
      const unsigned long long t0 = __builtin_amdgcn_s_memrealtime();
      while (true) {
        asm volatile("global_load_dwordx2 %0, %1, off sc0 sc1\n\ts_waitcnt vmcnt(0)" : "=&v"(pf) : "v"(pfp) : "memory");
        ready = __ballot(pf.x >= call);
        if ((ready >> w) & 1) break;
        if (spin_fail(P.ws, call, t0)) { dead = true; break; }
        __builtin_amdgcn_s_sleep(1);
      }
    }
    if (un >= NSLOT && (int)lds_ld(ctl + 8 + (un % NDEC) * 4) <= un - NSLOT) {
      const unsigned long long t0 = __builtin_amdgcn_s_memrealtime();
      while ((int)lds_ld(ctl + 8 + (un % NDEC) * 4) <= un - NSLOT && __builtin_amdgcn_s_memrealtime() - t0 < 2 * DEADLINE) {}
    }
    const int c = e >> 4, tensor = (e >> 3) & 1, g = e & 7, ci = c * 7 + q, step = 4 + (ci >> 3);
    const bool raw = ((unsigned)__builtin_amdgcn_readlane((int)pf.y, w) >> (e & 31)) & 1u;
    nl = raw ? 8 : 7 - tensor;
    src = __builtin_amdgcn_readfirstlane(raw ? slab_off(tensor, gstep_off(c >> 3, call & 1), c & 7, g) : pkb + e * 8192);
    const u32x4 dsc = {(unsigned)nl, (unsigned)slab_off(tensor, gstep_off(step, call & 1), ci & 7, g), (unsigned)uflag_off(tensor, step, ci & 7, g), 0u};
    if (lane == 0) asm volatile("ds_write_b128 %0, %1" ::"v"(ctl + 32 + sl * 16), "v"(dsc) : "memory");
  };
  unsigned tk = 0;
  int e = ORDER.e[li];
  setup(e, 0, 0, 0);
  STAMP(P.ws, 2);
  while (true) {
    const int s0 = src, n0 = nl;
    char* lp = ring + slot * SLOTB;
#define PLANE(i) do { WAIT_V(WIN - 1); if ((i) == WIN - 3) lds_st(ctl, (unsigned)n);   \
      __builtin_amdgcn_raw_ptr_buffer_load_lds(rp, (LAS void*)(lp + ((i) >> 2) * 4096), 16, lane * 16, s0 + ((i) >> 2) * 4096, ((i) & 3) * 1024, 17); } while (0)
    ticket_issue(tk, claim, 1ull << (n & 1));
    __builtin_amdgcn_raw_ptr_buffer_load_lds(rp, (LAS void*)(lp + 8192), 4, lane * 4, pkb + e * 8192 + PK_EXC, 0, 17);
    PLANE(0); PLANE(1);
    asm volatile("" : "+v"(tk));
    const int kn = n == 0 ? li + LPP : (int)((unsigned)__builtin_amdgcn_readlane((int)tk, ~n & 1) - cbase);
    const bool more = kn < NPULL;
    e = ORDER.e[more ? kn : 0];
    PLANE(2); PLANE(3); PLANE(4); PLANE(5);
    if (n0 > 6) PLANE(6);
    if (n0 > 7) PLANE(7);
    ++n; slot = slot == NSLOT - 1 ? 0 : slot + 1;
    if (!more) break;
    setup(e, n, slot, n);
  }
  WAIT_V(0);
  asm volatile("" : "+v"(tk));
  lds_st(ctl, (unsigned)n);
  lds_st(ctl + 4, (unsigned)n);
  DBG(STAMPV(P.ws, 10, (unsigned long long)n);)
}

__device__ __forceinline__ void decoder_main(const Params& P, char* shm, int l, int j, int lane, unsigned call) {
  const unsigned ctl = (unsigned)(size_t)(LAS char*)shm + RING_CTL + l * 128;
  const rsrc_t rs = __builtin_amdgcn_make_buffer_rsrc((void*)P.ws, 0, WS_NEED, 0x00020000);
  const unsigned long long t0 = __builtin_amdgcn_s_memrealtime();
  DBG(unsigned long long tdec = 0; unsigned long long tst = 0; unsigned long long cntu = 0;)
  for (int n = j;; n += NDEC) {
    for (int spin = 0; (int)lds_ld(ctl) <= n; ++spin)
      if ((spin & 15) == 15 && (lds_ld(ctl + 4) <= (unsigned)n || __builtin_amdgcn_s_memrealtime() - t0 > 4 * DEADLINE)) return;
    DBG(unsigned long long ta = NOW();)
    const int slot = n % NSLOT;
    LAS char* sp = (LAS char*)shm + (l * NSLOT + slot) * SLOTB;
    const u32x4 dsc = *(const LAS u32x4*)((LAS char*)shm + RING_CTL + l * 128 + 32 + slot * 16);
    const unsigned h = *(const LAS unsigned*)(sp + 8192 + lane * 4);
    u32x4 r[8];
#pragma unroll
    for (int i = 0; i < 8; ++i) r[i] = *(const LAS u32x4*)(sp + i * 1024 + lane * 16);
    const int nl = __builtin_amdgcn_readfirstlane((int)dsc.x), dst = __builtin_amdgcn_readfirstlane((int)dsc.y), flag = __builtin_amdgcn_readfirstlane((int)dsc.z);
    if (nl == 7) unpack_unit<0>(r, h, sp, lane);
    else if (nl == 6) unpack_unit<1>(r, h, sp, lane);
    WAIT_L(0);
#pragma unroll
    for (int i = 0; i < 8; ++i) asm volatile("" : "+v"(r[i]));
    lds_st(ctl + 8 + j * 4, (unsigned)(n + 1));
    DBG(unsigned long long tb = NOW();)
    asm volatile("s_nop 4");
    store_unit(r, 8, rs, lane * 16, dst);
    WAIT_V(0);
    if (lane == 0) st4_sys(P.ws + flag, call);
    DBG(tdec += tb - ta; tst += NOW() - tb; ++cntu; STAMPV(P.ws, 8, tdec); STAMPV(P.ws, 9, tst); STAMPV(P.ws, 10, cntu);)
  }
}

typedef float f32x2 __attribute__((ext_vector_type(2)));
__device__ __forceinline__ f32x2 gate2(f32x2 a, float ke, float ys) {
  constexpr float LIM = 60.f * 1.44269504088896341f, EPS = 0x1p-24f;
  const f32x2 t = a * ke;
  const f32x2 e = {__builtin_amdgcn_exp2f(__builtin_amdgcn_fmed3f(t.x, -LIM, LIM)), __builtin_amdgcn_exp2f(__builtin_amdgcn_fmed3f(t.y, -LIM, LIM))};
  const f32x2 d = e * EPS + EPS;
  const float r = __builtin_amdgcn_rcpf(d.x * d.y) * ys;
  const f32x2 sw = {d.y, d.x};
  return sw * r;
}

#define SPREAD TAILS
constexpr int ufirst(int ph, int nu) { return ph >= 2 * SPREAD ? nu : (ph * nu + 2 * SPREAD - 1) / (2 * SPREAD); }
template <int I, int E, typename F> __device__ __forceinline__ void static_for(F&& f) {
  if constexpr (I < E) { f(std::integral_constant<int, I>{}); static_for<I + 1, E>(f); }
}

template <int MA0, int MA1>
__device__ __forceinline__ void gemm_main(const Params& P, char* shm, int browA, int browB, int bcol, int wid, int lane, unsigned call) {
  constexpr int RA0 = 32 * MA0, RA1 = 32 * MA1;
  constexpr int SLA0 = RA0 * 16, SLA1 = RA1 * 16, SLB = 128 * 16;
  constexpr int A0B = 8 * SLA0, A1B = 8 * SLA1, BB = 8 * SLB, BUFB = A0B + A1B + 2 * BB;
  constexpr int NA0 = (RA0 + 63) / 64, NA1 = (RA1 + 63) / 64;
  constexpr int S = TAILS, J = JSTEP, END = NSTEP + S;
  static_assert(2 * BUFB <= LDS_BYTES && S % 2 == 0 && S >= 2 && J >= 8, "shape");
  const int wr = wid >> 2, wc = wid & 3, fr = lane & 15, fq = lane >> 4;
  const unsigned par = call & 1;
  auto rX = __builtin_amdgcn_make_buffer_rsrc((void*)(P.ws + OFF_GX), 0, GSTEPS * STEPB, 0x00020000);
  auto rW = __builtin_amdgcn_make_buffer_rsrc((void*)(P.ws + OFF_GW), 0, GSTEPS * STEPB, 0x00020000);
  auto rU = __builtin_amdgcn_make_buffer_rsrc((void*)P.ws, 0, OFF_DBG, 0x00020000);
  auto rC = __builtin_amdgcn_make_buffer_rsrc((void*)P.out, 0, DIM * DIM * 4, 0x00020000);
  const int voff = lane * 16;
  const int sA0 = wid * SLAB + browA * 16, sA1 = wid * SLAB + browB * 16, sB = wid * SLAB + bcol * 16;
  constexpr unsigned M0 = 0x13, M1 = 0x0c, MALL = 0x1f, MBAD = 0x20;
  int pollv;
  { const int tg = lane == 0 ? browA >> 9 : lane == 1 ? (browA + RA0 - 1) >> 9 : lane == 2 ? browB >> 9 : lane == 3 ? (browB + RA1 - 1) >> 9 : 8 + (bcol >> 9);
    pollv = lane == 5 ? OFF_TMO : OFF_UF + (wid * 16 + tg) * 128; }
  unsigned pv = 0, got = 0;
  bool dead = false;
  const float scale = P.sx[0] * P.sw[0];
  int kei, ysi, isi;
  asm volatile("s_nop 1\n\tv_readfirstlane_b32 %0, %1" : "=s"(isi) : "v"(1.0f / scale));
  asm volatile("s_nop 1\n\tv_readfirstlane_b32 %0, %2\n\tv_readfirstlane_b32 %1, %3" : "=s"(kei), "=s"(ysi) : "v"(scale * -1.44269504088896341f), "v"(scale * 0x1p-24f));
  int t4 = 0;
#define POLL(step) asm volatile("buffer_load_dword %0, %1, %2, %3 offen offset:%4 sc0 sc1" : "=v"(pv) : "v"(pollv), "s"(rU), "s"(t4), "i"((step) * 4) : "memory")
#define TAKE() do { asm volatile("" : "+v"(pv)); got = (unsigned)__ballot(pv >= call); } while (0)
#define NEED(step, m) do { if ((got & (m)) != (m) && !dead) { const unsigned long long t0_ = __builtin_amdgcn_s_memrealtime(); \
    while (true) { unsigned sv_;     \
      asm volatile("buffer_load_dword %0, %1, %2, %3 offen offset:%4 sc0 sc1\n\ts_waitcnt vmcnt(0)" : "=&v"(sv_) : "v"(pollv), "s"(rU), "s"(t4), "i"((step) * 4) : "memory"); \
      got = (unsigned)__ballot(sv_ >= call); if ((got & (m)) == (m)) break; \
      if (got & MBAD) { dead = true; break; } \
      if (__builtin_amdgcn_s_memrealtime() - t0_ > DEADLINE) { give_up(P.ws, call); dead = true; break; } \
      __builtin_amdgcn_s_sleep(1); } \
    WAIT_L(0); } } while (0)
#define OFF_A0(b) ((b) * BUFB)
#define OFF_A1(b) ((b) * BUFB + A0B)
#define OFF_B(b, h) ((b) * BUFB + A0B + A1B + (h) * BB)
#define GL(rs, ldsoff, soff, imm) __builtin_amdgcn_raw_ptr_buffer_load_lds(rs, (LAS void*)(shm + (ldsoff)), 16, voff, soff, imm, 17)
#define STAGE_A0(b, go) do { const int so_ = sA0 + (go), lo_ = OFF_A0(b) + wid * SLA0; GL(rX, lo_, so_, 0); GL(rX, lo_, so_, 1024); \
    if (NA0 == 3) { if (lane < 32) GL(rX, lo_, so_, 2048); } } while (0)
#define STAGE_A1(b, go) do { const int so_ = sA1 + (go), lo_ = OFF_A1(b) + wid * SLA1; GL(rX, lo_, so_, 0); GL(rX, lo_, so_, 1024); \
    if (NA1 == 3) { if (lane < 32) GL(rX, lo_, so_, 2048); } } while (0)
#define STAGE_B(b, h, go) do { const int so_ = sB + (h) * 2048 + (go), lo_ = OFF_B(b, h) + wid * SLB; GL(rW, lo_, so_, 0); GL(rW, lo_, so_, 1024); } while (0)
  const LAS char* lsm = (const LAS char*)shm;
  const LAS char *pa0[2], *pa1[2], *pb[2];
#pragma unroll
  for (int b = 0; b < 2; ++b) {
    pa0[b] = lsm + OFF_A0(b) + fq * SLA0 + (wr * 16 * MA0 + fr) * 16;
    pa1[b] = lsm + OFF_A1(b) + fq * SLA1 + (wr * 16 * MA1 + fr) * 16;
    pb[b] = lsm + OFF_B(b, 0) + fq * SLB + (wc * 16 + fr) * 16;
    asm volatile("" : "+v"(pa0[b])); asm volatile("" : "+v"(pa1[b])); asm volatile("" : "+v"(pb[b]));
  }
#define LD8(p, sl) __builtin_shufflevector(*(const LAS v4i*)(p), *(const LAS v4i*)((p) + 4 * (sl)), 0, 1, 2, 3, 4, 5, 6, 7)
#define LDA0(b) _Pragma("unroll") for (int m = 0; m < MA0; ++m) At[m] = LD8(pa0[b] + m * 256, SLA0)
#define LDA1(b) _Pragma("unroll") for (int m = 0; m < MA1; ++m) At[m] = LD8(pa1[b] + m * 256, SLA1)
#define LDB(dst, b, h) _Pragma("unroll") for (int n = 0; n < 2; ++n) dst[n] = LD8(pb[b] + (h) * BB + n * 1024, SLB)
#define MFMA(c, a, b) asm volatile("v_mfma_f32_16x16x128_f8f6f4 %0, %1, %2, %0 blgp:1" : "+v"(c) : "v"(a), "v"(b))
#define MMA0(bj, Bf) do { __builtin_amdgcn_s_setprio(1); _Pragma("unroll") for (int m = 0; m < MA0; ++m) _Pragma("unroll") for (int n = 0; n < 2; ++n) \
    MFMA(acc0[m][(bj) * 2 + n], At[m], Bf[n]); __builtin_amdgcn_s_setprio(0); } while (0)
#define MMA1(bj, Bf) do { __builtin_amdgcn_s_setprio(1); _Pragma("unroll") for (int m = 0; m < MA1; ++m) _Pragma("unroll") for (int n = 0; n < 2; ++n) \
    MFMA(acc1[m][(bj) * 2 + n], At[m], Bf[n]); __builtin_amdgcn_s_setprio(0); } while (0)
#define TILE(b, nb, t, r) do { TAKE(); NEED((r) + 1, MALL); const int go_ = gstep_off((t) + 1, par); \
    LDB(B0, b, 0); LDA0(b); STAGE_B(nb, 0, go_); STAGE_A0(nb, go_); POLL((r) + 2); \
    WAIT_V(NA1 + 2 + NA0 + 1); BAR; WAIT_L(0); MMA0(0, B0); BAR; SCHED; \
    LDB(B1, b, 1); STAGE_B(nb, 1, go_); STAGE_A1(nb, go_); \
    WAIT_V(2 + NA0 + 1 + 2 + NA1); BAR; WAIT_L(0); MMA0(1, B1); BAR; SCHED; \
    LDA1(b); BAR; WAIT_L(0); MMA1(0, B0); BAR; SCHED; \
    WAIT_V(2 + NA1); BAR; MMA1(1, B1); BAR; SCHED; } while (0)
#define OUTPOL "sc1"
#define ST_ROW(o, row) do { const int so_ = ((row) * DIM + bcol + wc * 64) * 4; \
    asm volatile("buffer_store_dwordx4 %0, %1, %2, %3 offen " OUTPOL "\n\ts_nop 1" ::"v"(o), "v"(cv), "s"(rC), "s"(so_) : "memory"); } while (0)
#define EARLYPOL OUTPOL
#define ST_ROWP(o, row, POL, vo) do { const int so_ = ((row) * DIM + bcol + wc * 64) * 4; \
    asm volatile("buffer_store_dwordx4 %0, %1, %2, %3 offen " POL "\n\ts_nop 1" ::"v"(o), "v"(vo), "s"(rC), "s"(so_) : "memory"); } while (0)
#define ST_PAIR(acc, j, row0) do { f32x4 o0_, o1_; \
    _Pragma("unroll") for (int u = 0; u < 4; ++u) { const f32x2 a_ = {acc[u][j], acc[u][(j) + 1]}, g_ = gate2(a_, __builtin_bit_cast(float, kei), __builtin_bit_cast(float, ysi)); \
      float p_, q_; asm("v_mul_f32 %0, %1, %2" : "=v"(p_) : "v"(a_.x), "v"(g_.x)); asm("v_mul_f32 %0, %1, %2" : "=v"(q_) : "v"(a_.y), "v"(g_.y)); \
      o0_[u] = p_; o1_[u] = q_; } \
    ST_ROW(o0_, (row0) + (j)); ST_ROW(o1_, (row0) + (j) + 1); } while (0)
  f32x4 acc0[MA0][4] = {}, acc1[MA1][4] = {};
  v8i At[MA0 > MA1 ? MA0 : MA1], B0[2], B1[2];
  NEED(0, MALL);
  STAMP(P.ws, 2);
  { const int go = gstep_off(0, par);
    STAGE_B(0, 0, go); STAGE_A0(0, go); POLL(1); STAGE_B(0, 1, go); STAGE_A1(0, go); }
  if (wr == 1) BAR;
  WAIT_V(2 + NA1); BAR; BAR; SCHED;
  for (int t = 0; t < J - 2; t += 2) {
    asm volatile("s_lshl_b32 %0, %1, 2" : "=s"(t4) : "s"(t) : "scc");
    TILE(0, 1, t, 0);
    TILE(1, 0, t + 1, 1);
  }
  t4 = 0;
  TILE(0, 1, J - 2, J - 2);
  {
    TAKE(); NEED(J, M0);
    LDB(B0, 1, 0); LDA0(1); STAGE_B(0, 0, (J + PSTEPS) * STEPB); STAGE_A0(0, (J + PSTEPS) * STEPB); POLL(J + 1);
    WAIT_V(NA1 + 2 + NA0 + 1); BAR; WAIT_L(0); MMA0(0, B0); BAR; SCHED;
    LDB(B1, 1, 1); STAGE_B(0, 1, (J + PSTEPS) * STEPB);
    WAIT_V(2 + NA0 + 1 + 2); BAR; WAIT_L(0); MMA0(1, B1); BAR; SCHED;
    LDA1(1); BAR; WAIT_L(0); MMA1(0, B0); BAR; SCHED;
    WAIT_V(2); TAKE(); NEED(J + 1, M0); POLL(J + 2); STAGE_B(1, 0, (J + 1 + PSTEPS) * STEPB); STAGE_A0(1, (J + 1 + PSTEPS) * STEPB);
    BAR; MMA1(1, B1); BAR; SCHED; }
  int tz = threadIdx.x; asm volatile("" : "+v"(tz));
  const int cv = (((tz & 63) >> 4) * 4 * DIM + (tz & 15) * 4) * 4;
  float lim = 60.f * 1.44269504088896341f; asm volatile("" : "+v"(lim));
  f32x4 o0, o1;
#define PREOPS 3
  constexpr int VMAX = 2 * ((8 * MA0 + 2 * SPREAD - 1) / (2 * SPREAD));
  float tv[VMAX];
  auto epi_op = [&](auto opc, auto uc, auto vc) {
    constexpr int op = decltype(opc)::value, u = decltype(uc)::value, v = decltype(vc)::value;
    const float a = acc0[u >> 3][u & 3][(u >> 2 & 1) * 2 + (v & 1)];
    float& t = tv[v];
    if constexpr (op == 0) asm volatile("v_mul_f32 %0, %1, %2" : "=v"(t) : "s"(kei), "v"(a));
    if constexpr (op == 1) asm volatile("v_med3_f32 %0, %0, %1, %2" : "+v"(t) : "s"(0xc2ad1f97), "v"(lim));
    if constexpr (op == 2) asm volatile("v_exp_f32 %0, %0" : "+v"(t));
    if constexpr (op == 3) asm volatile("v_fma_f32 %0, %0, %1, %1" : "+v"(t) : "s"(isi));
    if constexpr (op == 4) asm volatile("v_rcp_f32 %0, %0" : "+v"(t));
    if constexpr (op == 5) {
      float r; asm volatile("v_mul_f32 %0, %1, %2" : "=v"(r) : "v"(a), "v"(t));
      if constexpr (v & 1) o1[u & 3] = r; else o0[u & 3] = r;
      if constexpr ((u & 3) == 3 && (v & 1)) {
        const int row = browA + wr * 16 * MA0 + (u >> 3) * 16 + (u >> 2 & 1) * 2;
        ST_ROWP(o0, row, EARLYPOL, cv); ST_ROWP(o1, row + 1, EARLYPOL, cv);
      }
    }
  };
  auto pre = [&](auto phc) {
    constexpr int ph = decltype(phc)::value, u0 = ufirst(ph, 8 * MA0), V = 2 * (ufirst(ph + 1, 8 * MA0) - u0);
    static_for<0, PREOPS * V>([&](auto kc) {
      constexpr int k = decltype(kc)::value;
      epi_op(std::integral_constant<int, k / V>{}, std::integral_constant<int, u0 + (k % V >> 1)>{}, std::integral_constant<int, k % V>{});
    });
  };
  auto cluster = [&](auto phc, auto bjc, v8i (&Bf)[2]) {
    constexpr int ph = decltype(phc)::value, bj = decltype(bjc)::value, G = 2 * MA1;
    constexpr int u0 = ufirst(ph, 8 * MA0), V = 2 * (ufirst(ph + 1, 8 * MA0) - u0);
    constexpr auto cost = [](int k) { const int op = k / (V ? V : 1), v = k % (V ? V : 1);
      return 4 * k + 4 * (op > 2 ? V : op == 2 ? v : 0) + 4 * (op > 4 ? V : op == 4 ? v : 0); };
    __builtin_amdgcn_s_setprio(1);
    static_for<0, G>([&](auto gc) {
      constexpr int g = decltype(gc)::value;
      MFMA(acc1[g >> 1][bj * 2 + (g & 1)], At[g >> 1], Bf[g & 1]);
      static_for<PREOPS * V, 6 * V>([&](auto kc) {
        constexpr int k = decltype(kc)::value;
        if constexpr ((cost(k) - cost(PREOPS * V)) * (G - 1) / (32 * V - cost(PREOPS * V)) == g)
          epi_op(std::integral_constant<int, k / V>{}, std::integral_constant<int, u0 + (k % V >> 1)>{}, std::integral_constant<int, k % V>{});
      });
    });
    __builtin_amdgcn_s_setprio(0);
  };
  unsigned pw[2] = {pv, pv};
  static_for<J, END>([&](auto ic) {
    constexpr int i = decltype(ic)::value, b = i & 1, nb = b ^ 1;
    constexpr bool h = i >= NSTEP;
    constexpr int MA = h ? MA1 : MA0;
    constexpr auto step = [](int k) { return k < NSTEP ? k : k - S; };
    constexpr auto nst = [](int k, int q) {
      const int ph = 2 * (k - NSTEP) + q;
      return ph < 0 ? 0 : 2 * (ufirst(ph + 1, 8 * MA0) / 4 - ufirst(ph, 8 * MA0) / 4); };
    constexpr auto nq1 = [](int k) { return (k + 1 < END ? 2 : 0) + (k + 3 < END ? 1 : 0); };
    constexpr auto nq2 = [](int k) { return k + 2 < END ? 2 + (k + 2 < NSTEP ? NA0 : NA1) : 0; };
    const int g1 = sB + 2048 + (step(i + 1) + PSTEPS) * STEPB, l1 = OFF_B(nb, 1) + wid * SLB;
    if constexpr (i + 1 < END) GL(rW, l1, g1, 0);
    SCHED; LDB(B0, b, 0); SCHED;
    if constexpr (i + 1 < END) GL(rW, l1, g1, 1024);
    SCHED;
    static_for<0, MA>([&](auto mc) {
      constexpr int m = decltype(mc)::value;
      if constexpr (h) At[m] = LD8(pa1[b] + m * 256, SLA1); else At[m] = LD8(pa0[b] + m * 256, SLA0);
      if constexpr (m == 1 && i + 3 < END) {
        SCHED; asm volatile("buffer_load_dword %0, %1, %2, 0 offen offset:%3 sc0 sc1" : "=v"(pw[b]) : "v"(pollv), "s"(rU), "i"(step(i + 3) * 4) : "memory"); SCHED; }
    });
    WAIT_V(nst(i - 1, 0) + nq2(i - 1) + nst(i - 1, 1) + nq1(i));
    asm volatile("" : "+v"(pw[nb])); got = (unsigned)__ballot(pw[nb] >= call);
    if constexpr (i == NSTEP) {
      STAMP(P.ws, 7);
      if (dead || (got & MBAD)) isi = 0x7fc00000;
      asm volatile("s_nop 15\n\ts_nop 15"); SCHED;
    }
    if constexpr (h) pre(std::integral_constant<int, 2 * (i - NSTEP)>{});
    WAIT_L(0); BAR;
    if constexpr (h) cluster(std::integral_constant<int, 2 * (i - NSTEP)>{}, std::integral_constant<int, 0>{}, B0); else MMA0(0, B0);
    BAR; SCHED;
    if constexpr (i + 2 < END) {
      constexpr bool h2 = i + 2 >= NSTEP;
      NEED(step(i + 2), (h2 ? M1 : M0));
      const int go = (step(i + 2) + PSTEPS) * STEPB;
      const int ga = (h2 ? sA1 : sA0) + go, la = h2 ? OFF_A1(b) + wid * SLA1 : OFF_A0(b) + wid * SLA0, gb = sB + go, lb = OFF_B(b, 0) + wid * SLB;
      GL(rX, la, ga, 0); SCHED;
      B1[0] = LD8(pb[b] + BB, SLB); SCHED;
      GL(rX, la, ga, 1024); SCHED;
      B1[1] = LD8(pb[b] + BB + 1024, SLB); SCHED;
      if ((h2 ? NA1 : NA0) == 3) { if (lane < 32) GL(rX, la, ga, 2048); }
      GL(rW, lb, gb, 0); GL(rW, lb, gb, 1024);
    } else {
      LDB(B1, b, 1);
    }
    if constexpr (i + 1 < END) WAIT_V(nst(i - 1, 1) + nq1(i) + nst(i, 0) + nq2(i));
    if constexpr (h) pre(std::integral_constant<int, 2 * (i - NSTEP) + 1>{});
    WAIT_L(0); BAR;
    if constexpr (h) cluster(std::integral_constant<int, 2 * (i - NSTEP) + 1>{}, std::integral_constant<int, 1>{}, B1); else MMA0(1, B1);
    if constexpr (i + 1 < END) { BAR; SCHED; }
  });
  asm volatile("s_nop 15\n\ts_nop 15"); SCHED;
  if (wr == 0) BAR;
  STAMP(P.ws, 3);
  POLL(0); WAIT_V(0); TAKE();
  if (dead || (got & MBAD)) kei = ysi = 0x7fc00000;
#pragma unroll
  for (int m = 0; m < MA1; ++m) { ST_PAIR(acc1[m], 0, browB + wr * 16 * MA1 + m * 16); ST_PAIR(acc1[m], 2, browB + wr * 16 * MA1 + m * 16); }
}

__global__ __launch_bounds__(512, 2) void fused_kernel(Params P) {
  extern __shared__ __attribute__((aligned(16))) char shm[];
  const int b = blockIdx.x;
  const int wid = __builtin_amdgcn_readfirstlane(threadIdx.x >> 6), lane = threadIdx.x & 63;
  STAMP(P.ws, 0);
  unsigned cnt;
  asm volatile("global_load_dword %0, %1, off sc0 sc1" : "=v"(cnt) : "v"(P.ws + OFF_CNT) : "memory");
  int pc = -1;
  const int j = b - NCOMM;
  if (b < NCOMM) { if (wid == 7) pc = b >> 4; }
  else if (wid < 2) pc = RAWC + 2 * (j >> 4) + wid;
  else if (wid == 2 && j < 32) pc = 30 + (j >> 4);
  const int ptensor = (b >> 3) & 1, pg = b & 7;
  u32x4 r[8];
  if (pc >= 0) prep_load(P, pc, ptensor, pg, lane, r);
  WAIT_V(0); asm volatile("" : "+v"(cnt));
  const unsigned call = (unsigned)__builtin_amdgcn_readfirstlane((int)cnt) / NBLK + 1;
  STAMP(P.ws, 6);
  if (b < NCOMM) {
    if (wid == 0) *(LAS unsigned*)((LAS char*)shm + RING_CTL + lane * 4) = (lane & 31) == 1 ? 0xffffffffu : 0u;
    WAIT_L(0); BAR;
  }
  if (pc >= RAWC) prep_store(P, (LAS char*)shm + wid * 8448, pc, ptensor, pg, lane, call, r);
  else if (pc >= 0) prep_push(P, pc, ptensor, pg, lane, call, r);
  STAMP(P.ws, 1);
  if (b < NCOMM) {
    if (wid < 2) loader_main(P, shm, wid, b * 2 + wid, lane, call);
    else decoder_main(P, shm, wid & 1, (wid - 2) >> 1, lane, call);
    if (threadIdx.x == 0) __hip_atomic_fetch_add((unsigned*)(P.ws + OFF_CNT), 1u, __ATOMIC_RELAXED, __HIP_MEMORY_SCOPE_AGENT);
    STAMP(P.ws, 5);
    return;
  }
  WAIT_V(0); WAIT_L(0); BAR;
  const int xcd = b & 7, idx = j >> 3;
  const int rb = idx % 7, bcol = ((xcd >> 1) * 4 + idx / 7) * 256;
  const int browA = (xcd & 1) * 1024 + (rb <= 4 ? rb * 160 : 128 + rb * 128);
  const int browB = 2048 + (xcd & 1) * 1024 + (rb == 0 ? 0 : rb <= 4 ? 32 + rb * 128 : rb * 160 - 96);
  if (rb == 0) gemm_main<5, 5>(P, shm, browA, browB, bcol, wid, lane, call);
  else if (rb < 4) gemm_main<5, 4>(P, shm, browA, browB, bcol, wid, lane, call);
  else gemm_main<4, 5>(P, shm, browA, browB, bcol, wid, lane, call);
  if (threadIdx.x == 0) __hip_atomic_fetch_add((unsigned*)(P.ws + OFF_CNT), 1u, __ATOMIC_RELAXED, __HIP_MEMORY_SCOPE_AGENT);
}

}

extern "C" void kernel_launch(int rank, int nranks, ncclComm_t comm, hipStream_t stream, void* const* d_in, const int* in_sizes, int n_in,
                              void* d_out, int out_size, const int* in_dtypes, int out_dtype, void* const* d_out_all,
                              void* const* d_ws_all, size_t ws_bytes) {
  if (nranks != NR || n_in != 4 || in_sizes[0] != DIM * KSH || in_sizes[1] != KSH * DIM || out_size != DIM * DIM || ws_bytes < (size_t)WS_NEED) {
    fprintf(stderr, "unexpected problem shape\n");
    return;
  }
  constexpr int lds = LDS_BYTES;
  HIP_CHECK(hipFuncSetAttribute((const void*)fused_kernel, hipFuncAttributeMaxDynamicSharedMemorySize, lds));
  Params P;
  P.x = (const unsigned char*)d_in[0];
  P.w = (const unsigned char*)d_in[1];
  P.sx = (const float*)d_in[2];
  P.sw = (const float*)d_in[3];
  P.out = (float*)d_out;
  P.ws_all = (char* const*)d_ws_all;
  P.ws = (char*)d_ws_all[rank];
  P.rank = rank;
  P.pad = 0;
  hipLaunchKernelGGL(fused_kernel, dim3(NBLK), dim3(512), lds, stream, P);
  HIP_CHECK(hipGetLastError());
}
```

```cpp
#include <hip/hip_runtime.h>
#include <rccl/rccl.h>
#include <cstdio>
#include <type_traits>

#define HIP_CHECK(expr) do { hipError_t _e = (expr); if (_e != hipSuccess) { \
    fprintf(stderr, "HIP error %s at %s:%d\n", hipGetErrorString(_e), __FILE__, __LINE__); \
    return; } } while (0)

namespace {

constexpr int NR = 8;
constexpr int DIM = 4096;
constexpr int KSH = DIM / NR;
constexpr int NSTEP = 32;
constexpr int SLAB = DIM * 16;
constexpr int STEPB = 8 * SLAB;
constexpr int NBLK = 256, NCOMM = 32;
constexpr int UPP = 512;
constexpr int RAWC = 2;
constexpr int NPULL = UPP - 16 * RAWC;
#define LPP 9
#define WIN 7
#define TAILS 4
constexpr int JSTEP = NSTEP - TAILS;
constexpr int OFF_CNT = 0, OFF_TMO = 128, OFF_CLAIM = 256;
constexpr int OFF_PCNT = 2048;
constexpr int OFF_PF = 4096;
constexpr int OFF_UF = 8192;
constexpr int OFF_DBG = 65536;
constexpr int PSTEPS = 6;
constexpr int GSTEPS = NSTEP + PSTEPS;
constexpr int OFF_GX = 1 << 20, OFF_GW = OFF_GX + GSTEPS * STEPB;
constexpr int OFF_PK = OFF_GW + GSTEPS * STEPB;
constexpr int WS_NEED = OFF_PK + 2 * UPP * 8192;
constexpr int LDS_BYTES = 2 * (128 * 320 + 32768);
constexpr unsigned long long DEADLINE = 30000000ull;

typedef int v8i __attribute__((ext_vector_type(8)));
typedef int v4i __attribute__((ext_vector_type(4)));
typedef unsigned u32x4 __attribute__((ext_vector_type(4)));
typedef unsigned u32x2 __attribute__((ext_vector_type(2)));
typedef unsigned u32x3 __attribute__((ext_vector_type(3)));
typedef float f32x4 __attribute__((ext_vector_type(4)));
#define LAS __attribute__((address_space(3)))
#define WAIT_V(n) asm volatile("s_waitcnt vmcnt(%0)" ::"i"(n) : "memory")
#define WAIT_L(n) asm volatile("s_waitcnt lgkmcnt(%0)" ::"i"(n) : "memory")
#define BAR __builtin_amdgcn_s_barrier()
#define SCHED __builtin_amdgcn_sched_barrier(0)

struct Params {
  const unsigned char* x;
  const unsigned char* w;
  const float* sx;
  const float* sw;
  float* out;
  char* const* ws_all;
  char* ws;
  int rank;
  int pad;
};

#define STAMP(ws, idx) do {} while (0)
#define DBG(x)

__device__ __forceinline__ void st16_sys(void* p, u32x4 v) { asm volatile("global_store_dwordx4 %0, %1, off sc0 sc1\n\ts_nop 1" ::"v"(p), "v"(v) : "memory"); }
__device__ __forceinline__ void st4_sys(void* p, unsigned v) { asm volatile("global_store_dword %0, %1, off sc0 sc1" ::"v"(p), "v"(v) : "memory"); }
__device__ __forceinline__ unsigned ld4_sys(const void* p) {
  unsigned v; asm volatile("global_load_dword %0, %1, off sc0 sc1\n\ts_waitcnt vmcnt(0)" : "=&v"(v) : "v"(p) : "memory"); return v;
}
__device__ __forceinline__ void ticket_issue(unsigned& t, void* p, unsigned long long lanes) {
  unsigned long long keep;
  asm volatile("s_mov_b64 %1, exec\n\ts_mov_b64 exec, %4\n\tglobal_atomic_add %0, %2, %3, off sc0\n\ts_mov_b64 exec, %1"
               : "+v"(t), "=&s"(keep) : "v"(p), "v"(1u), "s"(lanes) : "memory");
}
__device__ __forceinline__ void give_up(char* ws, unsigned call) { st4_sys(ws + OFF_TMO + (threadIdx.x & 31) * 4, call); WAIT_V(0); }
__device__ __forceinline__ bool spin_fail(char* ws, unsigned call, unsigned long long t0) {
  if ((unsigned)__builtin_amdgcn_readfirstlane((int)ld4_sys(ws + OFF_TMO)) == call) return true;
  if (__builtin_amdgcn_s_memrealtime() - t0 > DEADLINE) { give_up(ws, call); return true; }
  return false;
}
__device__ __forceinline__ int gstep_off(int step, unsigned par) { return (step < PSTEPS ? (int)par * PSTEPS + step : step + PSTEPS) * STEPB; }

typedef __amdgpu_buffer_rsrc_t rsrc_t;
#define BST16(v, rs, voff, soff, imm) asm volatile("buffer_store_dwordx4 %0, %1, %2, %3 offen offset:%4 sc0 sc1\n\ts_nop 1" ::"v"(v), "v"(voff), "s"(rs), "s"(soff), "i"(imm) : "memory")
#define BLD16(v, rs, voff, soff, imm) asm volatile("buffer_load_dwordx4 %0, %1, %2, %3 offen offset:%4 sc0 sc1" : "=v"(v) : "v"(voff), "s"(rs), "s"(soff), "i"(imm) : "memory")
__device__ __forceinline__ void store_unit(const u32x4 (&r)[8], int n, rsrc_t rs, int voff, int soff) {
#pragma unroll
  for (int i = 0; i < 8; ++i)
    if (i < n) BST16(r[i], rs, voff, soff + (i >> 2) * 4096, (i & 3) * 1024);
}
__device__ __forceinline__ int slab_off(int tensor, int gstep_bytes, int slot, int g) { return (tensor ? OFF_GW : OFF_GX) + gstep_bytes + slot * SLAB + g * 8192; }
__device__ __forceinline__ int uflag_off(int tensor, int step, int slot, int g) { return OFF_UF + (((slot * 2 + tensor) * 8 + g) * 32 + step) * 4; }

__device__ __forceinline__ void prep_load(const Params& P, int c, int tensor, int g, int lane, u32x4 (&r)[8]) {
  if (tensor == 0) {
    auto rs = __builtin_amdgcn_make_buffer_rsrc((void*)P.x, 0, DIM * KSH, 0x00020000);
#pragma unroll
    for (int i = 0; i < 8; ++i) r[i] = __builtin_amdgcn_raw_buffer_load_b128(rs, lane * KSH, (g * 512 + i * 64) * KSH + c * 16, 0);
  } else {
    auto rs = __builtin_amdgcn_make_buffer_rsrc((void*)P.w, 0, DIM * KSH, 0x00020000);
#pragma unroll
    for (int k = 0; k < 16; ++k)
#pragma unroll
      for (int h = 0; h < 2; ++h) r[4 * h + (k >> 2)][k & 3] = __builtin_amdgcn_raw_buffer_load_b32(rs, lane * 4 + h * 256, (c * 16 + k) * DIM + g * 512, 0);
  }
}
__device__ __forceinline__ void prep_transpose(u32x4 (&r)[8]) {
#pragma unroll
  for (int h = 0; h < 2; ++h) {
    u32x4 o[4];
#pragma unroll
    for (int q = 0; q < 4; ++q) {
      const u32x4 a = r[4 * h + q];
      const unsigned l01 = __builtin_amdgcn_perm(a[1], a[0], 0x05010400), h01 = __builtin_amdgcn_perm(a[1], a[0], 0x07030602);
      const unsigned l23 = __builtin_amdgcn_perm(a[3], a[2], 0x05010400), h23 = __builtin_amdgcn_perm(a[3], a[2], 0x07030602);
      o[0][q] = __builtin_amdgcn_perm(l23, l01, 0x05040100);
      o[1][q] = __builtin_amdgcn_perm(l23, l01, 0x07060302);
      o[2][q] = __builtin_amdgcn_perm(h23, h01, 0x05040100);
      o[3][q] = __builtin_amdgcn_perm(h23, h01, 0x07060302);
    }
#pragma unroll
    for (int q = 0; q < 4; ++q) r[4 * h + q] = o[q];
  }
}

constexpr int NCLS = 12, NQUAD = 256 + 4096;
struct QuadTab { unsigned short q[NQUAD]; };
constexpr QuadTab make_quads() {
  constexpr int cost[NCLS] = {86, 33, 27, 37, 51, 67, 83, 99, 114, 130, 146, 162};
  int hist[4 * 162 + 2] = {};
  for (int a = 0; a < NCLS; ++a) for (int b = 0; b < NCLS; ++b) for (int c = 0; c < NCLS; ++c) for (int d = 0; d < NCLS; ++d) ++hist[cost[a] + cost[b] + cost[c] + cost[d]];
  int t1 = 0, t2 = 0, n = 0;
  while (n + hist[t1] <= 256) n += hist[t1++];
  int tie1 = 256 - n;
  n = 0;
  while (n + hist[t2] <= NQUAD) n += hist[t2++];
  int tie2 = NQUAD - n, n1 = 0, n2 = 256;
  QuadTab t{};
  for (int a = 0; a < NCLS; ++a) for (int b = 0; b < NCLS; ++b) for (int c = 0; c < NCLS; ++c) for (int d = 0; d < NCLS; ++d) {
    const int k = cost[a] + cost[b] + cost[c] + cost[d];
    if (k < t1 || (k == t1 && tie1-- > 0)) t.q[n1++] = (unsigned short)(a | b << 4 | c << 8 | d << 12);
    else if (k < t2 || (k == t2 && tie2-- > 0)) t.q[n2++] = (unsigned short)(a | b << 4 | c << 8 | d << 12);
  }
  return t;
}
constexpr unsigned spread4(unsigned q) { return (q & 15u) | (q & 0xf0u) << 4 | (q & 0xf00u) << 8 | (q & 0xf000u) << 12; }
constexpr QuadTab QUADS = make_quads();
constexpr unsigned QUAD0 = spread4(QUADS.q[0]);
constexpr int ENC_KB = (2 * NCLS * NCLS * NCLS * NCLS + 1023) / 1024, DEC_KB = 34;
struct alignas(16) Tabs { unsigned short enc[ENC_KB * 512]; unsigned dec[DEC_KB * 256]; };
constexpr Tabs make_tabs() {
  Tabs t{};
  for (int i = 0; i < ENC_KB * 512; ++i) t.enc[i] = 0x8000;
  for (int i = 0; i < NQUAD; ++i) {
    const int q = QUADS.q[i];
    t.enc[(q & 15) + NCLS * ((q >> 4 & 15) + NCLS * ((q >> 8 & 15) + NCLS * (q >> 12)))] = (unsigned short)(i < 256 ? i : ((i - 256) >> 4) | ((i & 15) << 8) | 0x1000);
    const int a = i < 256 ? 17 * i : 17 * ((i - 256) >> 4) + 1 + (i & 15);
    t.dec[a] = 0u - (spread4(q) << 3);
    t.dec[17 * 256 + a] = 0u - (spread4(q) << 2);
  }
  return t;
}
__constant__ const Tabs TABS = make_tabs();
constexpr int PK_SIDE = 6144;
constexpr int pool_dwords(int tensor) { return tensor ? 160 : 144; }
__device__ __forceinline__ void tab_copy(const void* src, char* lds, int kb, int w, int nw, int lane) {
  const auto rs = __builtin_amdgcn_make_buffer_rsrc((void*)src, 0, kb * 1024, 0x00020000);
  for (int k = w; k < kb; k += nw) __builtin_amdgcn_raw_ptr_buffer_load_lds(rs, (LAS void*)(lds + k * 1024), 16, lane * 16, k * 1024, 0, 0);
}
__device__ __forceinline__ unsigned wave_sum(unsigned v) {
  v += (unsigned)__builtin_amdgcn_update_dpp(0, (int)v, 0xb1, 0xf, 0xf, true);
  v += (unsigned)__builtin_amdgcn_update_dpp(0, (int)v, 0x4e, 0xf, 0xf, true);
  v += (unsigned)__builtin_amdgcn_update_dpp(0, (int)v, 0x141, 0xf, 0xf, true);
  v += (unsigned)__builtin_amdgcn_update_dpp(0, (int)v, 0x140, 0xf, 0xf, true);
  return (unsigned)(__builtin_amdgcn_readlane((int)v, 0) + __builtin_amdgcn_readlane((int)v, 16) + __builtin_amdgcn_readlane((int)v, 32) + __builtin_amdgcn_readlane((int)v, 48));
}
__device__ __forceinline__ unsigned wave_scan(unsigned v) {
  v += (unsigned)__builtin_amdgcn_update_dpp(0, (int)v, 0x111, 0xf, 0xf, true);
  v += (unsigned)__builtin_amdgcn_update_dpp(0, (int)v, 0x112, 0xf, 0xf, true);
  v += (unsigned)__builtin_amdgcn_update_dpp(0, (int)v, 0x114, 0xf, 0xf, true);
  v += (unsigned)__builtin_amdgcn_update_dpp(0, (int)v, 0x118, 0xf, 0xf, true);
  v += (unsigned)__builtin_amdgcn_update_dpp(0, (int)v, 0x142, 0xa, 0xf, false);
  v += (unsigned)__builtin_amdgcn_update_dpp(0, (int)v, 0x143, 0xc, 0xf, false);
  return v;
}

__device__ __forceinline__ unsigned shl_add(unsigned a, int n, unsigned b) { unsigned r; asm("v_lshl_add_u32 %0, %1, %2, %3" : "=v"(r) : "v"(a), "i"(n), "v"(b)); return r; }
__device__ __forceinline__ unsigned shl_or(unsigned a, int n, unsigned b) { unsigned r; asm("v_lshl_or_b32 %0, %1, %2, %3" : "=v"(r) : "v"(a), "i"(n), "v"(b)); return r; }
template <int TENSOR>
__device__ __forceinline__ bool pack_unit(u32x4 (&r)[8], LAS char* lds, const LAS char* enc, int lane) {
  constexpr unsigned ONE = 0x01010101u;
  constexpr int SH = TENSOR ? 2 : 3;
  constexpr unsigned LM = ((1u << SH) - 1) * ONE, CM = (0x7fu >> SH) * ONE;
  unsigned S = 0x80808080u, SL = 0x80808080u | LM, LR = LM;
  asm("" : "+s"(S), "+s"(SL), "+s"(LR));
  LAS unsigned* side = (LAS unsigned*)(lds + 8192);
  unsigned sum = 0;
#pragma unroll
  for (int i = 0; i < 8; ++i) {
    *(LAS u32x4*)(lds + i * 1024 + lane * 16) = r[i];
#pragma unroll
    for (int d = 0; d < 4; ++d) sum = __builtin_amdgcn_sad_u8(r[i][d] & 0x7f7f7f7fu, 0u, sum);
  }
  *(LAS u32x4*)(lds + 8192 + lane * 16) = (u32x4){0u, 0u, 0u, 0u};
  int top = ((int)wave_sum(sum) + (TENSOR ? 101581 : 204800)) >> 13;
  top = top > 127 ? 127 : top;
  const unsigned T = ((unsigned)top | 0x80u) * ONE;
  unsigned o[24], nib[4], fl = 0, miss = 0;
#pragma unroll
  for (int sg = 0; sg < 4; ++sg) {
    unsigned u[8], e[8];
#pragma unroll
    for (int k = 0; k < 8; ++k) {
      u[k] = T - (r[2 * sg + (k >> 2)][k & 3] & 0x7f7f7f7fu);
      const unsigned cb = (u[k] >> SH) & CM;
      const bool ok = __builtin_amdgcn_bitop3_b32(u[k], u[k] + (128 - (NCLS << SH)) * ONE, S, 0x80) == 0x80808080u;
      const unsigned a = __umul24(__builtin_amdgcn_udot4(cb, (2u | 2u * NCLS << 8) << 16, 0u, false), NCLS * NCLS) + __builtin_amdgcn_udot4(cb, 2u | 2u * NCLS << 8, 0u, false);
      e[k] = *(const LAS unsigned short*)(enc + (ok ? a : 2u * NCLS * NCLS * NCLS * NCLS));
    }
    unsigned cnt = 0, y[8];
    nib[sg] = 0;
#pragma unroll
    for (int k = 0; k < 8; ++k) {
      const int j = 8 * sg + k;
      const unsigned v = r[2 * sg + (k >> 2)][k & 3], f = __builtin_amdgcn_ubfe(e[k], 12, 1);
      fl = shl_or(f, j, fl);
      nib[sg] |= __builtin_amdgcn_ubfe(e[k], 8, 4) << cnt;
      cnt = shl_add(f, 2, cnt);
      miss = shl_or(e[k] >> 15, j, miss);
      const unsigned g = __builtin_amdgcn_bitop3_b32(u[k], v, LR, 0x82);
      y[k] = __builtin_amdgcn_bitop3_b32(v, e[k] >= 0x8000u ? 0u : g, SL, 0x28);
    }
#pragma unroll
    for (int k = 0; k < 4; ++k) {
      const unsigned pr = __builtin_amdgcn_perm(e[2 * k + 1], e[2 * k], 0x0c0c0400);
      unsigned& cw = o[(TENSOR ? 12 : 16) + 4 * (sg & 1) + k];
      cw = sg < 2 ? pr : shl_or(pr, 16, cw);
    }
    if (TENSOR == 0) {
#pragma unroll
      for (int k = 0; k < 4; ++k) o[4 * sg + k] = y[2 * k] | __builtin_amdgcn_alignbit(y[2 * k + 1], y[2 * k + 1], 28);
    } else {
#pragma unroll
      for (int k = 0; k < 3; ++k) o[3 * sg + k] = y[k] | __builtin_amdgcn_alignbit(y[3 + k], y[3 + k], 29);
      o[3 * sg] |= (y[6] & LM) << 5; o[3 * sg + 1] |= (y[7] & LM) << 5;
      o[3 * sg + 2] |= ((y[6] >> 2) & 0x20202020u) | ((y[7] >> 1) & 0x40404040u);
    }
  }
#pragma unroll
  for (int k = 0; k < (TENSOR ? 20 : 24); ++k) r[k >> 2][k & 3] = o[k];
  const unsigned nn = __builtin_popcount(fl), incl = wave_scan(nn | __builtin_popcount(miss) << 16);
  const unsigned tot = (unsigned)__builtin_amdgcn_readlane((int)incl, 63);
  const unsigned nnib = tot & 0xffffu, nexc = tot >> 16, ex0 = 65 + ((nnib + 7) >> 3);
  if (nexc > 64 || ex0 + nexc > 64 + pool_dwords(TENSOR)) return true;
  side[lane] = fl;
  if (lane == 0) side[64] = (unsigned)top | nexc << 8 | nnib << 16;
  unsigned pos = (incl & 0xffffu) - nn;
#pragma unroll
  for (int sg = 0; sg < 4; ++sg) {
    const unsigned long long w = (unsigned long long)nib[sg] << ((pos & 7u) * 4);
    __hip_atomic_fetch_or(side + 65 + (pos >> 3), (unsigned)w, __ATOMIC_RELAXED, __HIP_MEMORY_SCOPE_WORKGROUP);
    __hip_atomic_fetch_or(side + 66 + (pos >> 3), (unsigned)(w >> 32), __ATOMIC_RELAXED, __HIP_MEMORY_SCOPE_WORKGROUP);
    pos += __builtin_popcount((fl >> (8 * sg)) & 0xffu);
  }
  asm volatile("" ::: "memory");
  unsigned at = ex0 + (incl >> 16) - __builtin_popcount(miss);
  while (miss) {
    const unsigned j = __builtin_ctz(miss), q = (j >> 2) * 256 + lane * 4 + (j & 3);
    miss &= miss - 1;
    const unsigned h = (*(const LAS unsigned*)(lds + 4 * q) >> SH) & CM;
    constexpr int W = 7 - SH;
    side[at++] = q | ((h & 0xffu) | (h >> 8 & 0xffu) << W | (h >> 16 & 0xffu) << 2 * W | (h >> 24) << 3 * W) << 11;
  }
  return false;
}

template <int TENSOR, typename F>
__device__ __forceinline__ void unpack_unit(u32x4 (&r)[8], unsigned fl, unsigned hv, LAS char* sp, const LAS char* dec, int lane, F&& mid) {
  constexpr unsigned ONE = 0x01010101u;
  constexpr int SH = TENSOR ? 2 : 3, W = 7 - SH;
  constexpr unsigned LM = ((1u << SH) - 1) * ONE, SL = 0x80808080u | LM;
  const LAS unsigned* side = (const LAS unsigned*)(sp + 8192);
  const unsigned nn = __builtin_popcount(fl);
  unsigned pos = wave_scan(nn) - nn, w[4], w1[4], sh[4];
#pragma unroll
  for (int sg = 0; sg < 4; ++sg) {
    w[sg] = side[65 + (pos >> 3)]; w1[sg] = side[66 + (pos >> 3)]; sh[sg] = pos << 2;
    pos += __builtin_popcount((fl >> (8 * sg)) & 0xffu);
  }
  const unsigned hdr = (unsigned)__builtin_amdgcn_readfirstlane((int)hv);
  const unsigned T = (hdr & 0xffu) * ONE, nexc = hdr >> 8 & 0xffu;
  const unsigned ent = side[65 + (((hdr >> 16) + 7) >> 3) + lane];
  SCHED;
  unsigned q[24], t[32], c[32];
#pragma unroll
  for (int k = 0; k < 24; ++k) q[k] = r[k >> 2][k & 3];
#pragma unroll
  for (int j = 0; j < 32; ++j) {
    if (TENSOR == 0) t[j] = ((j & 1) ? __builtin_amdgcn_alignbit(q[j >> 1], q[j >> 1], 4) : q[j >> 1]) & SL;
    else {
      const int k = j & 7, o = 3 * (j >> 3);
      t[j] = k < 3 ? q[o + k] & SL : k < 6 ? __builtin_amdgcn_alignbit(q[o + k - 3], q[o + k - 3], 3) & SL
                                           : ((q[o + k - 6] >> 5) & LM) | ((q[o + 2] << (8 - k)) & 0x80808080u);
    }
  }
  SCHED;
#pragma unroll
  for (int sg = 0; sg < 4; ++sg) w[sg] = __builtin_amdgcn_alignbit(w1[sg], w[sg], sh[sg]);
  unsigned k68 = 0x00440044u, sel0 = 0x0c020c00u, sel1 = 0x0c030c01u;
  asm("" : "+s"(k68), "+s"(sel0), "+s"(sel1));
  unsigned wp;
#pragma unroll
  for (int j = 0; j < 16; ++j) {
    if ((j & 3) == 0) {
      if (j & 4) { w[j >> 3] >>= 4 * __builtin_popcount(fl & (15u << (j - 4))); w[2 + (j >> 3)] >>= 4 * __builtin_popcount(fl & (15u << (j + 12))); }
      wp = __builtin_amdgcn_perm(w[2 + (j >> 3)], w[j >> 3], 0x05040100);
    }
    const unsigned wd = (j >= 2 ? fl >> (j - 2) : fl << (2 - j)) & 0x00040004u;
    unsigned ad;
    asm("v_pk_mad_u16 %0, %1, %2, %2" : "=v"(ad) : "v"(wp & 0x000f000fu), "v"(wd));
    asm("v_pk_mad_u16 %0, %1, %2, %0" : "+v"(ad) : "v"(__builtin_amdgcn_perm(0u, q[(TENSOR ? 12 : 16) + (j >> 1)], (j & 1) ? sel1 : sel0)), "s"(k68));
    asm("v_pk_lshrrev_b16 %0, %1, %0" : "+v"(wp) : "v"(wd));
    c[j] = *(const LAS unsigned*)(dec + TENSOR * 17408 + (ad & 0xffffu));
    c[j + 16] = *(const LAS unsigned*)(dec + TENSOR * 17408 + (ad >> 16));
  }
  SCHED; mid(); SCHED;
#pragma unroll
  for (int i = 0; i < 8; ++i) {
#pragma unroll
    for (int d = 0; d < 4; ++d) r[i][d] = t[4 * i + d] + c[4 * i + d] + (T - LM);
    *(LAS u32x4*)(sp + i * 1024 + lane * 16) = r[i];
  }
  if ((unsigned)lane < nexc) {
    const unsigned e = ent >> 11, F7 = (1u << W) - 1;
    const unsigned hi = ((e & F7) | (e >> W & F7) << 8 | (e >> 2 * W & F7) << 16 | (e >> 3 * W) << 24) << SH;
    __hip_atomic_fetch_add((LAS unsigned*)(sp + 4 * (ent & 0x7ffu)), hi + LM + (QUAD0 << SH) - T, __ATOMIC_RELAXED, __HIP_MEMORY_SCOPE_WORKGROUP);
  }
  asm volatile("" ::: "memory");
#pragma unroll
  for (int i = 0; i < 8; ++i) r[i] = *(const LAS u32x4*)(sp + i * 1024 + lane * 16);
}

__device__ __forceinline__ void post_peers(const Params& P, int word, int lane, unsigned call, unsigned bits) {
  if (lane < NR - 1) {
    const unsigned p = (unsigned)(P.rank + 1 + lane) & (NR - 1);
    const u32x2 v = {call, bits};
    asm volatile("global_store_dwordx2 %0, %1, off sc0 sc1" ::"v"(P.ws_all[p] + OFF_PF + P.rank * 512 + word * 8), "v"(v) : "memory");
  }
}
__device__ __forceinline__ rsrc_t peer_rsrc(const Params& P, unsigned p) {
  const unsigned long long pa = (unsigned long long)P.ws_all[p & (NR - 1)];
  const unsigned long long pu = ((unsigned long long)(unsigned)__builtin_amdgcn_readfirstlane((int)(pa >> 32)) << 32) | (unsigned)__builtin_amdgcn_readfirstlane((int)pa);
  return __builtin_amdgcn_make_buffer_rsrc((void*)pu, 0, WS_NEED, 0x00020000);
}
#define PUSHWIN 13
__device__ __forceinline__ void prep_push(const Params& P, int c, int tensor, int g, int lane, unsigned call, const u32x4 (&r)[8]) {
  rsrc_t rp[NR - 1];
#pragma unroll
  for (int d = 0; d < NR - 1; ++d) rp[d] = peer_rsrc(P, (unsigned)(P.rank + 1 + d));
  asm volatile("s_nop 4");
#pragma unroll
  for (int i = 0; i < 8; ++i)
#pragma unroll
    for (int d = 0; d < NR - 1; ++d) {
      const int ci = c * 7 + 6 - d;
      BST16(r[i], rp[d], lane * 16, slab_off(tensor, gstep_off(4 + (ci >> 3), call & 1), ci & 7, g) + (i >> 2) * 4096, (i & 3) * 1024);
      WAIT_V(PUSHWIN);
    }
  const rsrc_t rs = __builtin_amdgcn_make_buffer_rsrc((void*)P.ws, 0, WS_NEED, 0x00020000);
  store_unit(r, 8, rs, lane * 16, slab_off(tensor, gstep_off(0, call & 1), c, g));
  WAIT_V(0);
  const int ci = c * 7 + 6 - lane;
  if (lane < NR - 1) st4_sys(P.ws_all[(unsigned)(P.rank + 1 + lane) & (NR - 1)] + uflag_off(tensor, 4 + (ci >> 3), ci & 7, g), call);
  if (lane == NR - 1) st4_sys(P.ws + uflag_off(tensor, 0, c, g), call);
}
__device__ __forceinline__ void prep_store(const Params& P, LAS char* lds, const LAS char* enc, int c, int tensor, int g, int lane, unsigned call, u32x4 (&r)[8]) {
  const rsrc_t rs = __builtin_amdgcn_make_buffer_rsrc((void*)P.ws, 0, WS_NEED, 0x00020000);
  int pk = OFF_PK + (int)(call & 1) * (UPP * 8192) + (c * 16 + tensor * 8 + g) * 8192;
  int raw = slab_off(tensor, gstep_off(c >> 3, call & 1), c & 7, g);
  asm volatile("" : "+s"(pk), "+s"(raw));
  const bool ovf = tensor ? pack_unit<1>(r, lds, enc, lane) : pack_unit<0>(r, lds, enc, lane);
  STAMP(P.ws, 30);
#define PDELAY 24
  if (c >= 4) __builtin_amdgcn_s_sleep(PDELAY);
  if (!ovf) {
    store_unit(r, tensor ? 5 : 6, rs, lane * 16, pk);
    const u32x4 sd = *(const LAS u32x4*)(lds + 8192 + lane * 16);
    BST16(sd, rs, lane * 16, pk + PK_SIDE, 0);
  }
#pragma unroll
  for (int i = 0; i < 8; ++i) r[i] = *(const LAS u32x4*)(lds + i * 1024 + lane * 16);
  if (c < 4 && !ovf) { WAIT_V(0); post_peers(P, (c - 2) * 16 + tensor * 8 + g, lane, call, 0u); }
  store_unit(r, 8, rs, lane * 16, raw);
  unsigned* pc = (unsigned*)(P.ws + OFF_PCNT + (c >> 1) * 128);
  if (ovf && c >= 4 && lane == 0) __hip_atomic_fetch_or(pc + 1 + (call & 1), 1u << ((c & 1) * 16 + tensor * 8 + g), __ATOMIC_RELAXED, __HIP_MEMORY_SCOPE_AGENT);
  WAIT_V(0);
  if (lane == 0) st4_sys(P.ws + uflag_off(tensor, c >> 3, c & 7, g), call);
  if (c < 4) { if (ovf) post_peers(P, (c - 2) * 16 + tensor * 8 + g, lane, call, ~0u); return; }
  unsigned tk = 0;
  ticket_issue(tk, pc, 1);
  WAIT_V(0); asm volatile("" : "+v"(tk));
  if ((unsigned)__builtin_amdgcn_readfirstlane((int)tk) == call * 32 - 1) {
    post_peers(P, 30 + (c >> 1), lane, call, ld4_sys(pc + 1 + (call & 1)));
    if (lane == 0) st4_sys(pc + 1 + (~call & 1), 0u);
  }
}

#define RAMP 12
#define LEADX 0, 0, 0, 0, 0, 0, 0, 0
#define LEADW 0, 0, 0, 0, 0, 0, 0, 0
struct Order { int e[NPULL]; };
constexpr Order make_order() {
  constexpr int lead[16] = {LEADX, LEADW};
  Order o{};
  int next[16] = {};
  for (int t = 0; t < 16; ++t) next[t] = RAWC;
  for (int k = 0; k < NPULL; ++k) {
    int best = -1, key = 0;
    for (int v = 0; v < 16; ++v) {
      const int t = (v >> 1) + 8 * (v & 1);
      const int kt = 4 * RAMP * next[t] - lead[t] * (next[t] - RAWC < RAMP ? next[t] - RAWC : RAMP) +
                     (t >= 4 && t < 8 && 7 * next[t] >= 8 * (JSTEP - 4) ? 1 << 20 : 0);
      if (next[t] < 32 && (best < 0 || kt < key)) { best = t; key = kt; }
    }
    o.e[k] = next[best] * 16 + best;
    ++next[best];
  }
  return o;
}
__constant__ const Order ORDER = make_order();

constexpr int NSLOT = 6, SLOTB = 8192 + 1024;
constexpr int RING = DEC_KB * 1024, RING_CTL = RING + 2 * NSLOT * SLOTB;
static_assert(3 * 9216 + ENC_KB * 1024 <= LDS_BYTES, "encoder table");
static_assert(WIN >= 3 && WIN <= 7 && RING % 16 == 0 && RING_CTL + 512 <= LDS_BYTES && LPP * 7 <= 2 * NCOMM, "ring");
__device__ __forceinline__ unsigned lds_ld(unsigned a) {
  unsigned v; asm volatile("ds_read_b32 %0, %1\n\ts_waitcnt lgkmcnt(0)" : "=v"(v) : "v"(a) : "memory");
  return (unsigned)__builtin_amdgcn_readfirstlane((int)v);
}
__device__ __forceinline__ void lds_st(unsigned a, unsigned v) { asm volatile("ds_write_b32 %0, %1" ::"v"(a), "v"(v) : "memory"); }

__device__ __forceinline__ void loader_main(const Params& P, char* shm, int l, int lid, int lane, unsigned call) {
  const unsigned ctl = (unsigned)(size_t)(LAS char*)shm + RING_CTL + l * 256;
  if (lid >= LPP * 7) { lds_st(ctl + 4, 0); return; }
  const int q = lid % 7, li = lid / 7;
  const unsigned p = (unsigned)(P.rank + 1 + q) & (NR - 1);
  const rsrc_t rp = peer_rsrc(P, p);
  const char* pfp = P.ws + OFF_PF + p * 512 + lane * 8;
  unsigned* claim = (unsigned*)(P.ws + OFF_CLAIM + q * 128);
  const unsigned cbase = (call - 1) * NPULL - 2 * LPP;
  const int pkb = OFF_PK + (int)(call & 1) * (UPP * 8192);
  char* ring = shm + RING + l * NSLOT * SLOTB;
  unsigned long long ready = 0;
  u32x2 pf = {0u, 0u};
  bool dead = false;
  int n = 0, slot = 0, src, nl;
  auto setup = [&](int e, int un, int sl, int landed) {
    const int w = e < 64 ? e - 32 : 30 + (e >> 5);
    if (!dead && !((ready >> w) & 1)) {
      WAIT_V(0); lds_st(ctl, (unsigned)landed);
      const unsigned long long t0 = __builtin_amdgcn_s_memrealtime();
      while (true) {
        asm volatile("global_load_dwordx2 %0, %1, off sc0 sc1\n\ts_waitcnt vmcnt(0)" : "=&v"(pf) : "v"(pfp) : "memory");
        ready = __ballot(pf.x >= call);
        if ((ready >> w) & 1) break;
        if (spin_fail(P.ws, call, t0)) { dead = true; break; }
        __builtin_amdgcn_s_sleep(1);
      }
    }
    if (un >= NSLOT && (int)lds_ld(ctl + 76 + sl * 16) <= un - NSLOT) {
      const unsigned long long t0 = __builtin_amdgcn_s_memrealtime();
      while ((int)lds_ld(ctl + 76 + sl * 16) <= un - NSLOT && __builtin_amdgcn_s_memrealtime() - t0 < 2 * DEADLINE) {}
    }
    const int c = e >> 4, tensor = (e >> 3) & 1, g = e & 7, ci = c * 7 + q, step = 4 + (ci >> 3);
    const bool raw = ((unsigned)__builtin_amdgcn_readlane((int)pf.y, w) >> (e & 31)) & 1u;
    nl = raw ? 8 : 6 - tensor;
    src = __builtin_amdgcn_readfirstlane(raw ? slab_off(tensor, gstep_off(c >> 3, call & 1), c & 7, g) : pkb + e * 8192);
    const u32x3 dsc = {(unsigned)nl, (unsigned)slab_off(tensor, gstep_off(step, call & 1), ci & 7, g), (unsigned)uflag_off(tensor, step, ci & 7, g)};
    if (lane == 0) asm volatile("ds_write_b96 %0, %1" ::"v"(ctl + 64 + sl * 16), "v"(dsc) : "memory");
  };
  unsigned tk = 0;
  int e = ORDER.e[li];
  setup(e, 0, 0, 0);
  STAMP(P.ws, 2);
  while (true) {
    const int s0 = src, n0 = nl;
    char* lp = ring + slot * SLOTB;
#define PLANE(i) do { WAIT_V(WIN - 1); if ((i) == WIN - 3) lds_st(ctl, (unsigned)n);   \
      __builtin_amdgcn_raw_ptr_buffer_load_lds(rp, (LAS void*)(lp + ((i) >> 2) * 4096), 16, lane * 16, s0 + ((i) >> 2) * 4096, ((i) & 3) * 1024, 17); } while (0)
    ticket_issue(tk, claim, 1ull << (n & 1));
    if (lane < (64 + pool_dwords(1)) / 4 - (n0 == 6 ? (pool_dwords(1) - pool_dwords(0)) / 4 : 0))
      __builtin_amdgcn_raw_ptr_buffer_load_lds(rp, (LAS void*)(lp + 8192), 16, lane * 16, pkb + e * 8192 + PK_SIDE, 0, 17);
    PLANE(0); PLANE(1);
    asm volatile("" : "+v"(tk));
    const int kn = n == 0 ? li + LPP : (int)((unsigned)__builtin_amdgcn_readlane((int)tk, ~n & 1) - cbase);
    const bool more = kn < NPULL;
    e = ORDER.e[more ? kn : 0];
    PLANE(2); PLANE(3); PLANE(4);
    if (n0 > 5) PLANE(5);
    if (n0 > 6) { PLANE(6); PLANE(7); }
    ++n; slot = slot == NSLOT - 1 ? 0 : slot + 1;
    if (!more) break;
    setup(e, n, slot, n);
  }
  WAIT_V(0);
  asm volatile("" : "+v"(tk));
  lds_st(ctl, (unsigned)n);
  lds_st(ctl + 4, (unsigned)n);
  DBG(STAMPV(P.ws, 10, (unsigned long long)n);)
}

__device__ __forceinline__ void decoder_main(const Params& P, char* shm, int l, int lane, unsigned call) {
  const unsigned ctl0 = (unsigned)(size_t)(LAS char*)shm + RING_CTL;
  const rsrc_t rs = __builtin_amdgcn_make_buffer_rsrc((void*)P.ws, 0, WS_NEED, 0x00020000);
  const unsigned long long t0 = __builtin_amdgcn_s_memrealtime();
  DBG(unsigned long long tdec = 0; unsigned long long cntu = 0;)
  int pflag = -1;
  auto post = [&] { if (pflag >= 0) { WAIT_V(0); if (lane == 0) st4_sys(P.ws + pflag, call); pflag = -1; } };
  unsigned tk = 0;
  auto draw = [&](int ring) {
    unsigned long long keep;
    asm volatile("s_mov_b64 %1, exec\n\ts_mov_b64 exec, 1\n\tds_add_rtn_u32 %0, %2, %3 offset:8\n\ts_mov_b64 exec, %1" : "+v"(tk), "=&s"(keep) : "v"(ctl0 + ring * 256), "v"(1u) : "memory");
  };
  while (lds_ld(ctl0 + 12) < 5 && __builtin_amdgcn_s_memrealtime() - t0 < DEADLINE) __builtin_amdgcn_s_sleep(1);
  draw(l);
  bool other = true;
  while (true) {
    WAIT_L(0); asm volatile("" : "+v"(tk));
    const int n = __builtin_amdgcn_readfirstlane((int)tk);
    const unsigned ctl = ctl0 + l * 256;
    bool have = true;
    for (int spin = 0; (int)lds_ld(ctl) <= n; ++spin) {
      post();
      if (lds_ld(ctl + 4) <= (unsigned)n || ((spin & 63) == 63 && __builtin_amdgcn_s_memrealtime() - t0 > 4 * DEADLINE)) { have = false; break; }
      __builtin_amdgcn_s_sleep(1);
    }
    if (!have) {
      if (!other) return;
      other = false; l ^= 1; draw(l);
      continue;
    }
    DBG(unsigned long long ta = NOW();)
    const int slot = n % NSLOT;
    LAS char* sp = (LAS char*)shm + RING + (l * NSLOT + slot) * SLOTB;
    const unsigned fl = *(const LAS unsigned*)(sp + 8192 + lane * 4), hv = *(const LAS unsigned*)(sp + 8192 + 256);
    const u32x4 dsc = *(const LAS u32x4*)((LAS char*)shm + RING_CTL + l * 256 + 64 + slot * 16);
    u32x4 r[8];
#pragma unroll
    for (int i = 0; i < 6; ++i) r[i] = *(const LAS u32x4*)(sp + i * 1024 + lane * 16);
    if (other) l ^= 1;
    draw(l);
    const int nl = __builtin_amdgcn_readfirstlane((int)dsc.x), dst = __builtin_amdgcn_readfirstlane((int)dsc.y), flag = __builtin_amdgcn_readfirstlane((int)dsc.z);
    if (nl == 6) unpack_unit<0>(r, fl, hv, sp, (const LAS char*)shm, lane, post);
    else if (nl == 5) unpack_unit<1>(r, fl, hv, sp, (const LAS char*)shm, lane, post);
    else { r[6] = *(const LAS u32x4*)(sp + 6144 + lane * 16); r[7] = *(const LAS u32x4*)(sp + 7168 + lane * 16); post(); }
    WAIT_L(0);
#pragma unroll
    for (int i = 0; i < 8; ++i) asm volatile("" : "+v"(r[i]));
    lds_st(ctl + 76 + slot * 16, (unsigned)(n + 1));
    asm volatile("s_nop 4");
    store_unit(r, 8, rs, lane * 16, dst);
    pflag = flag;
    DBG(tdec += NOW() - ta; ++cntu; STAMPV(P.ws, 8, tdec); STAMPV(P.ws, 9, 0); STAMPV(P.ws, 10, cntu);)
  }
}

typedef float f32x2 __attribute__((ext_vector_type(2)));
__device__ __forceinline__ f32x2 gate2(f32x2 a, float ke, float ys) {
  constexpr float LIM = 60.f * 1.44269504088896341f, EPS = 0x1p-24f;
  const f32x2 t = a * ke;
  const f32x2 e = {__builtin_amdgcn_exp2f(__builtin_amdgcn_fmed3f(t.x, -LIM, LIM)), __builtin_amdgcn_exp2f(__builtin_amdgcn_fmed3f(t.y, -LIM, LIM))};
  const f32x2 d = e * EPS + EPS;
  const float r = __builtin_amdgcn_rcpf(d.x * d.y) * ys;
  const f32x2 sw = {d.y, d.x};
  return sw * r;
}

#define SPREAD TAILS
constexpr int ufirst(int ph, int nu) { return ph >= 2 * SPREAD ? nu : (ph * nu + 2 * SPREAD - 1) / (2 * SPREAD); }
template <int I, int E, typename F> __device__ __forceinline__ void static_for(F&& f) {
  if constexpr (I < E) { f(std::integral_constant<int, I>{}); static_for<I + 1, E>(f); }
}

template <int MA0, int MA1>
__device__ __forceinline__ void gemm_main(const Params& P, char* shm, int browA, int browB, int bcol, int wid, int lane, unsigned call) {
  constexpr int RA0 = 32 * MA0, RA1 = 32 * MA1;
  constexpr int SLA0 = RA0 * 16, SLA1 = RA1 * 16, SLB = 128 * 16;
  constexpr int A0B = 8 * SLA0, A1B = 8 * SLA1, BB = 8 * SLB, BUFB = A0B + A1B + 2 * BB;
  constexpr int NA0 = (RA0 + 63) / 64, NA1 = (RA1 + 63) / 64;
  constexpr int S = TAILS, J = JSTEP, END = NSTEP + S;
  static_assert(2 * BUFB <= LDS_BYTES && S % 2 == 0 && S >= 2 && J >= 8, "shape");
  const int wr = wid >> 2, wc = wid & 3, fr = lane & 15, fq = lane >> 4;
  const unsigned par = call & 1;
  auto rX = __builtin_amdgcn_make_buffer_rsrc((void*)(P.ws + OFF_GX), 0, GSTEPS * STEPB, 0x00020000);
  auto rW = __builtin_amdgcn_make_buffer_rsrc((void*)(P.ws + OFF_GW), 0, GSTEPS * STEPB, 0x00020000);
  auto rU = __builtin_amdgcn_make_buffer_rsrc((void*)P.ws, 0, OFF_DBG, 0x00020000);
  auto rC = __builtin_amdgcn_make_buffer_rsrc((void*)P.out, 0, DIM * DIM * 4, 0x00020000);
  const int voff = lane * 16;
  const int sA0 = wid * SLAB + browA * 16, sA1 = wid * SLAB + browB * 16, sB = wid * SLAB + bcol * 16;
  constexpr unsigned M0 = 0x13, M1 = 0x0c, MALL = 0x1f, MBAD = 0x20;
  int pollv;
  { const int tg = lane == 0 ? browA >> 9 : lane == 1 ? (browA + RA0 - 1) >> 9 : lane == 2 ? browB >> 9 : lane == 3 ? (browB + RA1 - 1) >> 9 : 8 + (bcol >> 9);
    pollv = lane == 5 ? OFF_TMO : OFF_UF + (wid * 16 + tg) * 128; }
  unsigned pv = 0, got = 0;
  bool dead = false;
  const float scale = P.sx[0] * P.sw[0];
  int kei, ysi, isi;
  asm volatile("s_nop 1\n\tv_readfirstlane_b32 %0, %1" : "=s"(isi) : "v"(1.0f / scale));
  asm volatile("s_nop 1\n\tv_readfirstlane_b32 %0, %2\n\tv_readfirstlane_b32 %1, %3" : "=s"(kei), "=s"(ysi) : "v"(scale * -1.44269504088896341f), "v"(scale * 0x1p-24f));
  int t4 = 0;
#define POLL(step) asm volatile("buffer_load_dword %0, %1, %2, %3 offen offset:%4 sc0 sc1" : "=v"(pv) : "v"(pollv), "s"(rU), "s"(t4), "i"((step) * 4) : "memory")
#define TAKE() do { asm volatile("" : "+v"(pv)); got = (unsigned)__ballot(pv >= call); } while (0)
#define NEED(step, m) do { if ((got & (m)) != (m) && !dead) { const unsigned long long t0_ = __builtin_amdgcn_s_memrealtime(); \
    while (true) { unsigned sv_;     \
      asm volatile("buffer_load_dword %0, %1, %2, %3 offen offset:%4 sc0 sc1\n\ts_waitcnt vmcnt(0)" : "=&v"(sv_) : "v"(pollv), "s"(rU), "s"(t4), "i"((step) * 4) : "memory"); \
      got = (unsigned)__ballot(sv_ >= call); if ((got & (m)) == (m)) break; \
      if (got & MBAD) { dead = true; break; } \
      if (__builtin_amdgcn_s_memrealtime() - t0_ > DEADLINE) { give_up(P.ws, call); dead = true; break; } \
      __builtin_amdgcn_s_sleep(1); } \
    WAIT_L(0); } } while (0)
#define OFF_A0(b) ((b) * BUFB)
#define OFF_A1(b) ((b) * BUFB + A0B)
#define OFF_B(b, h) ((b) * BUFB + A0B + A1B + (h) * BB)
#define GL(rs, ldsoff, soff, imm) __builtin_amdgcn_raw_ptr_buffer_load_lds(rs, (LAS void*)(shm + (ldsoff)), 16, voff, soff, imm, 17)
#define STAGE_A0(b, go) do { const int so_ = sA0 + (go), lo_ = OFF_A0(b) + wid * SLA0; GL(rX, lo_, so_, 0); GL(rX, lo_, so_, 1024); \
    if (NA0 == 3) { if (lane < 32) GL(rX, lo_, so_, 2048); } } while (0)
#define STAGE_A1(b, go) do { const int so_ = sA1 + (go), lo_ = OFF_A1(b) + wid * SLA1; GL(rX, lo_, so_, 0); GL(rX, lo_, so_, 1024); \
    if (NA1 == 3) { if (lane < 32) GL(rX, lo_, so_, 2048); } } while (0)
#define STAGE_B(b, h, go) do { const int so_ = sB + (h) * 2048 + (go), lo_ = OFF_B(b, h) + wid * SLB; GL(rW, lo_, so_, 0); GL(rW, lo_, so_, 1024); } while (0)
  const LAS char* lsm = (const LAS char*)shm;
  const LAS char *pa0[2], *pa1[2], *pb[2];
#pragma unroll
  for (int b = 0; b < 2; ++b) {
    pa0[b] = lsm + OFF_A0(b) + fq * SLA0 + (wr * 16 * MA0 + fr) * 16;
    pa1[b] = lsm + OFF_A1(b) + fq * SLA1 + (wr * 16 * MA1 + fr) * 16;
    pb[b] = lsm + OFF_B(b, 0) + fq * SLB + (wc * 16 + fr) * 16;
    asm volatile("" : "+v"(pa0[b])); asm volatile("" : "+v"(pa1[b])); asm volatile("" : "+v"(pb[b]));
  }
#define LD8(p, sl) __builtin_shufflevector(*(const LAS v4i*)(p), *(const LAS v4i*)((p) + 4 * (sl)), 0, 1, 2, 3, 4, 5, 6, 7)
#define LDA0(b) _Pragma("unroll") for (int m = 0; m < MA0; ++m) At[m] = LD8(pa0[b] + m * 256, SLA0)
#define LDA1(b) _Pragma("unroll") for (int m = 0; m < MA1; ++m) At[m] = LD8(pa1[b] + m * 256, SLA1)
#define LDB(dst, b, h) _Pragma("unroll") for (int n = 0; n < 2; ++n) dst[n] = LD8(pb[b] + (h) * BB + n * 1024, SLB)
#define MFMA(c, a, b) asm volatile("v_mfma_f32_16x16x128_f8f6f4 %0, %1, %2, %0 blgp:1" : "+v"(c) : "v"(a), "v"(b))
#define MMA0(bj, Bf) do { __builtin_amdgcn_s_setprio(1); _Pragma("unroll") for (int m = 0; m < MA0; ++m) _Pragma("unroll") for (int n = 0; n < 2; ++n) \
    MFMA(acc0[m][(bj) * 2 + n], At[m], Bf[n]); __builtin_amdgcn_s_setprio(0); } while (0)
#define MMA1(bj, Bf) do { __builtin_amdgcn_s_setprio(1); _Pragma("unroll") for (int m = 0; m < MA1; ++m) _Pragma("unroll") for (int n = 0; n < 2; ++n) \
    MFMA(acc1[m][(bj) * 2 + n], At[m], Bf[n]); __builtin_amdgcn_s_setprio(0); } while (0)
#define TILE(b, nb, t, r) do { TAKE(); NEED((r) + 1, MALL); const int go_ = gstep_off((t) + 1, par); \
    LDB(B0, b, 0); LDA0(b); STAGE_B(nb, 0, go_); STAGE_A0(nb, go_); POLL((r) + 2); \
    WAIT_V(NA1 + 2 + NA0 + 1); BAR; WAIT_L(0); MMA0(0, B0); BAR; SCHED; \
    LDB(B1, b, 1); STAGE_B(nb, 1, go_); STAGE_A1(nb, go_); \
    WAIT_V(2 + NA0 + 1 + 2 + NA1); BAR; WAIT_L(0); MMA0(1, B1); BAR; SCHED; \
    LDA1(b); BAR; WAIT_L(0); MMA1(0, B0); BAR; SCHED; \
    WAIT_V(2 + NA1); BAR; MMA1(1, B1); BAR; SCHED; } while (0)
#define OUTPOL "sc1"
#define ST_ROW(o, row) do { const int so_ = ((row) * DIM + bcol + wc * 64) * 4; \
    asm volatile("buffer_store_dwordx4 %0, %1, %2, %3 offen " OUTPOL "\n\ts_nop 1" ::"v"(o), "v"(cv), "s"(rC), "s"(so_) : "memory"); } while (0)
#define EARLYPOL OUTPOL
#define ST_ROWP(o, row, POL, vo) do { const int so_ = ((row) * DIM + bcol + wc * 64) * 4; \
    asm volatile("buffer_store_dwordx4 %0, %1, %2, %3 offen " POL "\n\ts_nop 1" ::"v"(o), "v"(vo), "s"(rC), "s"(so_) : "memory"); } while (0)
#define ST_PAIR(acc, j, row0) do { f32x4 o0_, o1_; \
    _Pragma("unroll") for (int u = 0; u < 4; ++u) { const f32x2 a_ = {acc[u][j], acc[u][(j) + 1]}, g_ = gate2(a_, __builtin_bit_cast(float, kei), __builtin_bit_cast(float, ysi)); \
      float p_, q_; asm("v_mul_f32 %0, %1, %2" : "=v"(p_) : "v"(a_.x), "v"(g_.x)); asm("v_mul_f32 %0, %1, %2" : "=v"(q_) : "v"(a_.y), "v"(g_.y)); \
      o0_[u] = p_; o1_[u] = q_; } \
    ST_ROW(o0_, (row0) + (j)); ST_ROW(o1_, (row0) + (j) + 1); } while (0)
  f32x4 acc0[MA0][4] = {}, acc1[MA1][4] = {};
  v8i At[MA0 > MA1 ? MA0 : MA1], B0[2], B1[2];
  NEED(0, MALL);
  STAMP(P.ws, 2);
  { const int go = gstep_off(0, par);
    STAGE_B(0, 0, go); STAGE_A0(0, go); POLL(1); STAGE_B(0, 1, go); STAGE_A1(0, go); }
  if (wr == 1) BAR;
  WAIT_V(2 + NA1); BAR; BAR; SCHED;
  for (int t = 0; t < J - 2; t += 2) {
    asm volatile("s_lshl_b32 %0, %1, 2" : "=s"(t4) : "s"(t) : "scc");
    TILE(0, 1, t, 0);
    TILE(1, 0, t + 1, 1);
  }
  t4 = 0;
  TILE(0, 1, J - 2, J - 2);
  {
    TAKE(); NEED(J, M0);
    LDB(B0, 1, 0); LDA0(1); STAGE_B(0, 0, (J + PSTEPS) * STEPB); STAGE_A0(0, (J + PSTEPS) * STEPB); POLL(J + 1);
    WAIT_V(NA1 + 2 + NA0 + 1); BAR; WAIT_L(0); MMA0(0, B0); BAR; SCHED;
    LDB(B1, 1, 1); STAGE_B(0, 1, (J + PSTEPS) * STEPB);
    WAIT_V(2 + NA0 + 1 + 2); BAR; WAIT_L(0); MMA0(1, B1); BAR; SCHED;
    LDA1(1); BAR; WAIT_L(0); MMA1(0, B0); BAR; SCHED;
    WAIT_V(2); TAKE(); NEED(J + 1, M0); POLL(J + 2); STAGE_B(1, 0, (J + 1 + PSTEPS) * STEPB); STAGE_A0(1, (J + 1 + PSTEPS) * STEPB);
    BAR; MMA1(1, B1); BAR; SCHED; }
  int tz = threadIdx.x; asm volatile("" : "+v"(tz));
  const int cv = (((tz & 63) >> 4) * 4 * DIM + (tz & 15) * 4) * 4;
  float lim = 60.f * 1.44269504088896341f; asm volatile("" : "+v"(lim));
  f32x4 o0, o1;
#define PREOPS 3
  constexpr int VMAX = 2 * ((8 * MA0 + 2 * SPREAD - 1) / (2 * SPREAD));
  float tv[VMAX];
  auto epi_op = [&](auto opc, auto uc, auto vc) {
    constexpr int op = decltype(opc)::value, u = decltype(uc)::value, v = decltype(vc)::value;
    const float a = acc0[u >> 3][u & 3][(u >> 2 & 1) * 2 + (v & 1)];
    float& t = tv[v];
    if constexpr (op == 0) asm volatile("v_mul_f32 %0, %1, %2" : "=v"(t) : "s"(kei), "v"(a));
    if constexpr (op == 1) asm volatile("v_med3_f32 %0, %0, %1, %2" : "+v"(t) : "s"(0xc2ad1f97), "v"(lim));
    if constexpr (op == 2) asm volatile("v_exp_f32 %0, %0" : "+v"(t));
    if constexpr (op == 3) asm volatile("v_fma_f32 %0, %0, %1, %1" : "+v"(t) : "s"(isi));
    if constexpr (op == 4) asm volatile("v_rcp_f32 %0, %0" : "+v"(t));
    if constexpr (op == 5) {
      float r; asm volatile("v_mul_f32 %0, %1, %2" : "=v"(r) : "v"(a), "v"(t));
      if constexpr (v & 1) o1[u & 3] = r; else o0[u & 3] = r;
      if constexpr ((u & 3) == 3 && (v & 1)) {
        const int row = browA + wr * 16 * MA0 + (u >> 3) * 16 + (u >> 2 & 1) * 2;
        ST_ROWP(o0, row, EARLYPOL, cv); ST_ROWP(o1, row + 1, EARLYPOL, cv);
      }
    }
  };
  auto pre = [&](auto phc) {
    constexpr int ph = decltype(phc)::value, u0 = ufirst(ph, 8 * MA0), V = 2 * (ufirst(ph + 1, 8 * MA0) - u0);
    static_for<0, PREOPS * V>([&](auto kc) {
      constexpr int k = decltype(kc)::value;
      epi_op(std::integral_constant<int, k / V>{}, std::integral_constant<int, u0 + (k % V >> 1)>{}, std::integral_constant<int, k % V>{});
    });
  };
  auto cluster = [&](auto phc, auto bjc, v8i (&Bf)[2]) {
    constexpr int ph = decltype(phc)::value, bj = decltype(bjc)::value, G = 2 * MA1;
    constexpr int u0 = ufirst(ph, 8 * MA0), V = 2 * (ufirst(ph + 1, 8 * MA0) - u0);
    constexpr auto cost = [](int k) { const int op = k / (V ? V : 1), v = k % (V ? V : 1);
      return 4 * k + 4 * (op > 2 ? V : op == 2 ? v : 0) + 4 * (op > 4 ? V : op == 4 ? v : 0); };
    __builtin_amdgcn_s_setprio(1);
    static_for<0, G>([&](auto gc) {
      constexpr int g = decltype(gc)::value;
      MFMA(acc1[g >> 1][bj * 2 + (g & 1)], At[g >> 1], Bf[g & 1]);
      static_for<PREOPS * V, 6 * V>([&](auto kc) {
        constexpr int k = decltype(kc)::value;
        if constexpr ((cost(k) - cost(PREOPS * V)) * (G - 1) / (32 * V - cost(PREOPS * V)) == g)
          epi_op(std::integral_constant<int, k / V>{}, std::integral_constant<int, u0 + (k % V >> 1)>{}, std::integral_constant<int, k % V>{});
      });
    });
    __builtin_amdgcn_s_setprio(0);
  };
  unsigned pw[2] = {pv, pv};
  static_for<J, END>([&](auto ic) {
    constexpr int i = decltype(ic)::value, b = i & 1, nb = b ^ 1;
    constexpr bool h = i >= NSTEP;
    constexpr int MA = h ? MA1 : MA0;
    constexpr auto step = [](int k) { return k < NSTEP ? k : k - S; };
    constexpr auto nst = [](int k, int q) {
      const int ph = 2 * (k - NSTEP) + q;
      return ph < 0 ? 0 : 2 * (ufirst(ph + 1, 8 * MA0) / 4 - ufirst(ph, 8 * MA0) / 4); };
    constexpr auto nq1 = [](int k) { return (k + 1 < END ? 2 : 0) + (k + 3 < END ? 1 : 0); };
    constexpr auto nq2 = [](int k) { return k + 2 < END ? 2 + (k + 2 < NSTEP ? NA0 : NA1) : 0; };
    const int g1 = sB + 2048 + (step(i + 1) + PSTEPS) * STEPB, l1 = OFF_B(nb, 1) + wid * SLB;
    if constexpr (i + 1 < END) GL(rW, l1, g1, 0);
    SCHED; LDB(B0, b, 0); SCHED;
    if constexpr (i + 1 < END) GL(rW, l1, g1, 1024);
    SCHED;
    static_for<0, MA>([&](auto mc) {
      constexpr int m = decltype(mc)::value;
      if constexpr (h) At[m] = LD8(pa1[b] + m * 256, SLA1); else At[m] = LD8(pa0[b] + m * 256, SLA0);
      if constexpr (m == 1 && i + 3 < END) {
        SCHED; asm volatile("buffer_load_dword %0, %1, %2, 0 offen offset:%3 sc0 sc1" : "=v"(pw[b]) : "v"(pollv), "s"(rU), "i"(step(i + 3) * 4) : "memory"); SCHED; }
    });
    WAIT_V(nst(i - 1, 0) + nq2(i - 1) + nst(i - 1, 1) + nq1(i));
    asm volatile("" : "+v"(pw[nb])); got = (unsigned)__ballot(pw[nb] >= call);
    if constexpr (i == NSTEP) {
      STAMP(P.ws, 7);
      if (dead || (got & MBAD)) isi = 0x7fc00000;
      asm volatile("s_nop 15\n\ts_nop 15"); SCHED;
    }
    if constexpr (h) pre(std::integral_constant<int, 2 * (i - NSTEP)>{});
    WAIT_L(0); BAR;
    if constexpr (h) cluster(std::integral_constant<int, 2 * (i - NSTEP)>{}, std::integral_constant<int, 0>{}, B0); else MMA0(0, B0);
    BAR; SCHED;
    if constexpr (i + 2 < END) {
      constexpr bool h2 = i + 2 >= NSTEP;
      NEED(step(i + 2), (h2 ? M1 : M0));
      const int go = (step(i + 2) + PSTEPS) * STEPB;
      const int ga = (h2 ? sA1 : sA0) + go, la = h2 ? OFF_A1(b) + wid * SLA1 : OFF_A0(b) + wid * SLA0, gb = sB + go, lb = OFF_B(b, 0) + wid * SLB;
      GL(rX, la, ga, 0); SCHED;
      B1[0] = LD8(pb[b] + BB, SLB); SCHED;
      GL(rX, la, ga, 1024); SCHED;
      B1[1] = LD8(pb[b] + BB + 1024, SLB); SCHED;
      if ((h2 ? NA1 : NA0) == 3) { if (lane < 32) GL(rX, la, ga, 2048); }
      GL(rW, lb, gb, 0); GL(rW, lb, gb, 1024);
    } else {
      LDB(B1, b, 1);
    }
    if constexpr (i + 1 < END) WAIT_V(nst(i - 1, 1) + nq1(i) + nst(i, 0) + nq2(i));
    if constexpr (h) pre(std::integral_constant<int, 2 * (i - NSTEP) + 1>{});
    WAIT_L(0); BAR;
    if constexpr (h) cluster(std::integral_constant<int, 2 * (i - NSTEP) + 1>{}, std::integral_constant<int, 1>{}, B1); else MMA0(1, B1);
    if constexpr (i + 1 < END) { BAR; SCHED; }
  });
  asm volatile("s_nop 15\n\ts_nop 15"); SCHED;
  if (wr == 0) BAR;
  STAMP(P.ws, 3);
  POLL(0); WAIT_V(0); TAKE();
  if (dead || (got & MBAD)) kei = ysi = 0x7fc00000;
#pragma unroll
  for (int m = 0; m < MA1; ++m) { ST_PAIR(acc1[m], 0, browB + wr * 16 * MA1 + m * 16); ST_PAIR(acc1[m], 2, browB + wr * 16 * MA1 + m * 16); }
}

__global__ __launch_bounds__(512, 2) void fused_kernel(Params P) {
  extern __shared__ __attribute__((aligned(16))) char shm[];
  const int b = blockIdx.x;
  const int wid = __builtin_amdgcn_readfirstlane(threadIdx.x >> 6), lane = threadIdx.x & 63;
  STAMP(P.ws, 0);
  unsigned cnt;
  asm volatile("global_load_dword %0, %1, off sc0 sc1" : "=v"(cnt) : "v"(P.ws + OFF_CNT) : "memory");
  int pc = -1;
  const int j = b - NCOMM;
  if (b < NCOMM) { if (wid == 7) pc = b >> 4; }
  else if (wid < 2) pc = RAWC + 2 * (j >> 4) + wid;
  else if (wid == 2 && j < 32) pc = 30 + (j >> 4);
  const int ptensor = (b >> 3) & 1, pg = b & 7;
  constexpr int ENC = 3 * 9216;
  u32x4 r[8];
#define LDELAY 0
  if (LDELAY > 0 && j >= 16) __builtin_amdgcn_s_sleep(LDELAY);
  if (pc >= 0) prep_load(P, pc, ptensor, pg, lane, r);
  if (b >= NCOMM) {
    if (wid >= 3) tab_copy(TABS.enc, shm + ENC, ENC_KB, wid - 3, 5, lane);
  } else if (wid == 0) {
    *(LAS u32x2*)((LAS char*)shm + RING_CTL + lane * 8) = (u32x2){0u, (lane & 31) == 0 ? 0xffffffffu : 0u};
  }
  WAIT_V(0); asm volatile("" : "+v"(cnt));
  const unsigned call = (unsigned)__builtin_amdgcn_readfirstlane((int)cnt) / NBLK + 1;
  if (pc >= 0 && ptensor) prep_transpose(r);
  WAIT_L(0); BAR;
  STAMP(P.ws, 6);
  if (pc >= RAWC) prep_store(P, (LAS char*)shm + wid * 9216, (const LAS char*)shm + ENC, pc, ptensor, pg, lane, call, r);
  else if (pc >= 0) prep_push(P, pc, ptensor, pg, lane, call, r);
  else if (b < NCOMM && wid >= 2) {
    tab_copy(TABS.dec, shm, DEC_KB, wid - 2, 5, lane);
    WAIT_V(0);
    if (lane == 0) asm volatile("ds_add_u32 %0, %1" ::"v"((unsigned)(size_t)(LAS char*)shm + RING_CTL + 12), "v"(1u) : "memory");
  }
  STAMP(P.ws, 1);
  if (b < NCOMM) {
    if (wid < 2) loader_main(P, shm, wid, b * 2 + wid, lane, call);
    else decoder_main(P, shm, wid & 1, lane, call);
    if (threadIdx.x == 0) __hip_atomic_fetch_add((unsigned*)(P.ws + OFF_CNT), 1u, __ATOMIC_RELAXED, __HIP_MEMORY_SCOPE_AGENT);
    STAMP(P.ws, 5);
    return;
  }
  WAIT_V(0); WAIT_L(0); BAR;
  const int xcd = b & 7, idx = j >> 3;
  const int rb = idx % 7, bcol = ((xcd >> 1) * 4 + idx / 7) * 256;
  const int browA = (xcd & 1) * 1024 + (rb <= 4 ? rb * 160 : 128 + rb * 128);
  const int browB = 2048 + (xcd & 1) * 1024 + (rb == 0 ? 0 : rb <= 4 ? 32 + rb * 128 : rb * 160 - 96);
  if (rb == 0) gemm_main<5, 5>(P, shm, browA, browB, bcol, wid, lane, call);
  else if (rb < 4) gemm_main<5, 4>(P, shm, browA, browB, bcol, wid, lane, call);
  else gemm_main<4, 5>(P, shm, browA, browB, bcol, wid, lane, call);
  if (threadIdx.x == 0) __hip_atomic_fetch_add((unsigned*)(P.ws + OFF_CNT), 1u, __ATOMIC_RELAXED, __HIP_MEMORY_SCOPE_AGENT);
}

}

extern "C" void kernel_launch(int rank, int nranks, ncclComm_t comm, hipStream_t stream, void* const* d_in, const int* in_sizes, int n_in,
                              void* d_out, int out_size, const int* in_dtypes, int out_dtype, void* const* d_out_all,
                              void* const* d_ws_all, size_t ws_bytes) {
  if (nranks != NR || n_in != 4 || in_sizes[0] != DIM * KSH || in_sizes[1] != KSH * DIM || out_size != DIM * DIM || ws_bytes < (size_t)WS_NEED) {
    fprintf(stderr, "unexpected problem shape\n");
    return;
  }
  constexpr int lds = LDS_BYTES;
  HIP_CHECK(hipFuncSetAttribute((const void*)fused_kernel, hipFuncAttributeMaxDynamicSharedMemorySize, lds));
  Params P;
  P.x = (const unsigned char*)d_in[0];
  P.w = (const unsigned char*)d_in[1];
  P.sx = (const float*)d_in[2];
  P.sw = (const float*)d_in[3];
  P.out = (float*)d_out;
  P.ws_all = (char* const*)d_ws_all;
  P.ws = (char*)d_ws_all[rank];
  P.rank = rank;
  P.pad = 0;
  hipLaunchKernelGGL(fused_kernel, dim3(NBLK), dim3(512), lds, stream, P);
  HIP_CHECK(hipGetLastError());
}
```

```cpp
#include <hip/hip_runtime.h>
#include <rccl/rccl.h>
#include <cstdio>
#include <type_traits>

#define HIP_CHECK(expr) do { hipError_t _e = (expr); if (_e != hipSuccess) { \
    fprintf(stderr, "HIP error %s at %s:%d\n", hipGetErrorString(_e), __FILE__, __LINE__); \
    return; } } while (0)

namespace {

constexpr int NR = 8;
constexpr int DIM = 4096;
constexpr int KSH = DIM / NR;
constexpr int NSTEP = 32;
constexpr int SLAB = DIM * 16;
constexpr int STEPB = 8 * SLAB;
constexpr int NBLK = 256, NCOMM = 32;
constexpr int UPP = 512;
constexpr int RAWC = 2;
constexpr int NPULL = UPP - 16 * RAWC;
#define LPP 9
#define WIN 7
#define TAILS 4
constexpr int JSTEP = NSTEP - TAILS;
constexpr int OFF_CNT = 0, OFF_TMO = 128, OFF_CLAIM = 256;
constexpr int OFF_MAP = 1280;
constexpr int OFF_PCNT = 2048;
constexpr int OFF_PF = 4096;
constexpr int OFF_UF = 8192;
constexpr int OFF_DBG = 65536;
constexpr int PSTEPS = 6;
constexpr int GSTEPS = NSTEP + PSTEPS;
constexpr int OFF_GX = 1 << 20, OFF_GW = OFF_GX + GSTEPS * STEPB;
constexpr int OFF_PK = OFF_GW + GSTEPS * STEPB;
constexpr int WS_NEED = OFF_PK + 2 * UPP * 8192;
constexpr int LDS_BYTES = 2 * (128 * 320 + 32768);
constexpr unsigned long long DEADLINE = 30000000ull;

typedef int v8i __attribute__((ext_vector_type(8)));
typedef int v4i __attribute__((ext_vector_type(4)));
typedef unsigned u32x4 __attribute__((ext_vector_type(4)));
typedef unsigned u32x2 __attribute__((ext_vector_type(2)));
typedef unsigned u32x3 __attribute__((ext_vector_type(3)));
typedef float f32x4 __attribute__((ext_vector_type(4)));
#define LAS __attribute__((address_space(3)))
#define WAIT_V(n) asm volatile("s_waitcnt vmcnt(%0)" ::"i"(n) : "memory")
#define WAIT_L(n) asm volatile("s_waitcnt lgkmcnt(%0)" ::"i"(n) : "memory")
#define BAR __builtin_amdgcn_s_barrier()
#define SCHED __builtin_amdgcn_sched_barrier(0)

struct Params {
  const unsigned char* x;
  const unsigned char* w;
  const float* sx;
  const float* sw;
  float* out;
  char* const* ws_all;
  char* ws;
  int rank;
  int pad;
};

#define STAMP(ws, idx) do {} while (0)
#define DBG(...)

__device__ __forceinline__ void st16_sys(void* p, u32x4 v) { asm volatile("global_store_dwordx4 %0, %1, off sc0 sc1\n\ts_nop 1" ::"v"(p), "v"(v) : "memory"); }
__device__ __forceinline__ void st4_sys(void* p, unsigned v) { asm volatile("global_store_dword %0, %1, off sc0 sc1" ::"v"(p), "v"(v) : "memory"); }
__device__ __forceinline__ unsigned ld4_sys(const void* p) {
  unsigned v; asm volatile("global_load_dword %0, %1, off sc0 sc1\n\ts_waitcnt vmcnt(0)" : "=&v"(v) : "v"(p) : "memory"); return v;
}
__device__ __forceinline__ void ticket_issue(unsigned& t, void* p, unsigned long long lanes) {
  unsigned long long keep;
  asm volatile("s_mov_b64 %1, exec\n\ts_mov_b64 exec, %4\n\tglobal_atomic_add %0, %2, %3, off sc0\n\ts_mov_b64 exec, %1"
               : "+v"(t), "=&s"(keep) : "v"(p), "v"(1u), "s"(lanes) : "memory");
}
__device__ __forceinline__ void give_up(char* ws, unsigned call) { st4_sys(ws + OFF_TMO + (threadIdx.x & 31) * 4, call); WAIT_V(0); }
__device__ __forceinline__ bool spin_fail(char* ws, unsigned call, unsigned long long t0) {
  if ((unsigned)__builtin_amdgcn_readfirstlane((int)ld4_sys(ws + OFF_TMO)) == call) return true;
  if (__builtin_amdgcn_s_memrealtime() - t0 > DEADLINE) { give_up(ws, call); return true; }
  return false;
}
__device__ __forceinline__ int gstep_off(int step, unsigned par) { return (step < PSTEPS ? (int)par * PSTEPS + step : step + PSTEPS) * STEPB; }

__device__ __forceinline__ int vrow(int v) { return (v & 0x87f) | (v >> 3 & 0x80) | (v << 1 & 0x700); }
typedef __amdgpu_buffer_rsrc_t rsrc_t;
#define BST16(v, rs, voff, soff, imm) asm volatile("buffer_store_dwordx4 %0, %1, %2, %3 offen offset:%4 sc0 sc1\n\ts_nop 1" ::"v"(v), "v"(voff), "s"(rs), "s"(soff), "i"(imm) : "memory")
#define BLD16(v, rs, voff, soff, imm) asm volatile("buffer_load_dwordx4 %0, %1, %2, %3 offen offset:%4 sc0 sc1" : "=v"(v) : "v"(voff), "s"(rs), "s"(soff), "i"(imm) : "memory")
__device__ __forceinline__ void store_unit(const u32x4 (&r)[8], int n, rsrc_t rs, int voff, int soff) {
#pragma unroll
  for (int i = 0; i < 8; ++i)
    if (i < n) BST16(r[i], rs, voff, soff + (i >> 2) * 4096, (i & 3) * 1024);
}
__device__ __forceinline__ int slab_off(int tensor, int gstep_bytes, int slot, int g) { return (tensor ? OFF_GW : OFF_GX) + gstep_bytes + slot * SLAB + g * 8192; }
__device__ __forceinline__ int uflag_off(int tensor, int step, int slot, int g) { return OFF_UF + (((slot * 2 + tensor) * 8 + g) * 32 + step) * 4; }

__device__ __forceinline__ void prep_load(const Params& P, int c, int tensor, int g, int lane, u32x4 (&r)[8]) {
  if (tensor == 0) {
    auto rs = __builtin_amdgcn_make_buffer_rsrc((void*)P.x, 0, DIM * KSH, 0x00020000);
#pragma unroll
    for (int i = 0; i < 8; ++i) r[i] = __builtin_amdgcn_raw_buffer_load_b128(rs, lane * KSH, vrow(g * 512 + i * 64) * KSH + c * 16, 0);
  } else {
    auto rs = __builtin_amdgcn_make_buffer_rsrc((void*)P.w, 0, DIM * KSH, 0x00020000);
#pragma unroll
    for (int k = 0; k < 16; ++k)
#pragma unroll
      for (int h = 0; h < 2; ++h) r[4 * h + (k >> 2)][k & 3] = __builtin_amdgcn_raw_buffer_load_b32(rs, lane * 4 + h * 256, (c * 16 + k) * DIM + g * 512, 0);
  }
}
__device__ __forceinline__ void prep_transpose(u32x4 (&r)[8]) {
#pragma unroll
  for (int h = 0; h < 2; ++h) {
    u32x4 o[4];
#pragma unroll
    for (int q = 0; q < 4; ++q) {
      const u32x4 a = r[4 * h + q];
      const unsigned l01 = __builtin_amdgcn_perm(a[1], a[0], 0x05010400), h01 = __builtin_amdgcn_perm(a[1], a[0], 0x07030602);
      const unsigned l23 = __builtin_amdgcn_perm(a[3], a[2], 0x05010400), h23 = __builtin_amdgcn_perm(a[3], a[2], 0x07030602);
      o[0][q] = __builtin_amdgcn_perm(l23, l01, 0x05040100);
      o[1][q] = __builtin_amdgcn_perm(l23, l01, 0x07060302);
      o[2][q] = __builtin_amdgcn_perm(h23, h01, 0x05040100);
      o[3][q] = __builtin_amdgcn_perm(h23, h01, 0x07060302);
    }
#pragma unroll
    for (int q = 0; q < 4; ++q) r[4 * h + q] = o[q];
  }
}

__device__ __forceinline__ void prep_load2(const Params& P, int c, int tensor, int g, int sg, int lane, u32x4 (&d)[4]) {
  if (tensor == 0) {
    auto rs = __builtin_amdgcn_make_buffer_rsrc((void*)P.x, 0, DIM * KSH, 0x00020000);
#pragma unroll
    for (int i = 0; i < 2; ++i) d[i] = __builtin_amdgcn_raw_buffer_load_b128(rs, lane * KSH, vrow(g * 512 + sg * 128 + i * 64) * KSH + c * 16, 0);
  } else {
    auto rs = __builtin_amdgcn_make_buffer_rsrc((void*)P.w, 0, DIM * KSH, 0x00020000);
#pragma unroll
    for (int k = 0; k < 16; ++k) d[k >> 2][k & 3] = __builtin_amdgcn_raw_buffer_load_b32(rs, lane * 4, (c * 16 + k) * DIM + g * 512 + (sg >> 1) * 256, 0);
  }
}
__device__ __forceinline__ void prep_pieces2(int tensor, int sg, const u32x4 (&d)[4], u32x4 (&r)[2]) {
  if (tensor == 0) { r[0] = d[0]; r[1] = d[1]; return; }
  const unsigned sel = (sg & 1) ? 0x07030602u : 0x05010400u;
#pragma unroll
  for (int q = 0; q < 4; ++q) {
    const unsigned p01 = __builtin_amdgcn_perm(d[q][1], d[q][0], sel), p23 = __builtin_amdgcn_perm(d[q][3], d[q][2], sel);
    r[0][q] = __builtin_amdgcn_perm(p23, p01, 0x05040100);
    r[1][q] = __builtin_amdgcn_perm(p23, p01, 0x07060302);
  }
}

constexpr int NCLS = 12, NQUAD = 256 + 4096;
struct QuadTab { unsigned short q[NQUAD]; };
constexpr QuadTab make_quads() {
  constexpr int cost[NCLS] = {86, 33, 27, 37, 51, 67, 83, 99, 114, 130, 146, 162};
  int hist[4 * 162 + 2] = {};
  for (int a = 0; a < NCLS; ++a) for (int b = 0; b < NCLS; ++b) for (int c = 0; c < NCLS; ++c) for (int d = 0; d < NCLS; ++d) ++hist[cost[a] + cost[b] + cost[c] + cost[d]];
  int t1 = 0, t2 = 0, n = 0;
  while (n + hist[t1] <= 256) n += hist[t1++];
  int tie1 = 256 - n;
  n = 0;
  while (n + hist[t2] <= NQUAD) n += hist[t2++];
  int tie2 = NQUAD - n, n1 = 0, n2 = 256;
  QuadTab t{};
  for (int a = 0; a < NCLS; ++a) for (int b = 0; b < NCLS; ++b) for (int c = 0; c < NCLS; ++c) for (int d = 0; d < NCLS; ++d) {
    const int k = cost[a] + cost[b] + cost[c] + cost[d];
    if (k < t1 || (k == t1 && tie1-- > 0)) t.q[n1++] = (unsigned short)(a | b << 4 | c << 8 | d << 12);
    else if (k < t2 || (k == t2 && tie2-- > 0)) t.q[n2++] = (unsigned short)(a | b << 4 | c << 8 | d << 12);
  }
  return t;
}
constexpr unsigned spread4(unsigned q) { return (q & 15u) | (q & 0xf0u) << 4 | (q & 0xf00u) << 8 | (q & 0xf000u) << 12; }
constexpr QuadTab QUADS = make_quads();
constexpr unsigned QUAD0 = spread4(QUADS.q[0]);
constexpr int ENC_KB = (2 * NCLS * NCLS * NCLS * NCLS + 1023) / 1024, DEC_KB = 34;
struct alignas(16) Tabs { unsigned short enc[ENC_KB * 512]; unsigned dec[DEC_KB * 256]; };
constexpr Tabs make_tabs() {
  Tabs t{};
  for (int i = 0; i < ENC_KB * 512; ++i) t.enc[i] = 0x8000;
  for (int i = 0; i < NQUAD; ++i) {
    const int q = QUADS.q[i];
    t.enc[(q & 15) + NCLS * ((q >> 4 & 15) + NCLS * ((q >> 8 & 15) + NCLS * (q >> 12)))] = (unsigned short)(i < 256 ? i : ((i - 256) >> 4) | ((i & 15) << 8) | 0x1000);
    const int a = i < 256 ? 17 * i : 17 * ((i - 256) >> 4) + 1 + (i & 15);
    t.dec[a] = 0u - (spread4(q) << 3);
    t.dec[17 * 256 + a] = 0u - (spread4(q) << 2);
  }
  return t;
}
__constant__ const Tabs TABS = make_tabs();
constexpr int PK_SIDE = 6144;
constexpr int pool_dwords(int tensor) { return tensor ? 160 : 144; }
__device__ __forceinline__ void tab_copy(const void* src, char* lds, int kb, int w, int nw, int lane) {
  const auto rs = __builtin_amdgcn_make_buffer_rsrc((void*)src, 0, kb * 1024, 0x00020000);
  for (int k = w; k < kb; k += nw) __builtin_amdgcn_raw_ptr_buffer_load_lds(rs, (LAS void*)(lds + k * 1024), 16, lane * 16, k * 1024, 0, 0);
}
__device__ __forceinline__ unsigned wave_sum(unsigned v) {
  v += (unsigned)__builtin_amdgcn_update_dpp(0, (int)v, 0xb1, 0xf, 0xf, true);
  v += (unsigned)__builtin_amdgcn_update_dpp(0, (int)v, 0x4e, 0xf, 0xf, true);
  v += (unsigned)__builtin_amdgcn_update_dpp(0, (int)v, 0x141, 0xf, 0xf, true);
  v += (unsigned)__builtin_amdgcn_update_dpp(0, (int)v, 0x140, 0xf, 0xf, true);
  return (unsigned)(__builtin_amdgcn_readlane((int)v, 0) + __builtin_amdgcn_readlane((int)v, 16) + __builtin_amdgcn_readlane((int)v, 32) + __builtin_amdgcn_readlane((int)v, 48));
}
__device__ __forceinline__ unsigned wave_scan(unsigned v) {
  v += (unsigned)__builtin_amdgcn_update_dpp(0, (int)v, 0x111, 0xf, 0xf, true);
  v += (unsigned)__builtin_amdgcn_update_dpp(0, (int)v, 0x112, 0xf, 0xf, true);
  v += (unsigned)__builtin_amdgcn_update_dpp(0, (int)v, 0x114, 0xf, 0xf, true);
  v += (unsigned)__builtin_amdgcn_update_dpp(0, (int)v, 0x118, 0xf, 0xf, true);
  v += (unsigned)__builtin_amdgcn_update_dpp(0, (int)v, 0x142, 0xa, 0xf, false);
  v += (unsigned)__builtin_amdgcn_update_dpp(0, (int)v, 0x143, 0xc, 0xf, false);
  return v;
}

__device__ __forceinline__ unsigned shl_add(unsigned a, int n, unsigned b) { unsigned r; asm("v_lshl_add_u32 %0, %1, %2, %3" : "=v"(r) : "v"(a), "i"(n), "v"(b)); return r; }
__device__ __forceinline__ unsigned shl_or(unsigned a, int n, unsigned b) { unsigned r; asm("v_lshl_or_b32 %0, %1, %2, %3" : "=v"(r) : "v"(a), "i"(n), "v"(b)); return r; }
constexpr int PU_PLANES = 8192, PU_SIDE = PU_PLANES + 6144, PU_MISS = PU_SIDE + 1024, PU_SUM = PU_MISS + 256, PU_BYTES = PU_SUM + 64;
template <int TENSOR, typename F>
__device__ __forceinline__ bool pack_unit(const u32x4 (&r)[2], LAS char* lds, const LAS char* enc, int sg, int lane, F&& stored DBG(, char* ws)) {
  constexpr unsigned ONE = 0x01010101u;
  constexpr int SH = TENSOR ? 2 : 3;
  constexpr unsigned LM = ((1u << SH) - 1) * ONE, CM = (0x7fu >> SH) * ONE;
  unsigned S = 0x80808080u, SL = 0x80808080u | LM, LR = LM;
  asm("" : "+s"(S), "+s"(SL), "+s"(LR));
  LAS unsigned* side = (LAS unsigned*)(lds + PU_SIDE);
  unsigned sum = 0;
#pragma unroll
  for (int i = 0; i < 2; ++i) {
    *(LAS u32x4*)(lds + (2 * sg + i) * 1024 + lane * 16) = r[i];
#pragma unroll
    for (int d = 0; d < 4; ++d) sum = __builtin_amdgcn_sad_u8(r[i][d] & 0x7f7f7f7fu, 0u, sum);
  }
  side[sg * 64 + lane] = 0;
  *(LAS unsigned*)(lds + PU_SUM + sg * 4) = wave_sum(sum);
  STAMP(ws, 33);
  WAIT_L(0); BAR; asm volatile("" ::: "memory");
  STAMP(ws, 34);
  const u32x4 sums = *(const LAS u32x4*)(lds + PU_SUM);
  int top = (__builtin_amdgcn_readfirstlane((int)(sums.x + sums.y + sums.z + sums.w)) + (TENSOR ? 101581 : 204800)) >> 13;
  top = top > 127 ? 127 : top;
  const unsigned T = ((unsigned)top | 0x80u) * ONE;
  unsigned u[8], e[8], y[8], nib = 0, cnt = 0, fl = 0, miss = 0;
#pragma unroll
  for (int k = 0; k < 8; ++k) {
    u[k] = T - (r[k >> 2][k & 3] & 0x7f7f7f7fu);
    const unsigned cb = (u[k] >> SH) & CM;
    const bool ok = __builtin_amdgcn_bitop3_b32(u[k], u[k] + (128 - (NCLS << SH)) * ONE, S, 0x80) == 0x80808080u;
    const unsigned a = __umul24(__builtin_amdgcn_udot4(cb, (2u | 2u * NCLS << 8) << 16, 0u, false), NCLS * NCLS) + __builtin_amdgcn_udot4(cb, 2u | 2u * NCLS << 8, 0u, false);
    e[k] = *(const LAS unsigned short*)(enc + (ok ? a : 2u * NCLS * NCLS * NCLS * NCLS));
  }
#pragma unroll
  for (int k = 0; k < 8; ++k) {
    const unsigned v = r[k >> 2][k & 3], f = __builtin_amdgcn_ubfe(e[k], 12, 1);
    fl = shl_or(f, k, fl);
    nib |= __builtin_amdgcn_ubfe(e[k], 8, 4) << cnt;
    cnt = shl_add(f, 2, cnt);
    miss = shl_or(e[k] >> 15, k, miss);
    const unsigned g = __builtin_amdgcn_bitop3_b32(u[k], v, LR, 0x82);
    y[k] = __builtin_amdgcn_bitop3_b32(v, e[k] >= 0x8000u ? 0u : g, SL, 0x28);
  }
  LAS char* pl = lds + PU_PLANES + lane * 16;
#pragma unroll
  for (int k = 0; k < 4; ++k)
    *(LAS unsigned short*)(pl + ((TENSOR ? 3 : 4) + (sg & 1)) * 1024 + k * 4 + (sg & 2)) = (unsigned short)__builtin_amdgcn_perm(e[2 * k + 1], e[2 * k], 0x0c0c0400);
  if (TENSOR == 0) {
    u32x4 o;
#pragma unroll
    for (int k = 0; k < 4; ++k) o[k] = y[2 * k] | __builtin_amdgcn_alignbit(y[2 * k + 1], y[2 * k + 1], 28);
    *(LAS u32x4*)(pl + sg * 1024) = o;
  } else {
    unsigned o[3];
#pragma unroll
    for (int k = 0; k < 3; ++k) o[k] = y[k] | __builtin_amdgcn_alignbit(y[3 + k], y[3 + k], 29);
    o[0] |= (y[6] & LM) << 5; o[1] |= (y[7] & LM) << 5;
    o[2] |= ((y[6] >> 2) & 0x20202020u) | ((y[7] >> 1) & 0x40404040u);
#pragma unroll
    for (int k = 0; k < 3; ++k) *(LAS unsigned*)(pl + ((3 * sg + k) >> 2) * 1024 + ((3 * sg + k) & 3) * 4) = o[k];
  }
  *(LAS unsigned char*)(lds + PU_SIDE + lane * 4 + sg) = (unsigned char)fl;
  *(LAS unsigned char*)(lds + PU_MISS + lane * 4 + sg) = (unsigned char)miss;
  STAMP(ws, 35);
  WAIT_V(0); WAIT_L(0); BAR; asm volatile("" ::: "memory");
  STAMP(ws, 36);
  stored();
  const unsigned fa = side[lane], ma = *(const LAS unsigned*)(lds + PU_MISS + lane * 4), later = ~0u << (8 * sg);
  const unsigned incl = wave_scan(__builtin_popcount(fa) | __builtin_popcount(ma) << 16);
  const unsigned tot = (unsigned)__builtin_amdgcn_readlane((int)incl, 63);
  const unsigned nnib = tot & 0xffffu, nexc = tot >> 16, ex0 = 65 + ((nnib + 7) >> 3);
  const bool over = nexc > 64 || ex0 + nexc > 64 + pool_dwords(TENSOR);
  if (!over) {
    if (sg == 0 && lane == 0) side[64] = (unsigned)top | nexc << 8 | nnib << 16;
    const unsigned pos = (incl & 0xffffu) - __builtin_popcount(fa & later);
    const unsigned long long w = (unsigned long long)nib << ((pos & 7u) * 4);
    __hip_atomic_fetch_or(side + 65 + (pos >> 3), (unsigned)w, __ATOMIC_RELAXED, __HIP_MEMORY_SCOPE_WORKGROUP);
    __hip_atomic_fetch_or(side + 66 + (pos >> 3), (unsigned)(w >> 32), __ATOMIC_RELAXED, __HIP_MEMORY_SCOPE_WORKGROUP);
    unsigned at = ex0 + (incl >> 16) - __builtin_popcount(ma & later);
    while (miss) {
      const unsigned k = __builtin_ctz(miss), q = (2 * sg + (k >> 2)) * 256 + lane * 4 + (k & 3);
      miss &= miss - 1;
      const unsigned h = (*(const LAS unsigned*)(lds + 4 * q) >> SH) & CM;
      constexpr int W = 7 - SH;
      __hip_atomic_fetch_or(side + at++, q | ((h & 0xffu) | (h >> 8 & 0xffu) << W | (h >> 16 & 0xffu) << 2 * W | (h >> 24) << 3 * W) << 11, __ATOMIC_RELAXED, __HIP_MEMORY_SCOPE_WORKGROUP);
    }
  }
  WAIT_L(0); BAR; asm volatile("" ::: "memory");
  return over;
}

template <int TENSOR, typename F>
__device__ __forceinline__ void unpack_unit(u32x4 (&r)[8], unsigned fl, unsigned hv, LAS char* sp, const LAS char* dec, int lane, F&& mid) {
  constexpr unsigned ONE = 0x01010101u;
  constexpr int SH = TENSOR ? 2 : 3, W = 7 - SH;
  constexpr unsigned LM = ((1u << SH) - 1) * ONE, SL = 0x80808080u | LM;
  const LAS unsigned* side = (const LAS unsigned*)(sp + 8192);
  const unsigned nn = __builtin_popcount(fl);
  unsigned pos = wave_scan(nn) - nn, w[4], w1[4], sh[4];
#pragma unroll
  for (int sg = 0; sg < 4; ++sg) {
    w[sg] = side[65 + (pos >> 3)]; w1[sg] = side[66 + (pos >> 3)]; sh[sg] = pos << 2;
    pos += __builtin_popcount((fl >> (8 * sg)) & 0xffu);
  }
  const unsigned hdr = (unsigned)__builtin_amdgcn_readfirstlane((int)hv);
  const unsigned T = (hdr & 0xffu) * ONE, nexc = hdr >> 8 & 0xffu;
  const unsigned ent = side[65 + (((hdr >> 16) + 7) >> 3) + lane];
  SCHED;
  unsigned q[24], t[32], c[32];
#pragma unroll
  for (int k = 0; k < 24; ++k) q[k] = r[k >> 2][k & 3];
#pragma unroll
  for (int j = 0; j < 32; ++j) {
    if (TENSOR == 0) t[j] = ((j & 1) ? __builtin_amdgcn_alignbit(q[j >> 1], q[j >> 1], 4) : q[j >> 1]) & SL;
    else {
      const int k = j & 7, o = 3 * (j >> 3);
      t[j] = k < 3 ? q[o + k] & SL : k < 6 ? __builtin_amdgcn_alignbit(q[o + k - 3], q[o + k - 3], 3) & SL
                                           : ((q[o + k - 6] >> 5) & LM) | ((q[o + 2] << (8 - k)) & 0x80808080u);
    }
  }
  SCHED;
#pragma unroll
  for (int sg = 0; sg < 4; ++sg) w[sg] = __builtin_amdgcn_alignbit(w1[sg], w[sg], sh[sg]);
  unsigned k68 = 0x00440044u, sel0 = 0x0c020c00u, sel1 = 0x0c030c01u;
  asm("" : "+s"(k68), "+s"(sel0), "+s"(sel1));
  unsigned wp;
#pragma unroll
  for (int j = 0; j < 16; ++j) {
    if ((j & 3) == 0) {
      if (j & 4) { w[j >> 3] >>= 4 * __builtin_popcount(fl & (15u << (j - 4))); w[2 + (j >> 3)] >>= 4 * __builtin_popcount(fl & (15u << (j + 12))); }
      wp = __builtin_amdgcn_perm(w[2 + (j >> 3)], w[j >> 3], 0x05040100);
    }
    const unsigned wd = (j >= 2 ? fl >> (j - 2) : fl << (2 - j)) & 0x00040004u;
    unsigned ad;
    asm("v_pk_mad_u16 %0, %1, %2, %2" : "=v"(ad) : "v"(wp & 0x000f000fu), "v"(wd));
    asm("v_pk_mad_u16 %0, %1, %2, %0" : "+v"(ad) : "v"(__builtin_amdgcn_perm(0u, q[(TENSOR ? 12 : 16) + (j >> 1)], (j & 1) ? sel1 : sel0)), "s"(k68));
    asm("v_pk_lshrrev_b16 %0, %1, %0" : "+v"(wp) : "v"(wd));
    c[j] = *(const LAS unsigned*)(dec + TENSOR * 17408 + (ad & 0xffffu));
    c[j + 16] = *(const LAS unsigned*)(dec + TENSOR * 17408 + (ad >> 16));
  }
  SCHED; mid(); SCHED;
#pragma unroll
  for (int i = 0; i < 8; ++i) {
#pragma unroll
    for (int d = 0; d < 4; ++d) r[i][d] = t[4 * i + d] + c[4 * i + d] + (T - LM);
    *(LAS u32x4*)(sp + i * 1024 + lane * 16) = r[i];
  }
  if ((unsigned)lane < nexc) {
    const unsigned e = ent >> 11, F7 = (1u << W) - 1;
    const unsigned hi = ((e & F7) | (e >> W & F7) << 8 | (e >> 2 * W & F7) << 16 | (e >> 3 * W) << 24) << SH;
    __hip_atomic_fetch_add((LAS unsigned*)(sp + 4 * (ent & 0x7ffu)), hi + LM + (QUAD0 << SH) - T, __ATOMIC_RELAXED, __HIP_MEMORY_SCOPE_WORKGROUP);
  }
  asm volatile("" ::: "memory");
#pragma unroll
  for (int i = 0; i < 8; ++i) r[i] = *(const LAS u32x4*)(sp + i * 1024 + lane * 16);
}

__device__ __forceinline__ void post_peers(const Params& P, int word, int lane, unsigned call, unsigned bits) {
  if (lane < NR - 1) {
    const unsigned p = (unsigned)(P.rank + 1 + lane) & (NR - 1);
    const u32x2 v = {call, bits};
    asm volatile("global_store_dwordx2 %0, %1, off sc0 sc1" ::"v"(P.ws_all[p] + OFF_PF + P.rank * 512 + word * 8), "v"(v) : "memory");
  }
}
__device__ __forceinline__ rsrc_t peer_rsrc(const Params& P, unsigned p) {
  const unsigned long long pa = (unsigned long long)P.ws_all[p & (NR - 1)];
  const unsigned long long pu = ((unsigned long long)(unsigned)__builtin_amdgcn_readfirstlane((int)(pa >> 32)) << 32) | (unsigned)__builtin_amdgcn_readfirstlane((int)pa);
  return __builtin_amdgcn_make_buffer_rsrc((void*)pu, 0, WS_NEED, 0x00020000);
}
#define PUSHWIN 13
__device__ __forceinline__ void prep_push(const Params& P, int c, int tensor, int g, int lane, unsigned call, const u32x4 (&r)[8]) {
  rsrc_t rp[NR - 1];
#pragma unroll
  for (int d = 0; d < NR - 1; ++d) rp[d] = peer_rsrc(P, (unsigned)(P.rank + 1 + d));
  asm volatile("s_nop 4");
  const rsrc_t rs = __builtin_amdgcn_make_buffer_rsrc((void*)P.ws, 0, WS_NEED, 0x00020000);
#pragma unroll
  for (int i = 0; i < 8; ++i) {
#pragma unroll
    for (int d = 0; d < NR - 1; ++d) {
      const int ci = c * 7 + 6 - d;
      BST16(r[i], rp[d], lane * 16, slab_off(tensor, gstep_off(4 + (ci >> 3), call & 1), ci & 7, g) + (i >> 2) * 4096, (i & 3) * 1024);
      WAIT_V(PUSHWIN);
    }
    if (i == 0) store_unit(r, 8, rs, lane * 16, slab_off(tensor, gstep_off(0, call & 1), c, g));
    static_assert(PUSHWIN <= 14, "two planes of remote stores lie between the own copy and its flag");
    if (i == 2 && lane == 0) st4_sys(P.ws + uflag_off(tensor, 0, c, g), call);
  }
  WAIT_V(0);
  const int ci = c * 7 + 6 - lane;
  if (lane < NR - 1) st4_sys(P.ws_all[(unsigned)(P.rank + 1 + lane) & (NR - 1)] + uflag_off(tensor, 4 + (ci >> 3), ci & 7, g), call);
}
__device__ __forceinline__ void prep_raw(const Params& P, int c0, int tensor, int g, int wid, int lane, unsigned call, const u32x4 (&r)[2]) {
  const rsrc_t rs = __builtin_amdgcn_make_buffer_rsrc((void*)P.ws, 0, WS_NEED, 0x00020000);
  const int c = c0 + (wid >> 2), raw = slab_off(tensor, gstep_off(c >> 3, call & 1), c & 7, g) + (wid & 3) * 2048;
  BST16(r[0], rs, lane * 16, raw, 0);
  BST16(r[1], rs, lane * 16, raw, 1024);
}
__device__ __forceinline__ void prep_pack(const Params& P, LAS char* shm, const LAS char* enc, int c0, int tensor, int g, int wid, int lane, unsigned call,
                                          const u32x4 (&r)[2]) {
  const rsrc_t rs = __builtin_amdgcn_make_buffer_rsrc((void*)P.ws, 0, WS_NEED, 0x00020000);
  const int c = c0 + (wid >> 2), sg = wid & 3;
  LAS char* lds = shm + (wid >> 2) * PU_BYTES;
  int pk = OFF_PK + (int)(call & 1) * (UPP * 8192) + (c * 16 + tensor * 8 + g) * 8192;
  asm volatile("" : "+s"(pk));
  auto stored = [&] {
    const int cf = c + 2 * lane;
    if (sg == 0 && c0 < 30 && lane < (c0 == 28 ? 2 : 1)) st4_sys(P.ws + uflag_off(tensor, cf >> 3, cf & 7, g), call);
  };
  const bool ovf = tensor ? pack_unit<1>(r, lds, enc, sg, lane, stored DBG(, P.ws)) : pack_unit<0>(r, lds, enc, sg, lane, stored DBG(, P.ws));
  STAMP(P.ws, 7);
  unsigned* pc = (unsigned*)(P.ws + OFF_PCNT + (c >> 1) * 128);
  if (!ovf) {
    const u32x4 a = *(const LAS u32x4*)(lds + PU_PLANES + sg * 2048 + lane * 16);
    BST16(a, rs, lane * 16, pk + sg * 2048, 0);
    if (sg < 2 + (tensor == 0 && sg == 2)) {
      const u32x4 b2 = *(const LAS u32x4*)(lds + PU_PLANES + sg * 2048 + 1024 + lane * 16);
      BST16(b2, rs, lane * 16, pk + sg * 2048, 1024);
    }
  } else if (c0 >= 4 && sg == 0 && lane == 0) {
    __hip_atomic_fetch_or(pc + 1 + (call & 1), 1u << ((c & 1) * 16 + tensor * 8 + g), __ATOMIC_RELAXED, __HIP_MEMORY_SCOPE_AGENT);
  }
  WAIT_V(0); BAR;
  if (c0 < 4) {
    if (sg == 0) post_peers(P, (c - 2) * 16 + tensor * 8 + g, lane, call, ovf ? ~0u : 0u);
  } else if (wid == 7) {
    unsigned tk = 0;
    ticket_issue(tk, pc, 1);
    WAIT_V(0); asm volatile("" : "+v"(tk));
    if ((unsigned)__builtin_amdgcn_readfirstlane((int)tk) == call * 16 - 1) {
      post_peers(P, 30 + (c >> 1), lane, call, ld4_sys(pc + 1 + (call & 1)));
      if (lane == 0) st4_sys(pc + 1 + (~call & 1), 0u);
    }
  }
}

#define RAMP 12
#define LEADX 0, 0, 0, 0, 0, 0, 0, 0
#define LEADW 0, 0, 0, 0, 0, 0, 0, 0
#define NRAW2 0
#define NRAW 0
constexpr int FETCH_RAW = 1 << 16;
struct Order { int e[NPULL]; };
constexpr Order make_order() {
  constexpr int lead[16] = {LEADX, LEADW};
  Order o{};
  int next[16] = {}, early = 0;
  for (int t = 0; t < 16; ++t) next[t] = RAWC;
  for (int k = 0; k < NPULL; ++k) {
    int best = -1, key = 0;
    for (int v = 0; v < 16; ++v) {
      const int t = (v >> 1) + 8 * (v & 1);
      const int kt = 4 * RAMP * next[t] - lead[t] * (next[t] - RAWC < RAMP ? next[t] - RAWC : RAMP) +
                     (t >= 4 && t < 8 && 7 * next[t] >= 8 * (JSTEP - 4) ? 1 << 20 : 0);
      if (next[t] < 32 && (best < 0 || kt < key)) { best = t; key = kt; }
    }
    o.e[k] = next[best] * 16 + best;
    ++next[best];
    if (key < (1 << 20)) early = k + 1;
  }
  for (int k = early - NRAW; k < early; ++k) o.e[k] |= FETCH_RAW;
  for (int k = NPULL - NRAW2; k < NPULL; ++k) o.e[k] |= FETCH_RAW;
  return o;
}
__constant__ const Order ORDER = make_order();

constexpr int NSLOT = 6, SLOTB = 8192 + 1024;
constexpr int RING = DEC_KB * 1024, RING_CTL = RING + 2 * NSLOT * SLOTB;
static_assert(2 * PU_BYTES + ENC_KB * 1024 <= LDS_BYTES && PU_BYTES % 16 == 0 && PU_SIDE - PU_PLANES == PK_SIDE, "packer LDS");
static_assert(WIN >= 3 && WIN <= 7 && RING % 16 == 0 && RING_CTL + 512 + 1024 <= LDS_BYTES && LPP * 7 <= 2 * NCOMM, "ring");
__device__ __forceinline__ unsigned lds_ld(unsigned a) {
  unsigned v; asm volatile("ds_read_b32 %0, %1\n\ts_waitcnt lgkmcnt(0)" : "=v"(v) : "v"(a) : "memory");
  return (unsigned)__builtin_amdgcn_readfirstlane((int)v);
}
__device__ __forceinline__ void lds_st(unsigned a, unsigned v) { asm volatile("ds_write_b32 %0, %1" ::"v"(a), "v"(v) : "memory"); }

__device__ __forceinline__ void loader_main(const Params& P, char* shm, int l, int lid, int lane, unsigned call) {
  const unsigned ctl = (unsigned)(size_t)(LAS char*)shm + RING_CTL + l * 256;
  if (lid >= LPP * 7) { lds_st(ctl + 4, 0); return; }
  const int q = lid % 7, li = lid / 7;
  const unsigned p = (unsigned)(P.rank + 1 + q) & (NR - 1);
  const rsrc_t rp = peer_rsrc(P, p);
  const char* pfp = P.ws + OFF_PF + p * 512 + lane * 8;
  unsigned* claim = (unsigned*)(P.ws + OFF_CLAIM + q * 128);
  const unsigned cbase = (call - 1) * NPULL - 2 * LPP;
  const int pkb = OFF_PK + (int)(call & 1) * (UPP * 8192);
  char* ring = shm + RING + l * NSLOT * SLOTB;
  unsigned long long ready = 0;
  u32x2 pf = {0u, 0u};
  bool dead = false;
  int n = 0, slot = 0, src, nl;
  auto setup = [&](int ef, int un, int sl, int landed) {
    const int e = ef & (FETCH_RAW - 1);
    const int w = e < 64 ? e - 32 : 30 + (e >> 5);
    if (!dead && !((ready >> w) & 1)) {
      WAIT_V(0); lds_st(ctl, (unsigned)landed);
      const unsigned long long t0 = __builtin_amdgcn_s_memrealtime();
      while (true) {
        asm volatile("global_load_dwordx2 %0, %1, off sc0 sc1\n\ts_waitcnt vmcnt(0)" : "=&v"(pf) : "v"(pfp) : "memory");
        ready = __ballot(pf.x >= call);
        if ((ready >> w) & 1) break;
        if (spin_fail(P.ws, call, t0)) { dead = true; break; }
        __builtin_amdgcn_s_sleep(1);
      }
    }
    if (un >= NSLOT && (int)lds_ld(ctl + 76 + sl * 16) <= un - NSLOT) {
      const unsigned long long t0 = __builtin_amdgcn_s_memrealtime();
      while ((int)lds_ld(ctl + 76 + sl * 16) <= un - NSLOT && __builtin_amdgcn_s_memrealtime() - t0 < 2 * DEADLINE) {}
    }
    const int c = e >> 4, tensor = (e >> 3) & 1, g = e & 7, ci = c * 7 + q, step = 4 + (ci >> 3);
    const bool raw = ef >= FETCH_RAW || (((unsigned)__builtin_amdgcn_readlane((int)pf.y, w) >> (e & 31)) & 1u);
    nl = raw ? 8 : 6 - tensor;
    src = __builtin_amdgcn_readfirstlane(raw ? slab_off(tensor, gstep_off(c >> 3, call & 1), c & 7, g) : pkb + e * 8192);
    const u32x3 dsc = {(unsigned)nl, (unsigned)slab_off(tensor, gstep_off(step, call & 1), ci & 7, g), (unsigned)uflag_off(tensor, step, ci & 7, g)};
    if (lane == 0) asm volatile("ds_write_b96 %0, %1" ::"v"(ctl + 64 + sl * 16), "v"(dsc) : "memory");
  };
  unsigned tk = 0;
  int e = ORDER.e[li];
  setup(e, 0, 0, 0);
  STAMP(P.ws, 2);
  while (true) {
    const int s0 = src, n0 = nl;
    char* lp = ring + slot * SLOTB;
#define PLANE(i) do { WAIT_V(WIN - 1); if ((i) == WIN - 3) lds_st(ctl, (unsigned)n);   \
      __builtin_amdgcn_raw_ptr_buffer_load_lds(rp, (LAS void*)(lp + ((i) >> 2) * 4096), 16, lane * 16, s0 + ((i) >> 2) * 4096, ((i) & 3) * 1024, 17); } while (0)
    ticket_issue(tk, claim, 1ull << (n & 1));
    if (lane < (64 + pool_dwords(1)) / 4 - (n0 == 6 ? (pool_dwords(1) - pool_dwords(0)) / 4 : 0))
      __builtin_amdgcn_raw_ptr_buffer_load_lds(rp, (LAS void*)(lp + 8192), 16, lane * 16, pkb + (e & (FETCH_RAW - 1)) * 8192 + PK_SIDE, 0, 17);
    PLANE(0); PLANE(1);
    asm volatile("" : "+v"(tk));
    const int kn = n == 0 ? li + LPP : (int)((unsigned)__builtin_amdgcn_readlane((int)tk, ~n & 1) - cbase);
    const bool more = kn < NPULL;
    e = ORDER.e[more ? kn : 0];
    PLANE(2); PLANE(3); PLANE(4);
    if (n0 > 5) PLANE(5);
    if (n0 > 6) { PLANE(6); PLANE(7); }
    ++n; slot = slot == NSLOT - 1 ? 0 : slot + 1;
    if (!more) break;
    setup(e, n, slot, n);
  }
  WAIT_V(0);
  asm volatile("" : "+v"(tk));
  lds_st(ctl, (unsigned)n);
  lds_st(ctl + 4, (unsigned)n);
  DBG(STAMPV(P.ws, 10, (unsigned long long)n);)
}

__device__ __forceinline__ void decoder_main(const Params& P, char* shm, int l, int lane, unsigned call) {
  const unsigned ctl0 = (unsigned)(size_t)(LAS char*)shm + RING_CTL;
  const rsrc_t rs = __builtin_amdgcn_make_buffer_rsrc((void*)P.ws, 0, WS_NEED, 0x00020000);
  const unsigned long long t0 = __builtin_amdgcn_s_memrealtime();
  DBG(unsigned long long tdec = 0; unsigned long long cntu = 0;)
  int pflag = -1;
  auto post = [&] { if (pflag >= 0) { WAIT_V(0); if (lane == 0) st4_sys(P.ws + pflag, call); pflag = -1; } };
  unsigned tk = 0;
  auto draw = [&](int ring) {
    unsigned long long keep;
    asm volatile("s_mov_b64 %1, exec\n\ts_mov_b64 exec, 1\n\tds_add_rtn_u32 %0, %2, %3 offset:8\n\ts_mov_b64 exec, %1" : "+v"(tk), "=&s"(keep) : "v"(ctl0 + ring * 256), "v"(1u) : "memory");
  };
  while (lds_ld(ctl0 + 12) < 5 && __builtin_amdgcn_s_memrealtime() - t0 < DEADLINE) __builtin_amdgcn_s_sleep(1);
  draw(l);
  bool other = true;
  while (true) {
    WAIT_L(0); asm volatile("" : "+v"(tk));
    const int n = __builtin_amdgcn_readfirstlane((int)tk);
    const unsigned ctl = ctl0 + l * 256;
    bool have = true;
    for (int spin = 0; (int)lds_ld(ctl) <= n; ++spin) {
      post();
      if (lds_ld(ctl + 4) <= (unsigned)n || ((spin & 63) == 63 && __builtin_amdgcn_s_memrealtime() - t0 > 4 * DEADLINE)) { have = false; break; }
      __builtin_amdgcn_s_sleep(1);
    }
    if (!have) {
      if (!other) return;
      other = false; l ^= 1; draw(l);
      continue;
    }
    DBG(unsigned long long ta = NOW();)
    const int slot = n % NSLOT;
    LAS char* sp = (LAS char*)shm + RING + (l * NSLOT + slot) * SLOTB;
    const unsigned fl = *(const LAS unsigned*)(sp + 8192 + lane * 4), hv = *(const LAS unsigned*)(sp + 8192 + 256);
    const u32x4 dsc = *(const LAS u32x4*)((LAS char*)shm + RING_CTL + l * 256 + 64 + slot * 16);
    u32x4 r[8];
#pragma unroll
    for (int i = 0; i < 6; ++i) r[i] = *(const LAS u32x4*)(sp + i * 1024 + lane * 16);
    if (other) l ^= 1;
    draw(l);
    const int nl = __builtin_amdgcn_readfirstlane((int)dsc.x), dst = __builtin_amdgcn_readfirstlane((int)dsc.y), flag = __builtin_amdgcn_readfirstlane((int)dsc.z);
    if (nl == 6) unpack_unit<0>(r, fl, hv, sp, (const LAS char*)shm, lane, post);
    else if (nl == 5) unpack_unit<1>(r, fl, hv, sp, (const LAS char*)shm, lane, post);
    else { r[6] = *(const LAS u32x4*)(sp + 6144 + lane * 16); r[7] = *(const LAS u32x4*)(sp + 7168 + lane * 16); post(); }
    WAIT_L(0);
#pragma unroll
    for (int i = 0; i < 8; ++i) asm volatile("" : "+v"(r[i]));
    lds_st(ctl + 76 + slot * 16, (unsigned)(n + 1));
    asm volatile("s_nop 4");
    store_unit(r, 8, rs, lane * 16, dst);
    pflag = flag;
    DBG(tdec += NOW() - ta; ++cntu; STAMPV(P.ws, 8, tdec); STAMPV(P.ws, 9, 0); STAMPV(P.ws, 10, cntu);)
  }
}

__device__ __forceinline__ unsigned probe_times(const Params& P, int wid) {
  unsigned best[2] = {0xffffu, 0xffffu};
#pragma unroll
  for (int k = 0; k < 5; ++k) {
    const int cls = (k ^ wid) & 1;
    unsigned t = 0;
    unsigned long long t0, t1, keep;
    asm volatile("s_mov_b64 %3, exec\n\ts_mov_b64 exec, 1\n\ts_memtime %1\n\ts_waitcnt lgkmcnt(0)\n\tglobal_atomic_or %0, %4, %0, off sc0 sc1\n\ts_waitcnt vmcnt(0)\n\t"
                 "s_memtime %2\n\ts_mov_b64 exec, %3\n\ts_waitcnt lgkmcnt(0)"
                 : "+v"(t), "=&s"(t0), "=&s"(t1), "=&s"(keep) : "v"((const char*)P.out + cls * 8192 + wid * 256) : "memory");
    const unsigned d = (unsigned)(t1 - t0);
    if (k && d < best[0] && cls == 0) best[0] = d;
    if (k && d < best[1] && cls == 1) best[1] = d;
  }
  return best[0] | best[1] << 16;
}
__device__ __forceinline__ void probe_publish(const Params& P, const LAS unsigned* times, int b, int lane, unsigned call) {
  unsigned xcc, m0 = 0xffffu, m1 = 0xffffu;
  asm volatile("s_getreg_b32 %0, hwreg(HW_REG_XCC_ID)" : "=s"(xcc));
  xcc &= 7;
#pragma unroll
  for (int w = 0; w < 8; ++w) { const unsigned t = times[w]; m0 = min(m0, t & 0xffffu); m1 = min(m1, t >> 16); }
  const unsigned pol = m0 + 100 < m1 ? xcc >> 2 : m1 + 100 < m0 ? (xcc >> 2) ^ 1 : 2;
  if (lane == 0) st4_sys(P.ws + OFF_MAP, call << 8 | ((xcc - (unsigned)b) & 7) << 2 | pol);
  DBG(if (lane == 0) st4_sys(P.ws + OFF_MAP + 4, m0 | m1 << 16);)
}

typedef float f32x2 __attribute__((ext_vector_type(2)));
__device__ __forceinline__ f32x2 gate2(f32x2 a, float ke, float ys) {
  constexpr float LIM = 60.f * 1.44269504088896341f, EPS = 0x1p-24f;
  const f32x2 t = a * ke;
  const f32x2 e = {__builtin_amdgcn_exp2f(__builtin_amdgcn_fmed3f(t.x, -LIM, LIM)), __builtin_amdgcn_exp2f(__builtin_amdgcn_fmed3f(t.y, -LIM, LIM))};
  const f32x2 d = e * EPS + EPS;
  const float r = __builtin_amdgcn_rcpf(d.x * d.y) * ys;
  const f32x2 sw = {d.y, d.x};
  return sw * r;
}

#define PFT 1
#define PFA 1
#define CODE_MARK(name, id) asm volatile(name "%c0:" ::"i"(id))
#define CODE_RSRC(from, to, id) ({ unsigned long long pc_; int o_; unsigned n_; \
    asm volatile("s_getpc_b64 %0\n1:\n\ts_mov_b32 %1, " from "%c3-1b\n\ts_mov_b32 %2, " to "%c3-" from "%c3" : "=s"(pc_), "=s"(o_), "=s"(n_) : "i"(id)); \
    __builtin_amdgcn_make_buffer_rsrc((void*)(pc_ + o_), 0, n_ < (1u << 20) ? n_ : 0u, 0x00020000); })
#define CODE_FETCH(dst, from, to, id, off) do { if (PFT) { const rsrc_t rc_ = CODE_RSRC(from, to, id); \
    asm volatile("buffer_load_dword %0, %1, %2, 0 offen" : "+v"(dst) : "v"(off), "s"(rc_) : "memory"); } } while (0)
#define SPREAD TAILS
constexpr int ufirst(int ph, int nu) { return ph >= 2 * SPREAD ? nu : (ph * nu + 2 * SPREAD - 1) / (2 * SPREAD); }
template <int I, int E, typename F> __device__ __forceinline__ void static_for(F&& f) {
  if constexpr (I < E) { f(std::integral_constant<int, I>{}); static_for<I + 1, E>(f); }
}

template <int MA0, int MA1>
__device__ __forceinline__ void gemm_main(const Params& P, char* shm, int browA, int browB, int bcol, int wid, int lane, unsigned call) {
  constexpr int RA0 = 32 * MA0, RA1 = 32 * MA1;
  constexpr int SLA0 = RA0 * 16, SLA1 = RA1 * 16, SLB = 128 * 16;
  constexpr int A0B = 8 * SLA0, A1B = 8 * SLA1, BB = 8 * SLB, BUFB = A0B + A1B + 2 * BB;
  constexpr int NA0 = (RA0 + 63) / 64, NA1 = (RA1 + 63) / 64;
  constexpr int S = TAILS, J = JSTEP, END = NSTEP + S;
  static_assert(2 * BUFB <= LDS_BYTES && S % 2 == 0 && S >= 4 && J >= 8, "shape");
  const int wr = wid >> 2, wc = wid & 3, fr = lane & 15, fq = lane >> 4;
  const unsigned par = call & 1;
  auto rX = __builtin_amdgcn_make_buffer_rsrc((void*)(P.ws + OFF_GX), 0, GSTEPS * STEPB, 0x00020000);
  auto rW = __builtin_amdgcn_make_buffer_rsrc((void*)(P.ws + OFF_GW), 0, GSTEPS * STEPB, 0x00020000);
  auto rU = __builtin_amdgcn_make_buffer_rsrc((void*)P.ws, 0, OFF_DBG, 0x00020000);
  auto rC = __builtin_amdgcn_make_buffer_rsrc((void*)P.out, 0, DIM * DIM * 4, 0x00020000);
  const int voff = lane * 16;
  const int sA0 = wid * SLAB + browA * 16, sA1 = wid * SLAB + browB * 16, sB = wid * SLAB + bcol * 16;
  constexpr unsigned M0 = 0x13, M1 = 0x0c, MALL = 0x1f, MBAD = 0x20;
  int pollv;
  { const int tg = lane == 0 ? browA >> 9 : lane == 1 ? (browA + RA0 - 1) >> 9 : lane == 2 ? browB >> 9 : lane == 3 ? (browB + RA1 - 1) >> 9 : 8 + (bcol >> 9);
    pollv = lane == 5 ? OFF_TMO : OFF_UF + (wid * 16 + tg) * 128; }
  unsigned pv = 0, got = 0;
  bool dead = false;
  const float scale = P.sx[0] * P.sw[0];
  int kei, ysi, isi;
  asm volatile("s_nop 1\n\tv_readfirstlane_b32 %0, %1" : "=s"(isi) : "v"(1.0f / scale));
  asm volatile("s_nop 1\n\tv_readfirstlane_b32 %0, %2\n\tv_readfirstlane_b32 %1, %3" : "=s"(kei), "=s"(ysi) : "v"(scale * -1.44269504088896341f), "v"(scale * 0x1p-24f));
  int t4 = 0;
#define POLL(step) asm volatile("buffer_load_dword %0, %1, %2, %3 offen offset:%4 sc0 sc1" : "=v"(pv) : "v"(pollv), "s"(rU), "s"(t4), "i"((step) * 4) : "memory")
#define TAKE() do { asm volatile("" : "+v"(pv)); got = (unsigned)__ballot(pv >= call); } while (0)
#define NEED(step, m) do { if ((got & (m)) != (m) && !dead) { const unsigned long long t0_ = __builtin_amdgcn_s_memrealtime(); \
    while (true) { unsigned sv_;     \
      asm volatile("buffer_load_dword %0, %1, %2, %3 offen offset:%4 sc0 sc1\n\ts_waitcnt vmcnt(0)" : "=&v"(sv_) : "v"(pollv), "s"(rU), "s"(t4), "i"((step) * 4) : "memory"); \
      got = (unsigned)__ballot(sv_ >= call); if ((got & (m)) == (m)) break; \
      if (got & MBAD) { dead = true; break; } \
      if (__builtin_amdgcn_s_memrealtime() - t0_ > DEADLINE) { give_up(P.ws, call); dead = true; break; } \
      __builtin_amdgcn_s_sleep(1); } \
    WAIT_L(0); } } while (0)
#define OFF_A0(b) ((b) * BUFB)
#define OFF_A1(b) ((b) * BUFB + A0B)
#define OFF_B(b, h) ((b) * BUFB + A0B + A1B + (h) * BB)
#define GL(rs, ldsoff, soff, imm) __builtin_amdgcn_raw_ptr_buffer_load_lds(rs, (LAS void*)(shm + (ldsoff)), 16, voff, soff, imm, 17)
#define STAGE_A0(b, go) do { const int so_ = sA0 + (go), lo_ = OFF_A0(b) + wid * SLA0; GL(rX, lo_, so_, 0); GL(rX, lo_, so_, 1024); \
    if (NA0 == 3) { if (lane < 32) GL(rX, lo_, so_, 2048); } } while (0)
#define STAGE_A1(b, go) do { const int so_ = sA1 + (go), lo_ = OFF_A1(b) + wid * SLA1; GL(rX, lo_, so_, 0); GL(rX, lo_, so_, 1024); \
    if (NA1 == 3) { if (lane < 32) GL(rX, lo_, so_, 2048); } } while (0)
#define STAGE_B(b, h, go) do { const int so_ = sB + (h) * 2048 + (go), lo_ = OFF_B(b, h) + wid * SLB; GL(rW, lo_, so_, 0); GL(rW, lo_, so_, 1024); } while (0)
  const LAS char* lsm = (const LAS char*)shm;
  const LAS char *pa0[2], *pa1[2], *pb[2];
#pragma unroll
  for (int b = 0; b < 2; ++b) {
    pa0[b] = lsm + OFF_A0(b) + fq * SLA0 + (wr * 16 * MA0 + fr) * 16;
    pa1[b] = lsm + OFF_A1(b) + fq * SLA1 + (wr * 16 * MA1 + fr) * 16;
    pb[b] = lsm + OFF_B(b, 0) + fq * SLB + (wc * 16 + fr) * 16;
    asm volatile("" : "+v"(pa0[b])); asm volatile("" : "+v"(pa1[b])); asm volatile("" : "+v"(pb[b]));
  }
#define LD8(p, sl) __builtin_shufflevector(*(const LAS v4i*)(p), *(const LAS v4i*)((p) + 4 * (sl)), 0, 1, 2, 3, 4, 5, 6, 7)
#define LDA0(b) _Pragma("unroll") for (int m = 0; m < MA0; ++m) At[m] = LD8(pa0[b] + m * 256, SLA0)
#define LDA1(b) _Pragma("unroll") for (int m = 0; m < MA1; ++m) At[m] = LD8(pa1[b] + m * 256, SLA1)
#define LDB(dst, b, h) _Pragma("unroll") for (int n = 0; n < 2; ++n) dst[n] = LD8(pb[b] + (h) * BB + n * 1024, SLB)
#define MFMA(c, a, b) asm volatile("v_mfma_f32_16x16x128_f8f6f4 %0, %1, %2, %0 blgp:1" : "+v"(c) : "v"(a), "v"(b))
#define MMA0(bj, Bf) do { __builtin_amdgcn_s_setprio(1); _Pragma("unroll") for (int m = 0; m < MA0; ++m) _Pragma("unroll") for (int n = 0; n < 2; ++n) \
    MFMA(acc0[m][(bj) * 2 + n], At[m], Bf[n]); __builtin_amdgcn_s_setprio(0); } while (0)
#define MMA1(bj, Bf) do { __builtin_amdgcn_s_setprio(1); _Pragma("unroll") for (int m = 0; m < MA1; ++m) _Pragma("unroll") for (int n = 0; n < 2; ++n) \
    MFMA(acc1[m][(bj) * 2 + n], At[m], Bf[n]); __builtin_amdgcn_s_setprio(0); } while (0)
#define TILE(b, nb, t, r, X, NX, PX) do {     \
     TAKE(); NEED((r) + 1, MALL); const int go_ = gstep_off((t) + 1, par); \
    LDB(B0, b, 0); LDA0(b); STAGE_B(nb, 0, go_); STAGE_A0(nb, go_); POLL((r) + 2); \
    WAIT_V(NA1 + PX + 2 + NA0 + 1); BAR; WAIT_L(0); MMA0(0, B0); BAR; SCHED; \
    LDB(B1, b, 1); STAGE_B(nb, 1, go_); STAGE_A1(nb, go_); \
    WAIT_V(PX + 2 + NA0 + 1 + 2 + NA1); BAR; WAIT_L(0); MMA0(1, B1); BAR; SCHED; \
    LDA1(b); X; BAR; WAIT_L(0); MMA1(0, B0); BAR; SCHED; \
    WAIT_V(2 + NA1 + NX); BAR; MMA1(1, B1); BAR; SCHED; } while (0)
#define OUTPOL "sc1"
#define ST_ROW(o, row) do { const int so_ = ((row) * DIM + bcol + wc * 64) * 4; \
    asm volatile("buffer_store_dwordx4 %0, %1, %2, %3 offen " OUTPOL "\n\ts_nop 1" ::"v"(o), "v"(cv), "s"(rC), "s"(so_) : "memory"); } while (0)
#define EARLYPOL OUTPOL
#define ST_ROWP(o, row, POL, vo) do { const int so_ = ((row) * DIM + bcol + wc * 64) * 4; \
    asm volatile("buffer_store_dwordx4 %0, %1, %2, %3 offen " POL "\n\ts_nop 1" ::"v"(o), "v"(vo), "s"(rC), "s"(so_) : "memory"); } while (0)
#define ST_PAIR(acc, j, row0) do { f32x4 o0_, o1_; \
    _Pragma("unroll") for (int u = 0; u < 4; ++u) { const f32x2 a_ = {acc[u][j], acc[u][(j) + 1]}, g_ = gate2(a_, __builtin_bit_cast(float, kei), __builtin_bit_cast(float, ysi)); \
      float p_, q_; asm("v_mul_f32 %0, %1, %2" : "=v"(p_) : "v"(a_.x), "v"(g_.x)); asm("v_mul_f32 %0, %1, %2" : "=v"(q_) : "v"(a_.y), "v"(g_.y)); \
      o0_[u] = p_; o1_[u] = q_; } \
    ST_ROW(o0_, (row0) + (j)); ST_ROW(o1_, (row0) + (j) + 1); } while (0)
  f32x4 acc0[MA0][4] = {}, acc1[MA1][4] = {};
  v8i At[MA0 > MA1 ? MA0 : MA1], B0[2], B1[2];
  NEED(0, MALL);
  STAMP(P.ws, 2);
  { const int go = gstep_off(0, par);
    STAGE_B(0, 0, go); STAGE_A0(0, go); POLL(1); STAGE_B(0, 1, go); STAGE_A1(0, go); }
  if (wr == 1) BAR;
  WAIT_V(2 + NA1); BAR; BAR; SCHED;
  for (int t = 0; t < J - 2; t += 2) {
    asm volatile("s_lshl_b32 %0, %1, 2" : "=s"(t4) : "s"(t) : "scc");
    TILE(0, 1, t, 0, , 0, 0);
    TILE(1, 0, t + 1, 1, , 0, 0);
  }
  t4 = 0;
  unsigned cdum = 0;
  TILE(0, 1, J - 2, J - 2, CODE_FETCH(cdum, "fk_tail", "fk_tend", MA0 * 10 + MA1, (wid * 64 + lane) * 64), PFT, 0);
  CODE_MARK("fk_tail", MA0 * 10 + MA1);
  {
    TAKE(); NEED(J, M0);
    LDB(B0, 1, 0); LDA0(1); STAGE_B(0, 0, (J + PSTEPS) * STEPB); STAGE_A0(0, (J + PSTEPS) * STEPB); POLL(J + 1);
    WAIT_V(NA1 + PFT + 2 + NA0 + 1); BAR; WAIT_L(0); MMA0(0, B0); BAR; SCHED;
    LDB(B1, 1, 1); STAGE_B(0, 1, (J + PSTEPS) * STEPB);
    WAIT_V(PFT + 2 + NA0 + 1 + 2); BAR; WAIT_L(0); MMA0(1, B1); BAR; SCHED;
    LDA1(1); BAR; WAIT_L(0); MMA1(0, B0); BAR; SCHED;
    WAIT_V(2); asm volatile("" : "+v"(cdum)); TAKE(); NEED(J + 1, M0); POLL(J + 2); STAGE_B(1, 0, (J + 1 + PSTEPS) * STEPB); STAGE_A0(1, (J + 1 + PSTEPS) * STEPB);
    BAR; MMA1(1, B1); BAR; SCHED; }
  int tz = threadIdx.x; asm volatile("" : "+v"(tz));
  const int cv = (((tz & 63) >> 4) * 4 * DIM + (tz & 15) * 4) * 4;
  float lim = 60.f * 1.44269504088896341f; asm volatile("" : "+v"(lim));
  f32x4 o0, o1;
#define PREOPS 3
  constexpr int VMAX = 2 * ((8 * MA0 + 2 * SPREAD - 1) / (2 * SPREAD));
  float tv[VMAX];
  auto epi_op = [&](auto opc, auto uc, auto vc) {
    constexpr int op = decltype(opc)::value, u = decltype(uc)::value, v = decltype(vc)::value;
    const float a = acc0[u >> 3][u & 3][(u >> 2 & 1) * 2 + (v & 1)];
    float& t = tv[v];
    if constexpr (op == 0) asm volatile("v_mul_f32 %0, %1, %2" : "=v"(t) : "s"(kei), "v"(a));
    if constexpr (op == 1) asm volatile("v_med3_f32 %0, %0, %1, %2" : "+v"(t) : "s"(0xc2ad1f97), "v"(lim));
    if constexpr (op == 2) asm volatile("v_exp_f32 %0, %0" : "+v"(t));
    if constexpr (op == 3) asm volatile("v_fma_f32 %0, %0, %1, %1" : "+v"(t) : "s"(isi));
    if constexpr (op == 4) asm volatile("v_rcp_f32 %0, %0" : "+v"(t));
    if constexpr (op == 5) {
      float r; asm volatile("v_mul_f32 %0, %1, %2" : "=v"(r) : "v"(a), "v"(t));
      if constexpr (v & 1) o1[u & 3] = r; else o0[u & 3] = r;
      if constexpr ((u & 3) == 3 && (v & 1)) {
        const int row = vrow(browA + wr * 16 * MA0 + (u >> 3) * 16) + (u >> 2 & 1) * 2;
        ST_ROWP(o0, row, EARLYPOL, cv); ST_ROWP(o1, row + 1, EARLYPOL, cv);
      }
    }
  };
  auto pre = [&](auto phc) {
    constexpr int ph = decltype(phc)::value, u0 = ufirst(ph, 8 * MA0), V = 2 * (ufirst(ph + 1, 8 * MA0) - u0);
    static_for<0, PREOPS * V>([&](auto kc) {
      constexpr int k = decltype(kc)::value;
      epi_op(std::integral_constant<int, k / V>{}, std::integral_constant<int, u0 + (k % V >> 1)>{}, std::integral_constant<int, k % V>{});
    });
  };
  auto cluster = [&](auto phc, auto bjc, v8i (&Bf)[2]) {
    constexpr int ph = decltype(phc)::value, bj = decltype(bjc)::value, G = 2 * MA1;
    constexpr int u0 = ufirst(ph, 8 * MA0), V = 2 * (ufirst(ph + 1, 8 * MA0) - u0);
    constexpr auto cost = [](int k) { const int op = k / (V ? V : 1), v = k % (V ? V : 1);
      return 4 * k + 4 * (op > 2 ? V : op == 2 ? v : 0) + 4 * (op > 4 ? V : op == 4 ? v : 0); };
    __builtin_amdgcn_s_setprio(1);
    static_for<0, G>([&](auto gc) {
      constexpr int g = decltype(gc)::value;
      MFMA(acc1[g >> 1][bj * 2 + (g & 1)], At[g >> 1], Bf[g & 1]);
      static_for<PREOPS * V, 6 * V>([&](auto kc) {
        constexpr int k = decltype(kc)::value;
        if constexpr ((cost(k) - cost(PREOPS * V)) * (G - 1) / (32 * V - cost(PREOPS * V)) == g)
          epi_op(std::integral_constant<int, k / V>{}, std::integral_constant<int, u0 + (k % V >> 1)>{}, std::integral_constant<int, k % V>{});
      });
    });
    __builtin_amdgcn_s_setprio(0);
  };
  unsigned pw[2] = {pv, pv};
  static_for<J, END>([&](auto ic) {
    constexpr int i = decltype(ic)::value, b = i & 1, nb = b ^ 1;
    constexpr bool h = i >= NSTEP;
    constexpr int MA = h ? MA1 : MA0;
    constexpr auto step = [](int k) { return k < NSTEP ? k : k - S; };
    constexpr auto nst = [](int k, int q) {
      const int ph = 2 * (k - NSTEP) + q;
      return ph < 0 ? 0 : 2 * (ufirst(ph + 1, 8 * MA0) / 4 - ufirst(ph, 8 * MA0) / 4); };
    constexpr auto nq1 = [](int k) { return (k + 1 < END ? 2 : 0) + (k + 3 < END ? 1 : 0); };
    constexpr auto nq2 = [](int k) { return k + 2 < END ? 2 + (k + 2 < NSTEP ? NA0 : NA1) : 0; };
    const int g1 = sB + 2048 + (step(i + 1) + PSTEPS) * STEPB, l1 = OFF_B(nb, 1) + wid * SLB;
    if constexpr (i + 1 < END) GL(rW, l1, g1, 0);
    SCHED; LDB(B0, b, 0); SCHED;
    if constexpr (i + 1 < END) GL(rW, l1, g1, 1024);
    SCHED;
    static_for<0, MA>([&](auto mc) {
      constexpr int m = decltype(mc)::value;
      if constexpr (h) At[m] = LD8(pa1[b] + m * 256, SLA1); else At[m] = LD8(pa0[b] + m * 256, SLA0);
      if constexpr (m == 1 && i + 3 < END) {
        SCHED; asm volatile("buffer_load_dword %0, %1, %2, 0 offen offset:%3 sc0 sc1" : "=v"(pw[b]) : "v"(pollv), "s"(rU), "i"(step(i + 3) * 4) : "memory"); SCHED; }
    });
    WAIT_V(nst(i - 1, 0) + nq2(i - 1) + nst(i - 1, 1) + nq1(i));
    asm volatile("" : "+v"(pw[nb])); got = (unsigned)__ballot(pw[nb] >= call);
    if constexpr (i == NSTEP) {
      STAMP(P.ws, 39);
      if (dead || (got & MBAD)) isi = 0x7fc00000;
      asm volatile("s_nop 15\n\ts_nop 15"); SCHED;
    }
    if constexpr (h) pre(std::integral_constant<int, 2 * (i - NSTEP)>{});
    WAIT_L(0); BAR;
    if constexpr (h) cluster(std::integral_constant<int, 2 * (i - NSTEP)>{}, std::integral_constant<int, 0>{}, B0); else MMA0(0, B0);
    BAR; SCHED;
    if constexpr (i + 2 < END) {
      constexpr bool h2 = i + 2 >= NSTEP;
      NEED(step(i + 2), (h2 ? M1 : M0));
      const int go = (step(i + 2) + PSTEPS) * STEPB;
      const int ga = (h2 ? sA1 : sA0) + go, la = h2 ? OFF_A1(b) + wid * SLA1 : OFF_A0(b) + wid * SLA0, gb = sB + go, lb = OFF_B(b, 0) + wid * SLB;
      GL(rX, la, ga, 0); SCHED;
      B1[0] = LD8(pb[b] + BB, SLB); SCHED;
      GL(rX, la, ga, 1024); SCHED;
      B1[1] = LD8(pb[b] + BB + 1024, SLB); SCHED;
      if ((h2 ? NA1 : NA0) == 3) { if (lane < 32) GL(rX, la, ga, 2048); }
      GL(rW, lb, gb, 0); GL(rW, lb, gb, 1024);
    } else {
      LDB(B1, b, 1);
    }
    if constexpr (i + 1 < END) WAIT_V(nst(i - 1, 1) + nq1(i) + nst(i, 0) + nq2(i));
    if constexpr (h) pre(std::integral_constant<int, 2 * (i - NSTEP) + 1>{});
    WAIT_L(0); BAR;
    if constexpr (h) cluster(std::integral_constant<int, 2 * (i - NSTEP) + 1>{}, std::integral_constant<int, 1>{}, B1); else MMA0(1, B1);
    if constexpr (i + 1 < END) { BAR; SCHED; }
  });
  asm volatile("s_nop 15\n\ts_nop 15"); SCHED;
  if (wr == 0) BAR;
  STAMP(P.ws, 3);
  POLL(0); WAIT_V(0); TAKE();
  if (dead || (got & MBAD)) kei = ysi = 0x7fc00000;
#pragma unroll
  for (int m = 0; m < MA1; ++m) { const int row = vrow(browB + wr * 16 * MA1 + m * 16); ST_PAIR(acc1[m], 0, row); ST_PAIR(acc1[m], 2, row); }
  CODE_MARK("fk_tend", MA0 * 10 + MA1);
}

__global__ __launch_bounds__(512, 2) void fused_kernel(Params P) {
  extern __shared__ __attribute__((aligned(16))) char shm[];
  const int b = blockIdx.x;
  const int wid = __builtin_amdgcn_readfirstlane(threadIdx.x >> 6), lane = threadIdx.x & 63;
  CODE_MARK("fk_all", 0);
  STAMP(P.ws, 0);
  const unsigned cnt = __hip_atomic_load((unsigned*)(P.ws + OFF_CNT), __ATOMIC_RELAXED, __HIP_MEMORY_SCOPE_AGENT);
  const int j = b - NCOMM, ptensor = (b >> 3) & 1, pg = b & 7;
  unsigned call;
  if (b < NCOMM) {
    u32x4 r[8];
    if (wid == 7) prep_load(P, b >> 4, ptensor, pg, lane, r);
    if (PFA && wid >= 2 && wid < 6)
      __builtin_amdgcn_raw_ptr_buffer_load_lds(CODE_RSRC("fk_all", "fk_aend", 0), (LAS void*)(shm + RING_CTL + 512 + (wid - 2) * 256), 4,
                                               (((b >> 3) * 4 + wid - 2) * 64 + lane) * 128, 0, 0, 0);
    if (wid == 0) *(LAS u32x2*)((LAS char*)shm + RING_CTL + lane * 8) = (u32x2){0u, (lane & 31) == 0 ? 0xffffffffu : 0u};
    call = (unsigned)__builtin_amdgcn_readfirstlane((int)cnt) / NBLK + 1;
    if (wid == 7 && ptensor) prep_transpose(r);
    WAIT_L(0); BAR;
    STAMP(P.ws, 6);
    if (wid == 7) prep_push(P, b >> 4, ptensor, pg, lane, call, r);
    else if (wid >= 2) {
      tab_copy(TABS.dec, shm, DEC_KB, wid - 2, 5, lane);
      WAIT_V(0);
      if (lane == 0) asm volatile("ds_add_u32 %0, %1" ::"v"((unsigned)(size_t)(LAS char*)shm + RING_CTL + 12), "v"(1u) : "memory");
    }
  } else {
    constexpr int ENC = 2 * PU_BYTES;
    u32x4 d[4], d2[4], r[2], r2[2];
    int c0 = RAWC + 2 * (j >> 4);
    if (j == 200) {
      *(LAS unsigned*)((LAS char*)shm + wid * 4) = probe_times(P, wid);
      WAIT_L(0); BAR; asm volatile("" ::: "memory");
      if (wid == 0) probe_publish(P, (const LAS unsigned*)shm, b, lane, (unsigned)__builtin_amdgcn_readfirstlane((int)cnt) / NBLK + 1);
      WAIT_L(0); BAR;
    }
    prep_load2(P, c0 + (wid >> 2), ptensor, pg, wid & 3, lane, d);
    if (c0 == 28) prep_load2(P, 30 + (wid >> 2), ptensor, pg, wid & 3, lane, d2);
    tab_copy(TABS.enc, shm + ENC, ENC_KB, wid, 8, lane);
    call = (unsigned)__builtin_amdgcn_readfirstlane((int)cnt) / NBLK + 1;
    WAIT_V(0);
    STAMP(P.ws, 6);
    prep_pieces2(ptensor, wid & 3, d, r);
    prep_raw(P, c0, ptensor, pg, wid, lane, call, r);
    if (c0 == 28) {
      prep_pieces2(ptensor, wid & 3, d2, r2);
      prep_raw(P, 30, ptensor, pg, wid, lane, call, r2);
    }
    while (true) {
      prep_pack(P, (LAS char*)shm, (const LAS char*)shm + ENC, c0, ptensor, pg, wid, lane, call, r);
      if (c0 != 28) break;
      c0 = 30; r[0] = r2[0]; r[1] = r2[1];
    }
  }
  STAMP(P.ws, 1);
  if (b < NCOMM) {
    if (wid < 2) loader_main(P, shm, wid, b * 2 + wid, lane, call);
    else decoder_main(P, shm, wid & 1, lane, call);
    if (threadIdx.x == 0) __hip_atomic_fetch_add((unsigned*)(P.ws + OFF_CNT), 1u, __ATOMIC_RELAXED, __HIP_MEMORY_SCOPE_AGENT);
    STAMP(P.ws, 5);
    CODE_MARK("fk_aend", 0);
    return;
  }
  WAIT_V(0);
  unsigned map = (unsigned)__builtin_amdgcn_readfirstlane((int)ld4_sys(P.ws + OFF_MAP));
  if ((map >> 8) < call) {
    const unsigned long long t0 = __builtin_amdgcn_s_memrealtime();
    do {
      if (spin_fail(P.ws, call, t0)) break;
      map = (unsigned)__builtin_amdgcn_readfirstlane((int)ld4_sys(P.ws + OFF_MAP));
    } while ((map >> 8) < call);
  }
  const int xcd = (b + (map >> 2)) & 7, idx = j >> 3, par = xcd & 1, side = xcd >> 2;
  const int chalf = (map & 2) ? side : (int)(par ^ side ^ (map & 1));
  const int rb = idx % 7, bcol = (chalf * 8 + (xcd & 2) * 2 + idx / 7) * 256;
  const int browA = par * 1024 + (rb <= 4 ? rb * 160 : 128 + rb * 128);
  const int browB = 2048 + par * 1024 + (rb == 0 ? 0 : rb <= 4 ? 32 + rb * 128 : rb * 160 - 96);
  if (rb == 0) gemm_main<5, 5>(P, shm, browA, browB, bcol, wid, lane, call);
  else if (rb < 4) gemm_main<5, 4>(P, shm, browA, browB, bcol, wid, lane, call);
  else gemm_main<4, 5>(P, shm, browA, browB, bcol, wid, lane, call);
  if (threadIdx.x == 0) __hip_atomic_fetch_add((unsigned*)(P.ws + OFF_CNT), 1u, __ATOMIC_RELAXED, __HIP_MEMORY_SCOPE_AGENT);
}

}

extern "C" void kernel_launch(int rank, int nranks, ncclComm_t comm, hipStream_t stream, void* const* d_in, const int* in_sizes, int n_in,
                              void* d_out, int out_size, const int* in_dtypes, int out_dtype, void* const* d_out_all,
                              void* const* d_ws_all, size_t ws_bytes) {
  if (nranks != NR || n_in != 4 || in_sizes[0] != DIM * KSH || in_sizes[1] != KSH * DIM || out_size != DIM * DIM || ws_bytes < (size_t)WS_NEED) {
    fprintf(stderr, "unexpected problem shape\n");
    return;
  }
  constexpr int lds = LDS_BYTES;
  HIP_CHECK(hipFuncSetAttribute((const void*)fused_kernel, hipFuncAttributeMaxDynamicSharedMemorySize, lds));
  Params P;
  P.x = (const unsigned char*)d_in[0];
  P.w = (const unsigned char*)d_in[1];
  P.sx = (const float*)d_in[2];
  P.sw = (const float*)d_in[3];
  P.out = (float*)d_out;
  P.ws_all = (char* const*)d_ws_all;
  P.ws = (char*)d_ws_all[rank];
  P.rank = rank;
  P.pad = 0;
  hipLaunchKernelGGL(fused_kernel, dim3(NBLK), dim3(512), lds, stream, P);
  HIP_CHECK(hipGetLastError());
}
```
